```python
import numpy as np

kernel_name = "gpu_gemm_rs_m4096_k4096_n4096_bf16_qe4m3"


ABSMAX_RTOL = 0.09

COMPUTE_DTYPE = "bfloat16"
EPILOGUE = "quant_e4m3"
IN_QUANT = None
OUT_QUANT = "e4m3"
IN_DTYPES = {"x": "bfloat16", "w_mat": "bfloat16"}
WORLD_SIZE = 8

def _to_bf16(x):
    u = np.ascontiguousarray(x, dtype=np.float32).view(np.uint32)
    return ((u + 0x7FFF + ((u >> 16) & 1)) & 0xFFFF0000).view(np.float32)

def setup_inputs():

    x = _to_bf16(np.random.randn(4096, 4096).astype(np.float32))
    w_mat = _to_bf16((np.random.randn(4096, 4096) / np.sqrt(4096)).astype(np.float32))
    return {"x": x, "w_mat": w_mat}

def _snap_fp8(x, mant_bits, max_abs, min_normal):
    a = np.abs(x.astype(np.float64))
    _m, _e = np.frexp(a)
    q = np.where(a >= min_normal, np.ldexp(1.0, _e - 1 - mant_bits),
                 min_normal * 2.0 ** -mant_bits)
    snapped = np.minimum(np.round(a / q) * q, max_abs)
    return (np.sign(x) * snapped).astype(np.float32)
def _quant_dequant(y, amax):
    scale = amax / 448.0
    q = _snap_fp8(y / scale, 3, 448.0, 2.0 ** -6)
    return (q * scale).astype(np.float32)
def _epilogue(y):
    return _quant_dequant(y, np.abs(y).max())

def reference(x, w_mat):
    y = (x @ w_mat).astype(np.float32)
    return _epilogue(y)
```

```cpp
#include <hip/hip_runtime.h>
#include <rccl/rccl.h>
#include <cstdio>
#include <cstdint>

#define HIP_CHECK(expr) do { hipError_t _e = (expr); if (_e != hipSuccess) { \
    fprintf(stderr, "HIP error %s at %s:%d\n", hipGetErrorString(_e), __FILE__, __LINE__); \
    return; } } while (0)

namespace grs {

typedef unsigned u32;
typedef unsigned long long u64;
typedef unsigned u32x4 __attribute__((ext_vector_type(4)));
typedef unsigned u32x2 __attribute__((ext_vector_type(2)));
typedef unsigned short u16x2 __attribute__((ext_vector_type(2)));
typedef float f32x16 __attribute__((ext_vector_type(16)));
typedef short bf16x8 __attribute__((ext_vector_type(8)));
typedef __amdgpu_buffer_rsrc_t rsrc_t;
#define LAS __attribute__((address_space(3)))
#define GAS __attribute__((address_space(1)))

constexpr int NR = 8;
constexpr int AUX_SYS = 17;
constexpr u32 RSRC_W3 = 0x00020000u;
constexpr u64 DEADLINE = 20000000ull;

constexpr u32 OFF_CALL  = 0;
constexpr u32 OFF_TMO   = 128;
constexpr u32 OFF_XFLAG = 4096;
constexpr u32 OFF_WFLAG = 16384;
constexpr u32 OFF_PFLAG = 32768;
constexpr u32 OFF_LG    = 65536;
constexpr u32 OFF_GG    = 69632;
constexpr u32 OFF_DBG   = 48u << 20;
constexpr u32 BLK = 3072, ENT = 32, RAWB = 4096;
constexpr u32 OFF_XP = 1u << 20, OFF_WP = 7u << 20, OFF_PP = 13u << 20;
constexpr u32 OFF_XE = 19u << 20, OFF_WE = OFF_XE + 65536, OFF_PE = OFF_WE + 65536;
constexpr u32 OFF_XR = 20u << 20, OFF_WR = 28u << 20, OFF_PR = 36u << 20;
constexpr u32 WS_NEED = 44u << 20;

constexpr u32 SLOT = 16384, UNITB = 8192;
constexpr u32 LDS_CTL = 4 * SLOT;
constexpr u32 LDS_BYTES = 96 * 1024;

struct Params {
    const void* x; const void* w; void* out;
    char* ws[NR];
    int rank; u32 pad;
};

__device__ __forceinline__ u32 perm(u32 s0, u32 s1, u32 sel) { return __builtin_amdgcn_perm(s0, s1, sel); }
__device__ __forceinline__ u32 rotr4(u32 v) { return __builtin_amdgcn_alignbit(v, v, 4); }
__device__ __forceinline__ u32 rotl4(u32 v) { return __builtin_amdgcn_alignbit(v, v, 28); }
__device__ __forceinline__ u32 rfl(u32 v) { return (u32)__builtin_amdgcn_readfirstlane((int)v); }
__device__ __forceinline__ u32 rdl(u32 v, u32 l) { return (u32)__builtin_amdgcn_readlane((int)v, (int)l); }
__device__ __forceinline__ u32 pkmax(u32 a, u32 b) {
    return __builtin_bit_cast(u32, __builtin_elementwise_max(__builtin_bit_cast(u16x2, a), __builtin_bit_cast(u16x2, b)));
}
__device__ __forceinline__ u32 wave_umax(u32 v) {
    v = max(v, (u32)__builtin_amdgcn_update_dpp(0, (int)v, 0x111, 0xf, 0xf, true));
    v = max(v, (u32)__builtin_amdgcn_update_dpp(0, (int)v, 0x112, 0xf, 0xf, true));
    v = max(v, (u32)__builtin_amdgcn_update_dpp(0, (int)v, 0x114, 0xf, 0xf, true));
    v = max(v, (u32)__builtin_amdgcn_update_dpp(0, (int)v, 0x118, 0xf, 0xf, true));
    v = max(v, (u32)__builtin_amdgcn_update_dpp(0, (int)v, 0x142, 0xa, 0xf, false));
    v = max(v, (u32)__builtin_amdgcn_update_dpp(0, (int)v, 0x143, 0xc, 0xf, false));
    return rdl(v, 63);
}
__device__ __forceinline__ void drain() { asm volatile("s_waitcnt vmcnt(0)" ::: "memory"); }
__device__ __forceinline__ void cfence() { asm volatile("" ::: "memory"); }
__device__ __forceinline__ rsrc_t mk_rsrc(const void* p, u32 bytes) { return __builtin_amdgcn_make_buffer_rsrc((void*)p, 0, (int)bytes, RSRC_W3); }
#define EXP_LDAUX AUX_SYS
__device__ __forceinline__ u32x4 ld16(rsrc_t r, u32 voff, u32 soff) { return __builtin_amdgcn_raw_buffer_load_b128(r, (int)voff, (int)soff, EXP_LDAUX); }
__device__ __forceinline__ u32 ld4(rsrc_t r, u32 voff, u32 soff) { return __builtin_amdgcn_raw_buffer_load_b32(r, (int)voff, (int)soff, AUX_SYS); }
__device__ __forceinline__ void st16(rsrc_t r, u32 voff, u32 soff, u32x4 v) { __builtin_amdgcn_raw_buffer_store_b128(v, r, (int)(voff + soff), 0, AUX_SYS); }
__device__ __forceinline__ void st4(rsrc_t r, u32 voff, u32 soff, u32 v) { __builtin_amdgcn_raw_buffer_store_b32(v, r, (int)voff, (int)soff, AUX_SYS); }
__device__ __forceinline__ u32 sys_load(const char* p) { return __hip_atomic_load((const GAS u32*)p, __ATOMIC_RELAXED, __HIP_MEMORY_SCOPE_SYSTEM); }
__device__ __forceinline__ u64 sys_load64(const char* p) { return __hip_atomic_load((const GAS u64*)p, __ATOMIC_RELAXED, __HIP_MEMORY_SCOPE_SYSTEM); }
__device__ __forceinline__ void sys_store(char* p, u32 v) { __hip_atomic_store((GAS u32*)p, v, __ATOMIC_RELAXED, __HIP_MEMORY_SCOPE_SYSTEM); }
__device__ __forceinline__ void sys_store64(char* p, u64 v) { __hip_atomic_store((GAS u64*)p, v, __ATOMIC_RELAXED, __HIP_MEMORY_SCOPE_SYSTEM); }
__device__ __forceinline__ u32 agent_load(const char* p) { return __hip_atomic_load((const GAS u32*)p, __ATOMIC_RELAXED, __HIP_MEMORY_SCOPE_AGENT); }
__device__ __forceinline__ void agent_store(char* p, u32 v) { __hip_atomic_store((GAS u32*)p, v, __ATOMIC_RELAXED, __HIP_MEMORY_SCOPE_AGENT); }

#define STAMPC(idx) do { } while (0)
#define STAMP(idx) do { } while (0)

__device__ __forceinline__ u32 poll_sys(const char* p) { u32 x; asm volatile("global_load_dword %0, %1, off sc0 sc1\n\ts_waitcnt vmcnt(0)" : "=&v"(x) : "v"(p) : "memory"); return x; }
__device__ __forceinline__ u32 poll_agent(const char* p) { u32 x; asm volatile("global_load_dword %0, %1, off sc1\n\ts_waitcnt vmcnt(0)" : "=&v"(x) : "v"(p) : "memory"); return x; }
__device__ __forceinline__ void ld16x2_now(const char* p0, const char* p1, u32x4& a, u32x4& b) {
    asm volatile("global_load_dwordx4 %0, %2, off sc0 sc1\n\tglobal_load_dwordx4 %1, %3, off sc0 sc1\n\ts_waitcnt vmcnt(0)" : "=&v"(a), "=&v"(b) : "v"(p0), "v"(p1) : "memory");
}
__device__ __forceinline__ void give_up(char* my, u32 call) {
    asm volatile("global_store_dword %0, %1, off sc1" :: "v"(my + OFF_TMO), "v"(call) : "memory");
}
__device__ __forceinline__ bool wait_tags(const char* addr, u32 tag, u32& v, char* my, u32 call) {
    const u64 t0 = __builtin_amdgcn_s_memrealtime();
    for (u32 it = 0;; ++it) {
        const u32 x = poll_sys(addr);
        if (__all((x >> 11) == tag)) { v = x; return true; }
        if ((it & 15u) == 15u) {
            if (rfl(poll_agent(my + OFF_TMO)) == call || __builtin_amdgcn_s_memrealtime() - t0 > DEADLINE) {
                give_up(my, call); v = 0; return false; }
        }
        __builtin_amdgcn_s_sleep(2);
    }
}

__device__ __forceinline__ u32 lane_emax(const u32 (&H)[8]) {
    u32 m1 = 0, m2 = 0;
#pragma unroll
    for (int t = 0; t < 8; ++t) { const u32 g = H[t] | 0x80808080u; m1 = pkmax(m1, g); m2 = pkmax(m2, g << 8); }
    const u32 m = pkmax(m1, m2);
    return max((m >> 24) & 0x7fu, (m >> 8) & 0x7fu);
}
template <bool IS_W> __device__ __forceinline__ void chunks_from(const u32 (&L)[8], const u32 (&H)[8], u32x4 (&o)[4]) {
#pragma unroll
    for (int t = 0; t < 8; ++t) {
        const u32 e = perm(H[t], L[t], 0x05010400u), d = perm(H[t], L[t], 0x07030602u);
        if (IS_W) { o[2 * (t >> 2)][t & 3] = e; o[2 * (t >> 2) + 1][t & 3] = d; }
        else      { o[t >> 1][2 * (t & 1)] = e; o[t >> 1][2 * (t & 1) + 1] = d; }
    }
}
template <bool IS_W>
__device__ __forceinline__ u32 pack_store(const u32 (&L)[8], const u32 (&H)[8], u32 lb, rsrc_t dst, u32 blk_off, u32 ent_off, u32 raw_off, int lane) {
    const u32 lb4 = lb * 0x01010101u;
    u32 T[8], acc = 0xffffffffu;
#pragma unroll
    for (int t = 0; t < 8; ++t) { T[t] = (H[t] | 0x80808080u) - lb4; acc &= T[t]; }
    u32 n[4];
#pragma unroll
    for (int j = 0; j < 4; ++j) {
        const u32 a = (H[2 * j] & 0x80808080u) | (T[2 * j] & 0x07070707u);
        const u32 b = (H[2 * j + 1] & 0x80808080u) | (T[2 * j + 1] & 0x07070707u);
        n[j] = a | rotl4(b);
    }
    const auto s0 = __builtin_amdgcn_permlane32_swap(n[0], n[2], false, false);
    const auto s1 = __builtin_amdgcn_permlane32_swap(n[1], n[3], false, false);
    const u32 vo = (u32)lane * 16u;
    st16(dst, vo, blk_off, (u32x4){L[0], L[1], L[2], L[3]});
    st16(dst, vo, blk_off + 1024u, (u32x4){L[4], L[5], L[6], L[7]});
    st16(dst, vo, blk_off + 2048u, (u32x4){s0[0], s1[0], s0[1], s1[1]});
    u32 hdr = lb;
    const bool bad = (acc & 0x80808080u) != 0x80808080u;
    u64 bm = __ballot(bad);
    if (bm != 0ull) {
        u32 bits = 0;
#pragma unroll
        for (int t = 0; t < 8; ++t) { const u32 x = (~T[t] & 0x80808080u) >> 7;
            bits |= ((x & 1u) | ((x >> 7) & 2u) | ((x >> 14) & 4u) | ((x >> 21) & 8u)) << (4 * t); }
        const u32 nb = __popc(bits);
        u32 pos = 0, mypos = 0;
        while (bm) { const u32 l = (u32)__builtin_ctzll(bm); bm &= bm - 1; if ((u32)lane == l) mypos = pos; pos += rdl(nb, l); }
        if (pos <= 7u) {
            hdr |= pos << 7;
            while (bits) {
                const u32 idx = (u32)__builtin_ctz(bits); bits &= bits - 1;
                const u32 t = idx >> 2, by = idx & 3u;
                u32 hv = H[0];
#pragma unroll
                for (int tt = 1; tt < 8; ++tt) hv = (t == (u32)tt) ? H[tt] : hv;
                st4(dst, mypos * 4u, ent_off, ((u32)lane << 16) | (t << 10) | (by << 8) | ((hv >> (8 * by)) & 0xffu));
                ++mypos;
            }
        } else {
            hdr |= 1u << 10;
            u32x4 o[4]; chunks_from<IS_W>(L, H, o);
#pragma unroll
            for (int c = 0; c < 4; ++c) st16(dst, vo, raw_off + 1024u * c, o[c]);
        }
    }
    return hdr;
}
struct Half { u32x4 l; u32x2 n; };
__device__ __forceinline__ void high_bytes(u32x2 n, u32 hb, u32 hdr, u32 entv, u32 ebase, u32 (&H)[4], int lane) {
    const u32 lb4 = (hdr & 0x7fu) * 0x01010101u;
    H[0] = (n[0] & 0x87878787u) + lb4; H[1] = (rotr4(n[0]) & 0x87878787u) + lb4;
    H[2] = (n[1] & 0x87878787u) + lb4; H[3] = (rotr4(n[1]) & 0x87878787u) + lb4;
    const u32 cnt = (hdr >> 7) & 7u;
    for (u32 i = 0; i < cnt; ++i) {
        const u32 en = rdl(entv, ebase + i);
        const u32 sh = ((en >> 8) & 3u) * 8u, t = (en >> 10) & 7u;
        const u32 m = ((u32)lane == (en >> 16) && (t >> 2) == hb) ? (0xffu << sh) : 0u, v = (en & 0xffu) << sh;
#pragma unroll
        for (int j = 0; j < 4; ++j) { const u32 mm = ((t & 3u) == (u32)j) ? m : 0u; H[j] = (H[j] & ~mm) | (v & mm); }
    }
}

template <bool IS_W>
__device__ __forceinline__ void sender_wave(const Params& P, int d, int g, int wave, int lane, u32 tag) {
    const int di = d & 1, dj = (d >> 1) & 1, dk = d >> 2, me = P.rank;
    const int r = lane & 31, h = lane >> 5;
    const rsrc_t dst = mk_rsrc(P.ws[d], WS_NEED);
    const rsrc_t src = mk_rsrc(IS_W ? P.w : P.x, 4u << 20);
    u32 voff, soff0, sstep, blk0, bstep, flag0;
    if (!IS_W) {
        const int b = wave;
        voff = (u32)(g * 512 + di * 256 + 64 * b + r) * 1024u + 32u * h;
        soff0 = (u32)dk * 512u; sstep = 64u;
        blk0 = (u32)((me * 8) * 8 + g) * 4u + b; bstep = 32u;
        flag0 = OFF_XFLAG + (u32)(g * 8) * 128u + (u32)(b * 8 + me) * 4u;
    } else {
        const int wv = wave - 4, n = 2 * g + (wv >> 1), kf = wv & 1;
        const u32 base = 4u * (16u * ((r >> 2) & 1) + (r & 3) + 4u * (r >> 3));
        voff = (u32)(16 * h) * 8192u + base * 2u;
        soff0 = (u32)(dk * 256 + 8 * kf) * 8192u + (u32)(dj * 2048 + n * 128) * 2u; sstep = 32u * 8192u;
        blk0 = (u32)((me * 8) * 16 + n) * 2u + kf; bstep = 32u;
        flag0 = OFF_WFLAG + (u32)(n * 8) * 128u + (u32)(kf * 8 + me) * 4u;
    }
    const u32 offP = IS_W ? OFF_WP : OFF_XP, offE = IS_W ? OFF_WE : OFF_XE, offR = IS_W ? OFF_WR : OFF_XR;
    u32 in[16];
    auto load_raw = [&](int u) {
        const u32 so = soff0 + (u32)u * sstep;
        if (!IS_W) {
#pragma unroll
            for (int c = 0; c < 4; ++c) {
                const u32x4 v = __builtin_amdgcn_raw_buffer_load_b128(src, (int)(voff + (u32)(c >> 1) * 32768u + (u32)(c & 1) * 16u), (int)so, 0);
                in[4 * c] = v[0]; in[4 * c + 1] = v[1]; in[4 * c + 2] = v[2]; in[4 * c + 3] = v[3]; }
        } else {
#pragma unroll
            for (int j = 0; j < 8; ++j) {
                const u32x2 v = __builtin_amdgcn_raw_buffer_load_b64(src, (int)(voff + (u32)j * 8192u), (int)so, 0);
                in[2 * j] = v[0]; in[2 * j + 1] = v[1]; }
        }
    };
    load_raw(0);
    u32 prev_hdr = 0;
    for (int u = 0; u < 8; ++u) {
        u32 L[8], H[8];
#pragma unroll
        for (int t = 0; t < 8; ++t) {
            if (!IS_W) { L[t] = perm(in[2 * t + 1], in[2 * t], 0x06040200u); H[t] = perm(in[2 * t + 1], in[2 * t], 0x07050301u); }
            else { const int jp = t & 3, dd = t >> 2; const u32 a = in[4 * jp + dd], b2 = in[4 * jp + 2 + dd];
                   L[t] = perm(b2, a, 0x06020400u); H[t] = perm(b2, a, 0x07030501u); }
        }
        const u32 em = wave_umax(lane_emax(H));
        const u32 lb = em > 7u ? em - 7u : 0u;
        if (u > 0) {
            drain();
            if (lane == 0) sys_store(P.ws[d] + flag0 + (u32)(u - 1) * 128u, (tag << 11) | prev_hdr);
        }
        if (u < 7) load_raw(u + 1);
        const u32 blk = blk0 + (u32)u * bstep;
        prev_hdr = pack_store<IS_W>(L, H, lb, dst, offP + blk * BLK, offE + blk * ENT, offR + blk * RAWB, lane);
    }
    drain();
    if (lane == 0) sys_store(P.ws[d] + flag0 + 7u * 128u, (tag << 11) | prev_hdr);
}

__device__ __forceinline__ void lds_barrier() { asm volatile("s_waitcnt lgkmcnt(0)\n\ts_barrier" ::: "memory"); }
__device__ __forceinline__ u32 cvt_pk_bf16(float lo, float hi) { u32 r; asm("v_cvt_pk_bf16_f32 %0, %1, %2" : "=v"(r) : "v"(lo), "v"(hi)); return r; }
__device__ __forceinline__ u32x2 ld8(rsrc_t r, u32 voff, u32 soff) { return __builtin_amdgcn_raw_buffer_load_b64(r, (int)voff, (int)soff, EXP_LDAUX); }
struct StepRegs { Half xa, xb, w; u32 ev; };

__device__ __forceinline__ void gemm_wave(const Params& P, int q, int n, int w, int lane, u32 call, u32 tag, LAS unsigned char* lds, char* my, int wave) {
    const rsrc_t ws = mk_rsrc(my, WS_NEED);
    const u32 me = (u32)P.rank;
    const u32 vo = (u32)lane * 16u, von = (u32)(lane & 31) * 16u + (u32)(lane >> 5) * 8u;
    const u32 b = (u32)w >> 1, hb = (u32)w & 1u;
    const u32 wu = (u32)w >> 2, kf = ((u32)w >> 1) & 1u;
    bool ok = true;
    const u32 fo = (lane & 8) == 0 ? OFF_XFLAG + (u32)(q * 8) * 128u + (b * 8 + (u32)(lane & 7)) * 4u
                                   : OFF_WFLAG + (u32)(n * 8) * 128u + (kf * 8 + (u32)(lane & 7)) * 4u;
    const u32 elo = (u32)(lane & 7) * 4u;
    StepRegs R[4];
    auto issue = [&](StepRegs& r, u32 u, int s) {
        const u32 pa = (me + 2 * s) & 7u, pb = (me + 2 * s + 1) & 7u, pw = wu ? pb : pa;
        const u32 ba = ((pa * 8 + u) * 8 + q) * 4 + b, bb = ((pb * 8 + u) * 8 + q) * 4 + b, bw = ((pw * 8 + u) * 16 + n) * 2 + kf;
        const u32 oa = OFF_XP + ba * BLK, ob = OFF_XP + bb * BLK, ow = OFF_WP + bw * BLK;
        r.w.n = ld8(ws, von, ow + 2048u + hb * 512u);  r.w.l = ld16(ws, vo, ow + hb * 1024u);
        r.xa.n = ld8(ws, von, oa + 2048u + hb * 512u); r.xa.l = ld16(ws, vo, oa + hb * 1024u);
        r.xb.n = ld8(ws, von, ob + 2048u + hb * 512u); r.xb.l = ld16(ws, vo, ob + hb * 1024u);
        r.ev = ld4(ws, (lane < 8 ? OFF_XE + ba * ENT : (lane < 16 ? OFF_XE + bb * ENT : OFF_WE + bw * ENT)) + elo, 0);
    };
    constexpr u32 USTR = 32u * BLK;
    const u32 vl = vo + hb * 1024u, vn = von + 2048u + hb * 512u;
    u32 oA[4], oB[4], oW[4], eV[4], lA[4], lB[4], lW[4];
#pragma unroll
    for (int s = 0; s < 4; ++s) {
        const u32 pa = (me + 2 * s) & 7u, pb = (me + 2 * s + 1) & 7u, pw = wu ? pb : pa;
        const u32 ba = ((pa * 8) * 8 + q) * 4 + b, bb = ((pb * 8) * 8 + q) * 4 + b, bw = ((pw * 8) * 16 + n) * 2 + kf;
        oA[s] = OFF_XP + ba * BLK; oB[s] = OFF_XP + bb * BLK; oW[s] = OFF_WP + bw * BLK;
        eV[s] = (lane < 8 ? OFF_XE + ba * ENT : (lane < 16 ? OFF_XE + bb * ENT : OFF_WE + bw * ENT)) + elo;
        lA[s] = pa; lB[s] = pb; lW[s] = 8u + pw;
    }
    auto high4 = [&](u32x2 n, u32 hdr, u32 (&H)[4]) {
        const u32 lb4 = (hdr & 0x7fu) * 0x01010101u;
        H[0] = (n[0] & 0x87878787u) + lb4; H[1] = (rotr4(n[0]) & 0x87878787u) + lb4;
        H[2] = (n[1] & 0x87878787u) + lb4; H[3] = (rotr4(n[1]) & 0x87878787u) + lb4;
    };
    auto unpack_w = [&](const StepRegs& r, int s, u32 hdr) {
        u32x4 e, o;
        u32 H[4]; high4(r.w.n, hdr, H);
#pragma unroll
        for (int i = 0; i < 4; ++i) { e[i] = perm(H[i], r.w.l[i], 0x05010400u); o[i] = perm(H[i], r.w.l[i], 0x07030602u); }
        LAS unsigned char* p = lds + (s & 3) * SLOT + wu * UNITB + kf * 4096u + hb * 2048u + lane * 16;
        *(LAS u32x4*)p = e; *(LAS u32x4*)(p + 1024) = o;
    };
    auto unpack_x = [&](const Half& k, u32 hdr, u32x4& a, u32x4& c) {
        u32 H[4]; high4(k.n, hdr, H);
        a = (u32x4){perm(H[0], k.l[0], 0x05010400u), perm(H[0], k.l[0], 0x07030602u), perm(H[1], k.l[1], 0x05010400u), perm(H[1], k.l[1], 0x07030602u)};
        c = (u32x4){perm(H[2], k.l[2], 0x05010400u), perm(H[2], k.l[2], 0x07030602u), perm(H[3], k.l[3], 0x05010400u), perm(H[3], k.l[3], 0x07030602u)};
    };
    auto fix_x = [&](u32 hdr, u32 ev, u32 ebase, u32 u, u32 p, u32x4& a, u32x4& c) {
        if (hdr & (1u << 10)) {
            const u32 bx = ((p * 8 + u) * 8 + q) * 4 + b;
            const char* rp = my + OFF_XR + bx * RAWB + hb * 2048u + vo;
            ld16x2_now(rp, rp + 1024, a, c);
        }
        const u32 cnt = (hdr >> 7) & 7u;
        for (u32 i = 0; i < cnt; ++i) {
            const u32 en = rdl(ev, ebase + i), t = (en >> 10) & 7u, by = (en >> 8) & 3u;
            if ((t >> 2) != hb) continue;
            const u32 sh = (by & 1u) * 16u + 8u, D = 2u * (t & 3u) + (by >> 1);
            const u32 m = ((u32)lane == (en >> 16)) ? (0xffu << sh) : 0u, v = (en & 0xffu) << sh;
#pragma unroll
            for (int j = 0; j < 4; ++j) {
                const u32 ma = (D == (u32)j) ? m : 0u, mc = (D == (u32)(4 + j)) ? m : 0u;
                a[j] = (a[j] & ~ma) | (v & ma); c[j] = (c[j] & ~mc) | (v & mc); }
        }
    };
    auto fixup = [&](const StepRegs& r, u32 u, int s, u32 hw, u32 ha, u32 hc, u32x4 (&xf)[4]) {
        if (((hw | ha | hc) & 0x780u) == 0u) return;
        LAS unsigned char* p = lds + (s & 3) * SLOT + wu * UNITB + kf * 4096u + hb * 2048u;
        if (hw & (1u << 10)) {
            const u32 bw = ((((me + 2 * s + wu) & 7u) * 8 + u) * 16 + n) * 2 + kf;
            const char* rp = my + OFF_WR + bw * RAWB + hb * 2048u + vo;
            u32x4 v0, v1; ld16x2_now(rp, rp + 1024, v0, v1);
            *(LAS u32x4*)(p + lane * 16) = v0; *(LAS u32x4*)(p + 1024 + lane * 16) = v1;
        }
        const u32 cnt = (hw >> 7) & 7u;
        for (u32 i = 0; i < cnt; ++i) {
            const u32 en = rdl(r.ev, 16u + i), t = (en >> 10) & 7u, by = (en >> 8) & 3u;
            if ((t >> 2) != hb) continue;
            if (lane == 0) p[(en >> 16) * 16u + (by >> 1) * 1024u + (t & 3u) * 4u + (by & 1u) * 2u + 1u] = (unsigned char)(en & 0xffu);
        }
        fix_x(ha, r.ev, 0, u, (me + 2 * s) & 7u, xf[0], xf[1]);
        fix_x(hc, r.ev, 8, u, (me + 2 * s + 1) & 7u, xf[2], xf[3]);
        asm volatile("s_waitcnt lgkmcnt(0)" ::: "memory");
    };
    f32x16 acc[4];
#pragma unroll
    for (int c = 0; c < 4; ++c)
#pragma unroll
        for (int g = 0; g < 16; ++g) acc[c][g] = 0.f;
    u32x4 X[2][4]; bf16x8 wf[4][4];
#pragma unroll
    for (int c = 0; c < 4; ++c) { wf[2][c] = (bf16x8){0, 0, 0, 0, 0, 0, 0, 0}; wf[3][c] = wf[2][c]; X[1][c] = (u32x4){0u, 0u, 0u, 0u}; }
    LAS unsigned char* wb = lds + lane * 16;
#define GRS_MFMA(c, F, XF) acc[c] = __builtin_amdgcn_mfma_f32_32x32x16_bf16(F[c], __builtin_bit_cast(bf16x8, XF), acc[c], 0, 0, 0)
#define GRS_RD(F, s, sub) do { _Pragma("unroll") for (int c = 0; c < 4; ++c) \
        F[c] = *(const LAS bf16x8*)(wb + ((s) & 1) * SLOT + ((sub) >> 1) * UNITB + ((sub) & 1) * 4096 + c * 1024); } while (0)
#define GRS_MM(F, XF) do { _Pragma("unroll") for (int c = 0; c < 4; ++c) GRS_MFMA(c, F, XF); } while (0)
#define GRS_IL 4
#define GRS_BAR() asm volatile("s_waitcnt lgkmcnt(10)\n\ts_barrier" ::: "memory")
#define SB __builtin_amdgcn_sched_barrier(0)
#define GRS_FIX(a, b, c, d, e, f, g) fixup(a, b, c, d, e, f, g)
#define EXP_LDMASK 127
#define XL(bit, stmt) do { if ((EXP_LDMASK >> (bit)) & 1) { stmt; } } while (0)
#define RD1(s, sub, c) wf[sub][c] = *(const LAS bf16x8*)(wb + ((s) & 3) * SLOT + ((sub) >> 1) * UNITB + ((sub) & 1) * 4096 + (c) * 1024)
#define HI01(H, n) do { H[0] = (n[0] & 0x87878787u) + lb4_; H[1] = (rotr4(n[0]) & 0x87878787u) + lb4_; } while (0)
#define HI23(H, n) do { H[2] = (n[1] & 0x87878787u) + lb4_; H[3] = (rotr4(n[1]) & 0x87878787u) + lb4_; } while (0)
#define PE(H, l, i) perm(H[i], l[i], 0x05010400u)
#define PO(H, l, i) perm(H[i], l[i], 0x07030602u)
#define STEP_FENCE cfence()
#define GRS_STEP(s, ui, si, uc, hc) do { \
        constexpr int sn_ = ((s) + 1) & 3; \
        StepRegs& ri_ = R[(si) & 3]; const StepRegs& rc_ = R[sn_]; \
        u32x4 (&xo_)[4] = X[(s) & 1]; u32x4 (&xn_)[4] = X[sn_ & 1]; \
        const u32 uo_ = (ui) * USTR; \
        u32 H_[4], lb4_, hw_, ha_, hb_; u32x4 e_, o_; \
        LAS unsigned char* wp_ = wdst + sn_ * SLOT; \
        SB; \
        GRS_MFMA(0, wf[2], xn_[2]); RD1(s, 0, 0); XL(0, ri_.w.n = ld8(ws, vn, oW[si] + uo_)); hw_ = rdl(hc, lW[sn_]); lb4_ = (hw_ & 0x7fu) * 0x01010101u; SB; \
        GRS_MFMA(1, wf[2], xn_[2]); RD1(s, 0, 1); HI01(H_, rc_.w.n); SB; \
        GRS_MFMA(2, wf[2], xn_[2]); RD1(s, 0, 2); XL(1, ri_.w.l = ld16(ws, vl, oW[si] + uo_)); HI23(H_, rc_.w.n); SB; \
        GRS_MFMA(3, wf[2], xn_[2]); RD1(s, 0, 3); \
            e_[0] = PE(H_, rc_.w.l, 0); e_[1] = PE(H_, rc_.w.l, 1); e_[2] = PE(H_, rc_.w.l, 2); e_[3] = PE(H_, rc_.w.l, 3); SB; \
        GRS_MFMA(0, wf[3], xn_[3]); RD1(s, 1, 0); XL(2, ri_.xa.n = ld8(ws, vn, oA[si] + uo_)); *(LAS u32x4*)wp_ = e_; \
            o_[0] = PO(H_, rc_.w.l, 0); o_[1] = PO(H_, rc_.w.l, 1); o_[2] = PO(H_, rc_.w.l, 2); o_[3] = PO(H_, rc_.w.l, 3); SB; \
        GRS_MFMA(1, wf[3], xn_[3]); RD1(s, 1, 1); *(LAS u32x4*)(wp_ + 1024) = o_; \
            ha_ = rdl(hc, lA[sn_]); lb4_ = (ha_ & 0x7fu) * 0x01010101u; HI01(H_, rc_.xa.n); SB; \
        GRS_MFMA(2, wf[3], xn_[3]); RD1(s, 1, 2); XL(3, ri_.xa.l = ld16(ws, vl, oA[si] + uo_)); HI23(H_, rc_.xa.n); SB; \
        GRS_MFMA(3, wf[3], xn_[3]); RD1(s, 1, 3); \
            xn_[0] = (u32x4){PE(H_, rc_.xa.l, 0), PO(H_, rc_.xa.l, 0), PE(H_, rc_.xa.l, 1), PO(H_, rc_.xa.l, 1)}; SB; \
        GRS_MFMA(0, wf[0], xo_[0]); RD1(s, 2, 0); XL(4, ri_.xb.n = ld8(ws, vn, oB[si] + uo_)); \
            xn_[1] = (u32x4){PE(H_, rc_.xa.l, 2), PO(H_, rc_.xa.l, 2), PE(H_, rc_.xa.l, 3), PO(H_, rc_.xa.l, 3)}; SB; \
        GRS_MFMA(1, wf[0], xo_[0]); RD1(s, 2, 1); hb_ = rdl(hc, lB[sn_]); lb4_ = (hb_ & 0x7fu) * 0x01010101u; HI01(H_, rc_.xb.n); SB; \
        GRS_MFMA(2, wf[0], xo_[0]); RD1(s, 2, 2); XL(5, ri_.xb.l = ld16(ws, vl, oB[si] + uo_)); HI23(H_, rc_.xb.n); SB; \
        GRS_MFMA(3, wf[0], xo_[0]); RD1(s, 2, 3); \
            xn_[2] = (u32x4){PE(H_, rc_.xb.l, 0), PO(H_, rc_.xb.l, 0), PE(H_, rc_.xb.l, 1), PO(H_, rc_.xb.l, 1)}; SB; \
        GRS_MFMA(0, wf[1], xo_[1]); RD1(s, 3, 0); XL(6, ri_.ev = ld4(ws, eV[si], (ui) * 1024u)); \
            xn_[3] = (u32x4){PE(H_, rc_.xb.l, 2), PO(H_, rc_.xb.l, 2), PE(H_, rc_.xb.l, 3), PO(H_, rc_.xb.l, 3)}; SB; \
        GRS_MFMA(1, wf[1], xo_[1]); RD1(s, 3, 1); SB; \
        GRS_MFMA(2, wf[1], xo_[1]); RD1(s, 3, 2); SB; \
        GRS_MFMA(3, wf[1], xo_[1]); RD1(s, 3, 3); SB; \
        GRS_FIX(rc_, uc, sn_, hw_, ha_, hb_, xn_); \
        GRS_BAR(); } while (0)
    LAS unsigned char* wdst = lds + wu * UNITB + kf * 4096u + hb * 2048u + lane * 16;
    u32 hv = 0;
    ok = wait_tags(my + fo, tag, hv, my, call);
    STAMP(5);
    issue(R[0], 0, 0); issue(R[1], 0, 1);
    u32 pv = ld4(ws, fo, 128u);
    cfence();
    issue(R[2], 0, 2); issue(R[3], 0, 3);
    {
        const u32 hw_ = rdl(hv, 8u + ((me + wu) & 7u)), ha_ = rdl(hv, me & 7u), hb_ = rdl(hv, (me + 1) & 7u);
        unpack_w(R[0], 0, hw_); unpack_x(R[0].xa, ha_, X[0][0], X[0][1]); unpack_x(R[0].xb, hb_, X[0][2], X[0][3]);
        fixup(R[0], 0, 0, hw_, ha_, hb_, X[0]);
    }
    __builtin_amdgcn_s_waitcnt(0xc07f);
    lds_barrier();
    for (u32 u = 0; u < 8; ++u) {
        const u32 un = u < 7u ? u + 1u : 7u;
        u32 hn = pv;
        if (ok && !__all((pv >> 11) == tag)) ok = wait_tags(my + fo + un * 128u, tag, hn, my, call);
        STAMP(6 + 2 * u);
        GRS_STEP(0, un, 0, u, hv);
        GRS_STEP(1, un, 1, u, hv);
        pv = ld4(ws, fo, (u < 6u ? u + 2u : 7u) * 128u);
        cfence();
        GRS_STEP(2, un, 2, u, hv);
        GRS_STEP(3, un, 3, un, hn);
        STAMP(7 + 2 * u);
        hv = hn;
    }
    GRS_MM(wf[2], X[1][2]); GRS_MM(wf[3], X[1][3]);
#undef GRS_STEP
#undef GRS_RD
#undef GRS_MM
    STAMP(2);
    if (!ok || rfl(agent_load(my + OFF_TMO)) == call) return;
    float mx = 0.f;
#pragma unroll
    for (int c = 0; c < 4; ++c)
#pragma unroll
        for (int g = 0; g < 16; g += 2) mx = fmaxf(mx, fmaxf(fabsf(acc[c][g]), fabsf(acc[c][g + 1])));
    const u32 em = wave_umax((cvt_pk_bf16(mx, mx) >> 8) & 0x7fu);
    const u32 lb = em > 7u ? em - 7u : 0u;
    const rsrc_t dst = mk_rsrc(P.ws[q], WS_NEED);
    const u32 tau = ((me & 1u) * 2u + ((me >> 1) & 1u)) * 16u + (u32)n, k = me >> 2;
    u32 hd[2];
#pragma unroll
    for (int b2 = 0; b2 < 2; ++b2) {
        u32 L[8], H[8];
#pragma unroll
        for (int t = 0; t < 8; ++t) {
            const int g = 8 * b2 + t;
            const u32 d0 = cvt_pk_bf16(acc[0][g], acc[1][g]), d1 = cvt_pk_bf16(acc[2][g], acc[3][g]);
            L[t] = perm(d1, d0, 0x06040200u); H[t] = perm(d1, d0, 0x07050301u);
        }
        const u32 blk = ((k * 64 + tau) * 8 + (u32)w) * 2 + (u32)b2;
        hd[b2] = pack_store<false>(L, H, lb, dst, OFF_PP + blk * BLK, OFF_PE + blk * ENT, OFF_PR + blk * RAWB, lane);
    }
    STAMP(3);
    drain();
    if (lane < 2)
        sys_store(P.ws[q] + OFF_PFLAG + (tau * 4 + b) * 128u + (k * 4 + hb * 2 + (u32)lane) * 4u, (tag << 11) | (lane == 0 ? hd[0] : hd[1]));
    STAMP(4);
}

struct Blk { Half h[2]; };
__device__ __forceinline__ void unpack_f32(const Blk& k, u32 hdr, u32 entv, u32 ebase, rsrc_t ws, u32 raw_off, float (&f)[32], int lane) {
#pragma unroll
    for (int hb = 0; hb < 2; ++hb) {
        u32 H[4]; high_bytes(k.h[hb].n, (u32)hb, hdr, entv, ebase, H, lane);
#pragma unroll
        for (int i = 0; i < 4; ++i) {
            const int t = 4 * hb + i;
            f[4 * t]     = __uint_as_float(perm(H[i], k.h[hb].l[i], 0x04000c0cu));
            f[4 * t + 1] = __uint_as_float(perm(H[i], k.h[hb].l[i], 0x05010c0cu));
            f[4 * t + 2] = __uint_as_float(perm(H[i], k.h[hb].l[i], 0x06020c0cu));
            f[4 * t + 3] = __uint_as_float(perm(H[i], k.h[hb].l[i], 0x07030c0cu));
        }
    }
    if (hdr & (1u << 10)) {
#pragma unroll
        for (int c = 0; c < 4; ++c) { const u32x4 o = ld16(ws, (u32)lane * 16u, raw_off + 1024u * c);
#pragma unroll
            for (int i = 0; i < 4; ++i) { f[8 * c + 2 * i] = __uint_as_float(o[i] << 16); f[8 * c + 2 * i + 1] = __uint_as_float(o[i] & 0xffff0000u); } }
    }
}

__device__ __forceinline__ void reducer_wave(const Params& P, int be, int lane, u32 call, u32 tag, bool ok, LAS unsigned char* lds, char* my, int wave) {
    const int wg = blockIdx.x;
    const u32 tau = (u32)wg >> 2, wm = (u32)wg & 3u;
    const rsrc_t ws = mk_rsrc(my, WS_NEED);
    LAS u32* ctl = (LAS u32*)(lds + LDS_CTL);
    u32 hv = 0;
    if (ok) ok = wait_tags(my + OFF_PFLAG + (tau * 4 + wm) * 128u + (u32)((lane & 1) * 4 + be) * 4u, tag, hv, my, call);
    STAMP(22);
    float y[32];
    {
        const u32 b0 = ((0 * 64 + tau) * 4 + wm) * 4 + (u32)be, b1 = ((1 * 64 + tau) * 4 + wm) * 4 + (u32)be;
        const u32 vo = (u32)lane * 16u;
        const u32 von = (u32)(lane & 31) * 16u + (u32)(lane >> 5) * 8u;
        Blk k0, k1;
#pragma unroll
        for (int hb = 0; hb < 2; ++hb) {
            k0.h[hb].n = ld8(ws, von, OFF_PP + b0 * BLK + 2048u + hb * 512u); k0.h[hb].l = ld16(ws, vo, OFF_PP + b0 * BLK + hb * 1024u);
            k1.h[hb].n = ld8(ws, von, OFF_PP + b1 * BLK + 2048u + hb * 512u); k1.h[hb].l = ld16(ws, vo, OFF_PP + b1 * BLK + hb * 1024u);
        }
        const u32 ev = ld4(ws, (lane < 8 ? OFF_PE + b0 * ENT : OFF_PE + b1 * ENT) + (u32)(lane & 7) * 4u, 0);
        cfence();
        float z[32];
        unpack_f32(k0, rdl(hv, 0), ev, 0, ws, OFF_PR + b0 * RAWB, y, lane);
        unpack_f32(k1, rdl(hv, 1), ev, 8, ws, OFF_PR + b1 * RAWB, z, lane);
#pragma unroll
        for (int e = 0; e < 32; ++e) y[e] += z[e];
    }
    float mx = 0.f;
#pragma unroll
    for (int e = 0; e < 32; e += 2) mx = fmaxf(mx, fmaxf(fabsf(y[e]), fabsf(y[e + 1])));
    u32 mb = __float_as_uint(mx);
    mb = wave_umax(mb);
    STAMP(23);
    u32 last = 0;
    if (lane == 0) {
        __hip_atomic_fetch_max(ctl + 0, mb, __ATOMIC_RELAXED, __HIP_MEMORY_SCOPE_WORKGROUP);
        if (!ok) __hip_atomic_store(ctl + 4, 1u, __ATOMIC_RELAXED, __HIP_MEMORY_SCOPE_WORKGROUP);
        last = __hip_atomic_fetch_add(ctl + 1, 1u, __ATOMIC_ACQ_REL, __HIP_MEMORY_SCOPE_WORKGROUP);
    }
    last = rfl(last);
    u32 gbits = 0; bool good;
    if (last == 3u) {
        const u32 wmax = rfl(__hip_atomic_load(ctl + 0, __ATOMIC_RELAXED, __HIP_MEMORY_SCOPE_WORKGROUP));
        const u32 fail = rfl(__hip_atomic_load(ctl + 4, __ATOMIC_RELAXED, __HIP_MEMORY_SCOPE_WORKGROUP));
        good = fail == 0u;
        if (good) {
            if (lane == 0) sys_store64(my + OFF_LG + (u32)wg * 8u, ((u64)call << 32) | wmax);
            const u64 t0 = __builtin_amdgcn_s_memrealtime();
            for (;;) {
                u64 x = 0;
                if (lane < 8) x = sys_load64(my + OFF_GG + (u32)(wg & 7) * 128u + (u32)lane * 8u);
                else if (lane == 8) x = (u64)agent_load(my + OFF_TMO);
                const bool tm = __any(lane == 8 && (u32)x == call);
                const bool all = __all(lane >= 8 || (u32)(x >> 32) == call);
                if (tm) { good = false; break; }
                if (all) { gbits = wave_umax(lane < 8 ? (u32)x : 0u); break; }
                if (__builtin_amdgcn_s_memrealtime() - t0 > DEADLINE) { agent_store(my + OFF_TMO, call); good = false; break; }
                __builtin_amdgcn_s_sleep(1);
            }
        }
        if (lane == 0) {
            __hip_atomic_store(ctl + 2, gbits, __ATOMIC_RELAXED, __HIP_MEMORY_SCOPE_WORKGROUP);
            __hip_atomic_store(ctl + 3, good ? 1u : 2u, __ATOMIC_RELEASE, __HIP_MEMORY_SCOPE_WORKGROUP);
        }
    } else {
        u32 st;
        while ((st = rfl(__hip_atomic_load(ctl + 3, __ATOMIC_ACQUIRE, __HIP_MEMORY_SCOPE_WORKGROUP))) == 0u) __builtin_amdgcn_s_sleep(1);
        good = st == 1u;
        gbits = rfl(__hip_atomic_load(ctl + 2, __ATOMIC_RELAXED, __HIP_MEMORY_SCOPE_WORKGROUP));
    }
    STAMP(24);
    const float amax = __uint_as_float(gbits);
    const float scale = amax / 448.0f;
    const float inv = 1.0f / scale;
    u32x4 o0, o1;
#pragma unroll
    for (int i = 0; i < 8; ++i) {
        u32 v = (u32)__builtin_amdgcn_cvt_pk_fp8_f32(y[4 * i] * inv, y[4 * i + 1] * inv, 0, false);
        v = (u32)__builtin_amdgcn_cvt_pk_fp8_f32(y[4 * i + 2] * inv, y[4 * i + 3] * inv, (int)v, true);
        if (!good) v = 0x7f7f7f7fu;
        if (i < 4) o0[i] = v; else o1[i - 4] = v;
    }
    const u32 qi = tau >> 5, qj = (tau >> 4) & 1u, n = tau & 15u;
    const u32 row = qi * 256u + wm * 64u + (u32)(be >> 1) * 32u + (u32)(lane & 31);
    const u32 col = qj * 2048u + n * 128u + (u32)(lane >> 5) * 64u + (u32)(be & 1) * 32u;
    const rsrc_t out = mk_rsrc(P.out, 2u << 20);
    __builtin_amdgcn_raw_buffer_store_b128(o0, out, (int)(row * 4096u + col), 0, 0);
    __builtin_amdgcn_raw_buffer_store_b128(o1, out, (int)(row * 4096u + col + 16u), 0, 0);
    STAMP(25); STAMPC(31);
}

__device__ __forceinline__ void leader_wave(const Params& P, int lane, u32 call, char* my, int wave) {
    const u64 t0 = __builtin_amdgcn_s_memrealtime();
    u32 m = 0; bool good = true;
    for (u32 it = 0;; ++it) {
        bool all = true; m = 0;
#pragma unroll
        for (int i = 0; i < 4; ++i) { const u64 x = sys_load64(my + OFF_LG + (u32)(i * 64 + lane) * 8u); all = all && (u32)(x >> 32) == call; m = max(m, (u32)x); }
        if (__all(all)) break;
        if ((it & 15u) == 15u && (rfl(agent_load(my + OFF_TMO)) == call || __builtin_amdgcn_s_memrealtime() - t0 > DEADLINE)) {
            agent_store(my + OFF_TMO, call); good = false; break; }
        __builtin_amdgcn_s_sleep(1);
    }
    STAMP(26);
    if (good) {
        m = wave_umax(m);
        sys_store64(P.ws[lane >> 3] + OFF_GG + (u32)(lane & 7) * 128u + (u32)P.rank * 8u, ((u64)call << 32) | m);
    }
    if (lane == 0) agent_store(my + OFF_CALL, call);
}

__global__ void __launch_bounds__(512, 2) gemm_rs_fused(Params P) {
    __shared__ __attribute__((aligned(16))) unsigned char smem[LDS_BYTES];
    LAS unsigned char* lds = (LAS unsigned char*)smem;
    const int tid = threadIdx.x, lane = tid & 63, wave = (int)rfl((u32)tid >> 6);
    const int b = blockIdx.x, xcd = b & 7, s = b >> 3;
    char* my = P.ws[P.rank];
    if (tid < 64) ((LAS u32*)(lds + LDS_CTL))[tid] = 0u;
    const u32 call = rfl(agent_load(my + OFF_CALL)) + 1u;
    const u32 tag = call & 0x1fffffu;
    __syncthreads();
    STAMP(0); STAMPC(30);
    if (s < 8) {
        if (wave < 4) sender_wave<false>(P, s, xcd, wave, lane, tag);
        else sender_wave<true>(P, s, xcd, wave, lane, tag);
        STAMP(1);
    } else if (s < 24) {
        const int idx = s - 8, q = 4 * (xcd & 1) + (idx & 3), n = 4 * (xcd >> 1) + (idx >> 2);
        gemm_wave(P, q, n, wave, lane, call, tag, lds, my, wave);
    } else if (b == 255 && wave == 0) {
        leader_wave(P, lane, call, my, wave);
    }
    if (wave >= 4) reducer_wave(P, wave - 4, lane, call, tag, true, lds, my, wave);
}

}

extern "C" void kernel_launch(int rank, int nranks, ncclComm_t comm, hipStream_t stream,
                              void* const* d_in, const int* in_sizes, int n_in,
                              void* d_out, int out_size,
                              const int* in_dtypes, int out_dtype,
                              void* const* d_out_all,
                              void* const* d_ws_all, size_t ws_bytes) {
    (void)comm; (void)in_dtypes; (void)out_dtype; (void)d_out_all;
    if (nranks != grs::NR || n_in < 2 || ws_bytes < grs::WS_NEED || out_size != 512 * 4096 ||
        in_sizes[0] != 4096 * 512 || in_sizes[1] != 512 * 4096) {
        fprintf(stderr, "gemm_rs: unexpected problem geometry\n");
        return;
    }
    grs::Params p;
    p.x = d_in[0]; p.w = d_in[1]; p.out = d_out;
    for (int i = 0; i < grs::NR; ++i) p.ws[i] = (char*)d_ws_all[i];
    p.rank = rank; p.pad = 0;
    hipLaunchKernelGGL(grs::gemm_rs_fused, dim3(256), dim3(512), 0, stream, p);
    HIP_CHECK(hipGetLastError());
}
```

```cpp
#include <hip/hip_runtime.h>
#include <rccl/rccl.h>
#include <cstdio>
#include <cstdint>

#define HIP_CHECK(expr) do { hipError_t _e = (expr); if (_e != hipSuccess) { \
    fprintf(stderr, "HIP error %s at %s:%d\n", hipGetErrorString(_e), __FILE__, __LINE__); \
    return; } } while (0)

namespace grs {

typedef unsigned u32;
typedef unsigned long long u64;
typedef unsigned u32x4 __attribute__((ext_vector_type(4)));
typedef unsigned u32x2 __attribute__((ext_vector_type(2)));
typedef unsigned short u16x2 __attribute__((ext_vector_type(2)));
typedef float f32x16 __attribute__((ext_vector_type(16)));
typedef short bf16x8 __attribute__((ext_vector_type(8)));
typedef __amdgpu_buffer_rsrc_t rsrc_t;
#define LAS __attribute__((address_space(3)))
#define GAS __attribute__((address_space(1)))

constexpr int NR = 8;
constexpr int AUX_SYS = 17;
constexpr u32 RSRC_W3 = 0x00020000u;
constexpr u64 DEADLINE = 20000000ull;
constexpr u32 TAGSH = 14, TAGMASK = 0x3ffffu, RAWBIT = 1u << 13;

constexpr u32 OFF_CALL  = 0;
constexpr u32 OFF_TMO   = 128;
constexpr u32 OFF_XFLAG = 4096;
constexpr u32 OFF_WFLAG = 16384;
constexpr u32 OFF_PFLAG = 32768;
constexpr u32 OFF_SCTR  = 2048;
constexpr u32 OFF_LG    = 65536;
constexpr u32 OFF_GG    = 69632;
constexpr u32 OFF_DBG   = 48u << 20;
constexpr u32 BLK = 3072, ENT = 32, RAWB = 4096;
constexpr u32 OFF_XP = 1u << 20, OFF_WP = 7u << 20, OFF_PP = 13u << 20;
constexpr u32 OFF_XE = 19u << 20, OFF_WE = OFF_XE + 65536, OFF_PE = OFF_WE + 65536;
constexpr u32 OFF_XR = 20u << 20, OFF_WR = 28u << 20, OFF_PR = 36u << 20;
constexpr u32 WS_NEED = 44u << 20;

constexpr u32 SLOT = 16384, UNITB = 8192;
constexpr u32 LDS_CTL = 4 * SLOT;
constexpr u32 LDS_BYTES = 96 * 1024;

#define GRS_CFG 1
constexpr bool NARROW_A = GRS_CFG == 1, NARROW_B = GRS_CFG == 2;
__device__ __forceinline__ bool is_narrow(unsigned n) { return n < 8u ? NARROW_A : NARROW_B; }

struct Params {
    const void* x; const void* w; void* out;
    char* ws[NR];
    int rank; u32 pad;
};

__device__ __forceinline__ u32 perm(u32 s0, u32 s1, u32 sel) { return __builtin_amdgcn_perm(s0, s1, sel); }
__device__ __forceinline__ u32 rotr4(u32 v) { return __builtin_amdgcn_alignbit(v, v, 4); }
__device__ __forceinline__ u32 rotl4(u32 v) { return __builtin_amdgcn_alignbit(v, v, 28); }
__device__ __forceinline__ u32 rfl(u32 v) { return (u32)__builtin_amdgcn_readfirstlane((int)v); }
__device__ __forceinline__ u32 rdl(u32 v, u32 l) { return (u32)__builtin_amdgcn_readlane((int)v, (int)l); }
__device__ __forceinline__ u32 pkmax(u32 a, u32 b) {
    return __builtin_bit_cast(u32, __builtin_elementwise_max(__builtin_bit_cast(u16x2, a), __builtin_bit_cast(u16x2, b)));
}
__device__ __forceinline__ u32 wave_umax(u32 v) {
    v = max(v, (u32)__builtin_amdgcn_update_dpp(0, (int)v, 0x111, 0xf, 0xf, true));
    v = max(v, (u32)__builtin_amdgcn_update_dpp(0, (int)v, 0x112, 0xf, 0xf, true));
    v = max(v, (u32)__builtin_amdgcn_update_dpp(0, (int)v, 0x114, 0xf, 0xf, true));
    v = max(v, (u32)__builtin_amdgcn_update_dpp(0, (int)v, 0x118, 0xf, 0xf, true));
    v = max(v, (u32)__builtin_amdgcn_update_dpp(0, (int)v, 0x142, 0xa, 0xf, false));
    v = max(v, (u32)__builtin_amdgcn_update_dpp(0, (int)v, 0x143, 0xc, 0xf, false));
    return rdl(v, 63);
}
__device__ __forceinline__ void drain() { asm volatile("s_waitcnt vmcnt(0)" ::: "memory"); }
__device__ __forceinline__ void cfence() { asm volatile("" ::: "memory"); }
__device__ __forceinline__ rsrc_t mk_rsrc(const void* p, u32 bytes) { return __builtin_amdgcn_make_buffer_rsrc((void*)p, 0, (int)bytes, RSRC_W3); }
#define EXP_LDAUX AUX_SYS
__device__ __forceinline__ u32x4 ld16(rsrc_t r, u32 voff, u32 soff) { return __builtin_amdgcn_raw_buffer_load_b128(r, (int)voff, (int)soff, EXP_LDAUX); }
__device__ __forceinline__ u32 ld4(rsrc_t r, u32 voff, u32 soff) { return __builtin_amdgcn_raw_buffer_load_b32(r, (int)voff, (int)soff, AUX_SYS); }
__device__ __forceinline__ void st16(rsrc_t r, u32 voff, u32 soff, u32x4 v) { __builtin_amdgcn_raw_buffer_store_b128(v, r, (int)(voff + soff), 0, AUX_SYS); }
__device__ __forceinline__ void st4(rsrc_t r, u32 voff, u32 soff, u32 v) { __builtin_amdgcn_raw_buffer_store_b32(v, r, (int)voff, (int)soff, AUX_SYS); }
__device__ __forceinline__ u32 sys_load(const char* p) { return __hip_atomic_load((const GAS u32*)p, __ATOMIC_RELAXED, __HIP_MEMORY_SCOPE_SYSTEM); }
__device__ __forceinline__ u64 sys_load64(const char* p) { return __hip_atomic_load((const GAS u64*)p, __ATOMIC_RELAXED, __HIP_MEMORY_SCOPE_SYSTEM); }
__device__ __forceinline__ void sys_store(char* p, u32 v) { __hip_atomic_store((GAS u32*)p, v, __ATOMIC_RELAXED, __HIP_MEMORY_SCOPE_SYSTEM); }
__device__ __forceinline__ void sys_store64(char* p, u64 v) { __hip_atomic_store((GAS u64*)p, v, __ATOMIC_RELAXED, __HIP_MEMORY_SCOPE_SYSTEM); }
__device__ __forceinline__ u32 agent_load(const char* p) { return __hip_atomic_load((const GAS u32*)p, __ATOMIC_RELAXED, __HIP_MEMORY_SCOPE_AGENT); }
__device__ __forceinline__ void agent_store(char* p, u32 v) { __hip_atomic_store((GAS u32*)p, v, __ATOMIC_RELAXED, __HIP_MEMORY_SCOPE_AGENT); }

#define STAMPC(idx) do { } while (0)
#define STAMP2(idx, val) do { } while (0)
#define STAMP(idx) do { } while (0)

__device__ __forceinline__ u32 poll_sys(const char* p) { u32 x; asm volatile("global_load_dword %0, %1, off sc0 sc1\n\ts_waitcnt vmcnt(0)" : "=&v"(x) : "v"(p) : "memory"); return x; }
__device__ __forceinline__ u32 poll_agent(const char* p) { u32 x; asm volatile("global_load_dword %0, %1, off sc1\n\ts_waitcnt vmcnt(0)" : "=&v"(x) : "v"(p) : "memory"); return x; }
__device__ __forceinline__ void ld16x2_now(const char* p0, const char* p1, u32x4& a, u32x4& b) {
    asm volatile("global_load_dwordx4 %0, %2, off sc0 sc1\n\tglobal_load_dwordx4 %1, %3, off sc0 sc1\n\ts_waitcnt vmcnt(0)" : "=&v"(a), "=&v"(b) : "v"(p0), "v"(p1) : "memory");
}
__device__ __forceinline__ void give_up(char* my, u32 call) {
    asm volatile("global_store_dword %0, %1, off sc1" :: "v"(my + OFF_TMO), "v"(call) : "memory");
}
__device__ __forceinline__ bool wait_tags(const char* addr, u32 tag, u32& v, char* my, u32 call) {
    const u64 t0 = __builtin_amdgcn_s_memrealtime();
    for (u32 it = 0;; ++it) {
        const u32 x = poll_sys(addr);
        if (__all((x >> TAGSH) == tag)) { v = x; return true; }
        if ((it & 15u) == 15u) {
            if (rfl(poll_agent(my + OFF_TMO)) == call || __builtin_amdgcn_s_memrealtime() - t0 > DEADLINE) {
                give_up(my, call); v = 0; return false; }
        }
        __builtin_amdgcn_s_sleep(2);
    }
}

__device__ __forceinline__ u32 lane_emax(const u32 (&H)[8]) {
    u32 m1 = 0, m2 = 0;
#pragma unroll
    for (int t = 0; t < 8; ++t) { const u32 g = H[t] | 0x80808080u; m1 = pkmax(m1, g); m2 = pkmax(m2, g << 8); }
    const u32 m = pkmax(m1, m2);
    return max((m >> 24) & 0x7fu, (m >> 8) & 0x7fu);
}
template <bool IS_W> __device__ __forceinline__ void chunks_from(const u32 (&L)[8], const u32 (&H)[8], u32x4 (&o)[4]) {
#pragma unroll
    for (int t = 0; t < 8; ++t) {
        const u32 e = perm(H[t], L[t], 0x05010400u), d = perm(H[t], L[t], 0x07030602u);
        if (IS_W) { o[2 * (t >> 2)][t & 3] = e; o[2 * (t >> 2) + 1][t & 3] = d; }
        else      { o[t >> 1][2 * (t & 1)] = e; o[t >> 1][2 * (t & 1) + 1] = d; }
    }
}
__device__ __forceinline__ u32 pack_store(const u32 (&L)[8], const u32 (&H)[8], u32 lb, rsrc_t dst, u32 blk_off, u32 ent_off, u32 raw_off, int lane, bool is_w) {
    const u32 lb4 = lb * 0x01010101u;
    u32 T[8], acc = 0xffffffffu;
#pragma unroll
    for (int t = 0; t < 8; ++t) { T[t] = (H[t] | 0x80808080u) - lb4; acc &= T[t]; }
    u32 n[4];
#pragma unroll
    for (int j = 0; j < 4; ++j) {
        const u32 a = (H[2 * j] & 0x80808080u) | (T[2 * j] & 0x07070707u);
        const u32 b = (H[2 * j + 1] & 0x80808080u) | (T[2 * j + 1] & 0x07070707u);
        n[j] = a | rotl4(b);
    }
    const auto s0 = __builtin_amdgcn_permlane32_swap(n[0], n[2], false, false);
    const auto s1 = __builtin_amdgcn_permlane32_swap(n[1], n[3], false, false);
    const u32 vo = (u32)lane * 16u;
    st16(dst, vo, blk_off, (u32x4){L[0], L[1], L[2], L[3]});
    st16(dst, vo, blk_off + 1024u, (u32x4){L[4], L[5], L[6], L[7]});
    st16(dst, vo, blk_off + 2048u, (u32x4){s0[0], s1[0], s0[1], s1[1]});
    u32 hdr = lb;
    const bool bad = (acc & 0x80808080u) != 0x80808080u;
    u64 bm = __ballot(bad);
    if (bm != 0ull) {
        u32 bits = 0;
#pragma unroll
        for (int t = 0; t < 8; ++t) { const u32 x = (~T[t] & 0x80808080u) >> 7;
            bits |= ((x & 1u) | ((x >> 7) & 2u) | ((x >> 14) & 4u) | ((x >> 21) & 8u)) << (4 * t); }
        const u32 nb0 = __popc(bits & 0xffffu), nb1 = __popc(bits >> 16);
        u32 n0 = 0, n1 = 0, p0 = 0, p1 = 0;
        while (bm) { const u32 l = (u32)__builtin_ctzll(bm); bm &= bm - 1; if ((u32)lane == l) { p0 = n0; p1 = n1; } n0 += rdl(nb0, l); n1 += rdl(nb1, l); }
        if (n0 + n1 <= 7u) {
            hdr |= (n0 << 7) | (n1 << 10);
            p1 += n0;
            while (bits) {
                const u32 idx = (u32)__builtin_ctz(bits); bits &= bits - 1;
                const u32 t = idx >> 2, by = idx & 3u;
                u32 hv = H[0];
#pragma unroll
                for (int tt = 1; tt < 8; ++tt) hv = (t == (u32)tt) ? H[tt] : hv;
                const u32 slot = t < 4u ? p0++ : p1++;
                st4(dst, slot * 4u, ent_off, ((u32)lane << 16) | (t << 10) | (by << 8) | ((hv >> (8 * by)) & 0xffu));
            }
        } else {
            hdr |= RAWBIT;
            u32x4 o[4]; if (is_w) chunks_from<true>(L, H, o); else chunks_from<false>(L, H, o);
#pragma unroll
            for (int c = 0; c < 4; ++c) st16(dst, vo, raw_off + 1024u * c, o[c]);
        }
    }
    return hdr;
}
struct Half { u32x4 l; u32x2 n; };
__device__ __forceinline__ void high_bytes(u32x2 n, u32 hb, u32 hdr, u32 entv, u32 ebase, u32 (&H)[4], int lane) {
    const u32 lb4 = (hdr & 0x7fu) * 0x01010101u;
    H[0] = (n[0] & 0x87878787u) + lb4; H[1] = (rotr4(n[0]) & 0x87878787u) + lb4;
    H[2] = (n[1] & 0x87878787u) + lb4; H[3] = (rotr4(n[1]) & 0x87878787u) + lb4;
    const u32 n0 = (hdr >> 7) & 7u, first = hb ? n0 : 0u, cnt = hb ? (hdr >> 10) & 7u : n0;
    for (u32 i = 0; i < cnt; ++i) {
        const u32 en = rdl(entv, ebase + first + i);
        const u32 sh = ((en >> 8) & 3u) * 8u, t = (en >> 10) & 7u;
        const u32 m = ((u32)lane == (en >> 16)) ? (0xffu << sh) : 0u, v = (en & 0xffu) << sh;
#pragma unroll
        for (int j = 0; j < 4; ++j) { const u32 mm = ((t & 3u) == (u32)j) ? m : 0u; H[j] = (H[j] & ~mm) | (v & mm); }
    }
}

#define GRS_SW 8
#define GRS_DEFER 3
constexpr u32 DEFER = GRS_DEFER;
constexpr u32 NBLK = 512, SEND_WPW = GRS_SW, SEND_WAVES = 8 * SEND_WPW;
__device__ __forceinline__ void sender_wave(const Params& P, int d, u32 wid, int lane, u32 call, u32 tag, char* my) {
    const u32 di = (u32)d & 1u, dj = ((u32)d >> 1) & 1u, dk = (u32)d >> 2, me = (u32)P.rank;
    const u32 r = (u32)lane & 31u, h = (u32)lane >> 5;
    const rsrc_t dst = mk_rsrc(P.ws[d], WS_NEED);
    const rsrc_t srcx = mk_rsrc(P.x, 4u << 20), srcw = mk_rsrc(P.w, 4u << 20);
    const u32 vx = (di * 256u + r) * 1024u + 32u * h + dk * 512u;
    const u32 cg = 16u * ((r >> 2) & 1u) + (r & 3u) + 4u * (r >> 3);
    const u32 vw0 = (16u * h + dk * 256u) * 8192u + dj * 4096u;
    const u32 vw = vw0 + cg * 8u, vwa = vw0 + cg * 4u;
    GAS u32* ctr = (GAS u32*)(my + OFF_SCTR + (u32)d * 128u);
    const u32 base = (call - 1u) * NBLK - SEND_WAVES;
    auto draw = [&]() { u32 t = 0; const u32 one = 1u;
        if (lane == 0) asm volatile("global_atomic_add %0, %1, %2, off sc0" : "=v"(t) : "v"(ctr), "v"(one) : "memory");
        return t; };
    u32 in[16];
    auto unit_of = [&](u32 idx, u32& rr) {
        constexpr u32 N0 = (8u - DEFER) * 64u, N1 = N0 + DEFER * 48u;
        if (idx < N0) { rr = idx & 63u; return idx >> 6; }
        if (idx < N1) { const u32 j = idx - N0, uu = j / 48u; rr = j - uu * 48u; return 8u - DEFER + uu; }
        const u32 j = idx - N1; rr = 48u + (j & 15u); return 8u - DEFER + (j >> 4);
    };
    auto load_raw = [&](u32 u, u32 rr) {
        if (rr < 32u) {
            const u32 so = ((rr >> 2) * 512u + (rr & 3u) * 64u) * 1024u + u * 64u;
#pragma unroll
            for (int c = 0; c < 4; ++c) {
                const u32x4 v = __builtin_amdgcn_raw_buffer_load_b128(srcx, (int)(vx + (u32)(c >> 1) * 32768u + (u32)(c & 1) * 16u), (int)so, 0);
                in[4 * c] = v[0]; in[4 * c + 1] = v[1]; in[4 * c + 2] = v[2]; in[4 * c + 3] = v[3]; }
        } else {
            const u32 so = (u * 32u + (rr & 1u) * 8u) * 8192u + ((rr - 32u) >> 1) * 256u;
            if (is_narrow((rr - 32u) >> 1)) {
#pragma unroll
                for (int j = 0; j < 8; ++j) {
                    in[2 * (j >> 1) + (j & 1)]     = __builtin_amdgcn_raw_buffer_load_b32(srcw, (int)(vwa + (u32)j * 8192u), (int)so, 0);
                    in[8 + 2 * (j >> 1) + (j & 1)] = __builtin_amdgcn_raw_buffer_load_b32(srcw, (int)(vwa + (u32)j * 8192u + 128u), (int)so, 0); }
            } else {
#pragma unroll
                for (int j = 0; j < 8; ++j) {
                    const u32x2 v = __builtin_amdgcn_raw_buffer_load_b64(srcw, (int)(vw + (u32)j * 8192u), (int)so, 0);
                    in[2 * (j >> 1) + (j & 1)] = v[0]; in[8 + 2 * (j >> 1) + (j & 1)] = v[1]; }
            }
        }
    };
    u32 rr, u = unit_of(wid, rr);
    load_raw(u, rr);
    u32 prev_flag = 0, prev_hdr = 0; bool have_prev = false;
#define SSTAMP() do { } while (0)
    while (true) {
        const bool is_w = rr >= 32u;
        const u32 selL = is_w ? 0x06020400u : 0x06040200u, selH = is_w ? 0x07030501u : 0x07050301u;
        u32 L[8], H[8];
#pragma unroll
        for (int t = 0; t < 8; ++t) { L[t] = perm(in[2 * t + 1], in[2 * t], selL); H[t] = perm(in[2 * t + 1], in[2 * t], selH); }
        const u32 em = wave_umax(lane_emax(H));
        const u32 lb = em > 7u ? em - 7u : 0u;
        drain();
        if (have_prev && lane == 0) sys_store(P.ws[d] + prev_flag, (tag << TAGSH) | prev_hdr);
        u32 tn = draw();
        u32 blk, offP, offE, offR;
        if (!is_w) { blk = ((me * 8 + u) * 8 + (rr >> 2)) * 4 + (rr & 3u); offP = OFF_XP; offE = OFF_XE; offR = OFF_XR;
                     prev_flag = OFF_XFLAG + ((rr >> 2) * 8 + u) * 128u + ((rr & 3u) * 8 + me) * 4u; }
        else       { const u32 n = (rr - 32u) >> 1, kf = rr & 1u; blk = ((me * 8 + u) * 16 + n) * 2 + kf; offP = OFF_WP; offE = OFF_WE; offR = OFF_WR;
                     prev_flag = OFF_WFLAG + (n * 8 + u) * 128u + (kf * 8 + me) * 4u; }
        prev_hdr = pack_store(L, H, lb, dst, offP + blk * BLK, offE + blk * ENT, offR + blk * RAWB, lane, is_w);
        have_prev = true;
        asm volatile("s_waitcnt vmcnt(3)" : "+v"(tn) :: "memory");
        const u32 nxt = rfl(tn) - base;
        if (nxt >= NBLK) break;
        u = unit_of(nxt, rr);
        load_raw(u, rr);
    }
    drain();
    if (lane == 0) sys_store(P.ws[d] + prev_flag, (tag << TAGSH) | prev_hdr);
    SSTAMP();
}

__device__ __forceinline__ void lds_barrier() { asm volatile("s_waitcnt lgkmcnt(0)\n\ts_barrier" ::: "memory"); }
__device__ __forceinline__ u32 cvt_pk_bf16(float lo, float hi) { u32 r; asm("v_cvt_pk_bf16_f32 %0, %1, %2" : "=v"(r) : "v"(lo), "v"(hi)); return r; }
__device__ __forceinline__ u32x2 ld8(rsrc_t r, u32 voff, u32 soff) { return __builtin_amdgcn_raw_buffer_load_b64(r, (int)voff, (int)soff, EXP_LDAUX); }
struct StepRegs { Half xa, xb, w; u32 ev; };

__device__ __forceinline__ void ald16(u32x4& d, u32 voff, u32x4 rs, u32 soff) { asm volatile("buffer_load_dwordx4 %0, %1, %2, %3 offen sc0 sc1" : "+v"(d) : "v"(voff), "s"(rs), "s"(soff)); }
__device__ __forceinline__ void ald8(u32x2& d, u32 voff, u32x4 rs, u32 soff) { asm volatile("buffer_load_dwordx2 %0, %1, %2, %3 offen sc0 sc1" : "+v"(d) : "v"(voff), "s"(rs), "s"(soff)); }
__device__ __forceinline__ void ald4(u32& d, u32 voff, u32x4 rs, u32 soff) { asm volatile("buffer_load_dword %0, %1, %2, %3 offen sc0 sc1" : "+v"(d) : "v"(voff), "s"(rs), "s"(soff)); }
#define LOOK_SET(R, val) asm volatile("v_mov_b32 " R ", %0" :: "s"(val) : R)
#define LOOK_LD(R, voff, rs, soff) asm volatile("buffer_load_dword " R ", %0, %1, %2 offen sc0 sc1" :: "v"(voff), "s"(rs), "s"(soff) : R)
#define LOOK_GET(R, N, dstv) asm volatile("s_waitcnt vmcnt(" #N ")\n\tv_mov_b32 %0, " R : "=v"(dstv))
#define VM_WAIT_SET(N, r) asm volatile("s_waitcnt vmcnt(" #N ")" : "+v"(r.w.l), "+v"(r.w.n), "+v"(r.xa.l), "+v"(r.xa.n), "+v"(r.xb.l), "+v"(r.xb.n), "+v"(r.ev))
constexpr u32 LDS_CNT = LDS_CTL + 64;
__device__ __forceinline__ void sig_step(u32 cnt_addr, u32 add) {
    asm volatile("s_mov_b64 exec, 1\n\tds_add_u32 %0, %1\n\ts_mov_b64 exec, -1" :: "v"(cnt_addr), "v"(add) : "memory");
}
__device__ __forceinline__ void wait_step(LAS unsigned char* lds, u32 J, u32 seen) {
    if (rfl(seen) == 8u) return;
    while (rfl(__hip_atomic_load((LAS u32*)(lds + LDS_CNT) + J, __ATOMIC_RELAXED, __HIP_MEMORY_SCOPE_WORKGROUP)) != 8u) __builtin_amdgcn_s_sleep(1);
    cfence();
}

template <int NC>
__device__ __forceinline__ void gemm_wave(const Params& P, int q, int n, u32 ht, int w, int lane, u32 call, u32 tag, LAS unsigned char* lds, char* my, int wave) {
    constexpr u32 SLOTB = NC * 4096u, SUBB = NC * 1024u;
    const rsrc_t ws = mk_rsrc(my, WS_NEED);
    const u32 me = (u32)P.rank;
    const u32 vo = (u32)lane * 16u, von = (u32)(lane & 31) * 16u + (u32)(lane >> 5) * 8u;
    const u32 b = (u32)w >> 1, hb = (u32)w & 1u;
    const u32 wu = (u32)w >> 2, kf = ((u32)w >> 1) & 1u;
    const u32 wh = NC == 4 ? hb : ht;
    bool ok = true;
    const u32 fo = (lane & 8) == 0 ? OFF_XFLAG + (u32)(q * 8) * 128u + (b * 8 + (u32)(lane & 7)) * 4u
                                   : OFF_WFLAG + (u32)(n * 8) * 128u + (kf * 8 + (u32)(lane & 7)) * 4u;
    const u32 elo = (u32)(lane & 7) * 4u;
    StepRegs R[4];
#pragma unroll
    for (int s = 0; s < 4; ++s) { R[s].w.l = (u32x4){0u, 0u, 0u, 0u}; R[s].xa.l = R[s].w.l; R[s].xb.l = R[s].w.l; R[s].w.n = (u32x2){0u, 0u}; R[s].xa.n = R[s].w.n; R[s].xb.n = R[s].w.n; R[s].ev = 0u; }
    constexpr u32 USTR = 32u * BLK;
    const u32 vl = vo + hb * 1024u, vn = von + 2048u + hb * 512u;
    const u32 vwl = NC == 4 ? vl : vo + ht * 1024u + hb * 8u, vwn = NC == 4 ? vn : von + 2048u + ht * 512u + hb * 4u;
    u32 oA[4], oB[4], oW[4], eV[4], lA[4], lB[4], lW[4];
#pragma unroll
    for (int s = 0; s < 4; ++s) {
        const u32 pa = (me + 2 * s) & 7u, pb = (me + 2 * s + 1) & 7u, pw = wu ? pb : pa;
        const u32 ba = ((pa * 8) * 8 + q) * 4 + b, bb = ((pb * 8) * 8 + q) * 4 + b, bw = ((pw * 8) * 16 + n) * 2 + kf;
        oA[s] = OFF_XP + ba * BLK; oB[s] = OFF_XP + bb * BLK; oW[s] = OFF_WP + bw * BLK;
        eV[s] = (lane < 8 ? OFF_XE + ba * ENT : (lane < 16 ? OFF_XE + bb * ENT : OFF_WE + bw * ENT)) + elo;
        lA[s] = pa; lB[s] = pb; lW[s] = 8u + pw;
    }
    const u32 rs0 = rfl((u32)(size_t)my), rs1 = rfl((u32)((size_t)my >> 32)) & 0xffffu;
    const u32x4 rsw = {rs0, rs1, WS_NEED, RSRC_W3};
    auto ld_wn = [&](StepRegs& r, u32x4 rs, u32 so) {
        if (NC == 4) ald8(r.w.n, vwn, rs, so); else { u32 t = r.w.n[0]; ald4(t, vwn, rs, so); r.w.n[0] = t; } };
    auto ld_wl = [&](StepRegs& r, u32x4 rs, u32 so) {
        if (NC == 4) ald16(r.w.l, vwl, rs, so); else { u32x2 t = {r.w.l[0], r.w.l[1]}; ald8(t, vwl, rs, so); r.w.l[0] = t[0]; r.w.l[1] = t[1]; } };
    auto issue = [&](StepRegs& r, u32 u, int s) {
        const u32 uo = u * USTR;
        ld_wn(r, rsw, oW[s] + uo); ld_wl(r, rsw, oW[s] + uo);
        ald8(r.xa.n, vn, rsw, oA[s] + uo); ald16(r.xa.l, vl, rsw, oA[s] + uo);
        ald8(r.xb.n, vn, rsw, oB[s] + uo); ald16(r.xb.l, vl, rsw, oB[s] + uo);
        ald4(r.ev, eV[s], rsw, u * 1024u);
    };
    LAS unsigned char* wdst = lds + (wu * 2u + kf) * SUBB + (NC == 4 ? hb * 2048u + (u32)lane * 16u : (u32)lane * 16u + hb * 8u);
    const u32 cntsh = hb ? 10u : 7u, fixmask = (7u << cntsh) | RAWBIT;
    const u32 wcntsh = wh ? 10u : 7u, wfixmask = (7u << wcntsh) | RAWBIT;
    auto fix_x = [&](u32 hdr, u32 ev, u32 ebase, u32 u, u32 p, u32x4& a, u32x4& c) {
        if ((hdr & fixmask) == 0u) return;
        if (hdr & RAWBIT) {
            const u32 bx = ((p * 8 + u) * 8 + q) * 4 + b;
            const char* rp = my + OFF_XR + bx * RAWB + hb * 2048u + vo;
            ld16x2_now(rp, rp + 1024, a, c);
        }
        const u32 cnt = (hdr >> cntsh) & 7u, first = ebase + (hb ? (hdr >> 7) & 7u : 0u);
        for (u32 i = 0; i < cnt; ++i) {
            const u32 en = rdl(ev, first + i), t = (en >> 10) & 7u, by = (en >> 8) & 3u;
            const u32 sh = (by & 1u) * 16u + 8u, D = 2u * (t & 3u) + (by >> 1);
            const u32 m = ((u32)lane == (en >> 16)) ? (0xffu << sh) : 0u, v = (en & 0xffu) << sh;
#pragma unroll
            for (int j = 0; j < 4; ++j) {
                const u32 ma = (D == (u32)j) ? m : 0u, mc = (D == (u32)(4 + j)) ? m : 0u;
                a[j] = (a[j] & ~ma) | (v & ma); c[j] = (c[j] & ~mc) | (v & mc); }
        }
    };
    auto fix_w = [&](const StepRegs& r, u32 u, int s, u32 hw) {
        if ((hw & wfixmask) == 0u) return;
        LAS unsigned char* p = lds + s * SLOTB + (wu * 2u + kf) * SUBB + (NC == 4 ? hb * 2048u : 0u);
        if (hw & RAWBIT) {
            const u32 bw = ((((me + 2 * s + wu) & 7u) * 8 + u) * 16 + n) * 2 + kf;
            const char* rp = my + OFF_WR + bw * RAWB + wh * 2048u + vo;
            u32x4 v0, v1; ld16x2_now(rp, rp + 1024, v0, v1);
            if (NC == 4) { *(LAS u32x4*)(p + lane * 16) = v0; *(LAS u32x4*)(p + 1024 + lane * 16) = v1; }
            else { *(LAS u32x2*)(p + lane * 16 + hb * 8u) = hb ? (u32x2){v0[2], v0[3]} : (u32x2){v0[0], v0[1]};
                   *(LAS u32x2*)(p + 1024 + lane * 16 + hb * 8u) = hb ? (u32x2){v1[2], v1[3]} : (u32x2){v1[0], v1[1]}; }
        }
        const u32 cnt = (hw >> wcntsh) & 7u, first = 16u + (wh ? (hw >> 7) & 7u : 0u);
        for (u32 i = 0; i < cnt; ++i) {
            const u32 en = rdl(r.ev, first + i), t = (en >> 10) & 7u, by = (en >> 8) & 3u;
            if (NC == 2 && ((t >> 1) & 1u) != hb) continue;
            if (lane == 0) p[(en >> 16) * 16u + (by >> 1) * 1024u + (t & 3u) * 4u + (by & 1u) * 2u + 1u] = (unsigned char)(en & 0xffu);
        }
        asm volatile("s_waitcnt lgkmcnt(0)" ::: "memory");
    };
    f32x16 acc[NC];
#pragma unroll
    for (int c = 0; c < NC; ++c)
#pragma unroll
        for (int g = 0; g < 16; ++g) acc[c][g] = 0.f;
    u32x4 X[2][4]; bf16x8 wf[2][NC];
#pragma unroll
    for (int c = 0; c < NC; ++c) wf[1][c] = (bf16x8){0, 0, 0, 0, 0, 0, 0, 0};
    X[1][3] = (u32x4){0u, 0u, 0u, 0u};
    LAS unsigned char* wb = lds + lane * 16;
    const u32 cnt0 = (u32)(size_t)(lds + LDS_CNT);
#define MF(c, F, XF) acc[c] = __builtin_amdgcn_mfma_f32_32x32x16_bf16(F[c], __builtin_bit_cast(bf16x8, XF), acc[c], 0, 0, 0)
#define SB __builtin_amdgcn_sched_barrier(0)
#define RD1(s, sub, c) wf[(sub) & 1][c] = *(const LAS bf16x8*)(wb + (s) * SLOTB + (sub) * SUBB + (c) * 1024)
#define LB4(h) (((h) & 0x7fu) * 0x01010101u)
#define HI01(H, n) do { H[0] = (n[0] & 0x87878787u) + lb4_; H[1] = (rotr4(n[0]) & 0x87878787u) + lb4_; } while (0)
#define HI23(H, n) do { H[2] = (n[1] & 0x87878787u) + lb4_; H[3] = (rotr4(n[1]) & 0x87878787u) + lb4_; } while (0)
#define PE(H, l, i) perm(H[i], l[i], 0x05010400u)
#define PO(H, l, i) perm(H[i], l[i], 0x07030602u)
#define XPAIR(H, l, i) (u32x4){PE(H, l, i), PO(H, l, i), PE(H, l, (i) + 1), PO(H, l, (i) + 1)}
#define GRS_STEP4(s, ui, uc, hc, hm, sg, J) do { \
        constexpr int sn_ = ((s) + 1) & 3; \
        StepRegs& ri_ = R[s]; StepRegs& rc_ = R[sn_]; \
        u32x4 (&xo_)[4] = X[(s) & 1]; u32x4 (&xn_)[4] = X[sn_ & 1]; \
        const u32 uo_ = (ui) * USTR; \
        u32 H_[4], lb4_, hw_, ha_, hb_, seen_; u32x4 e_, o_; \
        LAS unsigned char* wp_ = wdst + sn_ * SLOTB; \
        SB; \
        MF(0, wf[1], xn_[3]); RD1(s, 0, 0); ld_wn(ri_, rsl, oW[s] + uo_); hw_ = rdl(hc, lW[sn_]) & (hm); lb4_ = LB4(hw_); SB; \
        VM_WAIT_SET(15, rc_); \
        MF(1, wf[1], xn_[3]); RD1(s, 0, 1); HI01(H_, rc_.w.n); SB; \
        MF(2, wf[1], xn_[3]); RD1(s, 0, 2); ld_wl(ri_, rsl, oW[s] + uo_); HI23(H_, rc_.w.n); SB; \
        MF(3, wf[1], xn_[3]); RD1(s, 0, 3); \
            e_[0] = PE(H_, rc_.w.l, 0); e_[1] = PE(H_, rc_.w.l, 1); e_[2] = PE(H_, rc_.w.l, 2); e_[3] = PE(H_, rc_.w.l, 3); SB; \
        MF(0, wf[0], xo_[0]); RD1(s, 1, 0); ald8(ri_.xa.n, vn, rsl, oA[s] + uo_); \
            *(LAS u32x4*)wp_ = e_; o_[0] = PO(H_, rc_.w.l, 0); o_[1] = PO(H_, rc_.w.l, 1); o_[2] = PO(H_, rc_.w.l, 2); o_[3] = PO(H_, rc_.w.l, 3); SB; \
        MF(1, wf[0], xo_[0]); RD1(s, 1, 1); \
            *(LAS u32x4*)(wp_ + 1024) = o_; ha_ = rdl(hc, lA[sn_]) & (hm); lb4_ = LB4(ha_); HI01(H_, rc_.xa.n); SB; \
        fix_w(rc_, uc, sn_, hw_); SB; \
        MF(2, wf[0], xo_[0]); RD1(s, 1, 2); ald16(ri_.xa.l, vl, rsl, oA[s] + uo_); HI23(H_, rc_.xa.n); SB; \
        MF(3, wf[0], xo_[0]); RD1(s, 1, 3); xn_[0] = XPAIR(H_, rc_.xa.l, 0); SB; \
        asm volatile("s_waitcnt lgkmcnt(2)" ::: "memory"); sig_step(cntv + (((s) + 1) * 4), sg); \
        MF(0, wf[1], xo_[1]); RD1(s, 2, 0); ald8(ri_.xb.n, vn, rsl, oB[s] + uo_); xn_[1] = XPAIR(H_, rc_.xa.l, 2); SB; \
        MF(1, wf[1], xo_[1]); RD1(s, 2, 1); hb_ = rdl(hc, lB[sn_]) & (hm); lb4_ = LB4(hb_); HI01(H_, rc_.xb.n); SB; \
        MF(2, wf[1], xo_[1]); RD1(s, 2, 2); ald16(ri_.xb.l, vl, rsl, oB[s] + uo_); HI23(H_, rc_.xb.n); SB; \
        MF(3, wf[1], xo_[1]); RD1(s, 2, 3); xn_[2] = XPAIR(H_, rc_.xb.l, 0); SB; \
        MF(0, wf[0], xo_[2]); RD1(s, 3, 0); ald4(ri_.ev, eV[s], rsl, (ui) * 1024u); xn_[3] = XPAIR(H_, rc_.xb.l, 2); SB; \
        MF(1, wf[0], xo_[2]); RD1(s, 3, 1); seen_ = *(volatile LAS u32*)(lds + LDS_CNT + ((J) + 1) * 4u); SB; \
        MF(2, wf[0], xo_[2]); RD1(s, 3, 2); SB; \
        MF(3, wf[0], xo_[2]); RD1(s, 3, 3); SB; \
        if (((ha_ | hb_) & fixmask) != 0u) { fix_x(ha_, rc_.ev, 0, uc, lA[sn_], xn_[0], xn_[1]); fix_x(hb_, rc_.ev, 8, uc, lB[sn_], xn_[2], xn_[3]); } \
        if (sg) wait_step(lds, (J) + 1, seen_); \
        } while (0)
#define GRS_STEP2(s, ui, uc, hc, hm, sg, J) do { \
        constexpr int sn_ = ((s) + 1) & 3; \
        StepRegs& ri_ = R[s]; StepRegs& rc_ = R[sn_]; \
        u32x4 (&xo_)[4] = X[(s) & 1]; u32x4 (&xn_)[4] = X[sn_ & 1]; \
        const u32 uo_ = (ui) * USTR; \
        u32 H_[4], lb4_, hw_, ha_, hb_, seen_; \
        LAS unsigned char* wp_ = wdst + sn_ * SLOTB; \
        SB; \
        VM_WAIT_SET(14, rc_); \
        MF(0, wf[1], xn_[3]); RD1(s, 0, 0); ld_wn(ri_, rsl, oW[s] + uo_); \
            hw_ = rdl(hc, lW[sn_]) & (hm); lb4_ = LB4(hw_); H_[0] = (rc_.w.n[0] & 0x87878787u) + lb4_; H_[1] = (rotr4(rc_.w.n[0]) & 0x87878787u) + lb4_; SB; \
        MF(1, wf[1], xn_[3]); RD1(s, 0, 1); ld_wl(ri_, rsl, oW[s] + uo_); \
            *(LAS u32x2*)wp_ = (u32x2){PE(H_, rc_.w.l, 0), PE(H_, rc_.w.l, 1)}; *(LAS u32x2*)(wp_ + 1024) = (u32x2){PO(H_, rc_.w.l, 0), PO(H_, rc_.w.l, 1)}; SB; \
        fix_w(rc_, uc, sn_, hw_); SB; \
        MF(0, wf[0], xo_[0]); RD1(s, 1, 0); ald8(ri_.xa.n, vn, rsl, oA[s] + uo_); \
            ha_ = rdl(hc, lA[sn_]) & (hm); lb4_ = LB4(ha_); HI01(H_, rc_.xa.n); SB; \
        MF(1, wf[0], xo_[0]); RD1(s, 1, 1); ald16(ri_.xa.l, vl, rsl, oA[s] + uo_); HI23(H_, rc_.xa.n); xn_[0] = XPAIR(H_, rc_.xa.l, 0); SB; \
        asm volatile("s_waitcnt lgkmcnt(2)" ::: "memory"); sig_step(cntv + (((s) + 1) * 4), sg); \
        MF(0, wf[1], xo_[1]); RD1(s, 2, 0); ald8(ri_.xb.n, vn, rsl, oB[s] + uo_); \
            xn_[1] = XPAIR(H_, rc_.xa.l, 2); hb_ = rdl(hc, lB[sn_]) & (hm); lb4_ = LB4(hb_); SB; \
        MF(1, wf[1], xo_[1]); RD1(s, 2, 1); ald16(ri_.xb.l, vl, rsl, oB[s] + uo_); \
            HI01(H_, rc_.xb.n); seen_ = *(volatile LAS u32*)(lds + LDS_CNT + ((J) + 1) * 4u); SB; \
        MF(0, wf[0], xo_[2]); RD1(s, 3, 0); ald4(ri_.ev, eV[s], rsl, (ui) * 1024u); HI23(H_, rc_.xb.n); xn_[2] = XPAIR(H_, rc_.xb.l, 0); SB; \
        MF(1, wf[0], xo_[2]); RD1(s, 3, 1); xn_[3] = XPAIR(H_, rc_.xb.l, 2); SB; \
        if (((ha_ | hb_) & fixmask) != 0u) { fix_x(ha_, rc_.ev, 0, uc, lA[sn_], xn_[0], xn_[1]); fix_x(hb_, rc_.ev, 8, uc, lB[sn_], xn_[2], xn_[3]); } \
        if (sg) wait_step(lds, (J) + 1, seen_); \
        } while (0)
#define GRS_STEP(s, ui, uc, hc, hm, sg, J) do { if constexpr (NC == 4) GRS_STEP4(s, ui, uc, hc, hm, sg, J); else GRS_STEP2(s, ui, uc, hc, hm, sg, J); } while (0)
    const u32 notag = ((tag + 1u) & TAGMASK) << TAGSH;
    u32 hv = 0; bool cold = true, pf_ok = false;
    LOOK_SET("v228", notag); LOOK_SET("v229", notag);
    for (u32 u = 0; u < 8; ++u) {
        const u32 J = 4u * u, cntv = cnt0 + J * 4u, un = u < 7u ? u + 1u : 7u;
        u32 pv; LOOK_GET("v228", 14, pv);
        if (!pf_ok) pv = notag;
        if (cold) {
            LOOK_GET("v229", 7, hv);
            if (ok && !__all((hv >> TAGSH) == tag)) ok = wait_tags(my + fo + u * 128u, tag, hv, my, call);
            STAMP(6 + 2 * u); STAMP2(u, 4);
            issue(R[0], u, 0); issue(R[1], u, 1);
            LOOK_LD("v230", fo, rsw, un * 128u);
            issue(R[2], u, 2); issue(R[3], u, 3);
            const u32 hw_ = rdl(hv, lW[0]), ha_ = rdl(hv, lA[0]), hb_ = rdl(hv, lB[0]);
            u32 H_[4], lb4_ = LB4(hw_);
            VM_WAIT_SET(22, R[0]);
            if constexpr (NC == 4) {
                HI01(H_, R[0].w.n); HI23(H_, R[0].w.n);
                *(LAS u32x4*)wdst = (u32x4){PE(H_, R[0].w.l, 0), PE(H_, R[0].w.l, 1), PE(H_, R[0].w.l, 2), PE(H_, R[0].w.l, 3)};
                *(LAS u32x4*)(wdst + 1024) = (u32x4){PO(H_, R[0].w.l, 0), PO(H_, R[0].w.l, 1), PO(H_, R[0].w.l, 2), PO(H_, R[0].w.l, 3)};
            } else {
                H_[0] = (R[0].w.n[0] & 0x87878787u) + lb4_; H_[1] = (rotr4(R[0].w.n[0]) & 0x87878787u) + lb4_;
                *(LAS u32x2*)wdst = (u32x2){PE(H_, R[0].w.l, 0), PE(H_, R[0].w.l, 1)}; *(LAS u32x2*)(wdst + 1024) = (u32x2){PO(H_, R[0].w.l, 0), PO(H_, R[0].w.l, 1)};
            }
            fix_w(R[0], u, 0, hw_);
            asm volatile("s_waitcnt lgkmcnt(0)" ::: "memory");
            sig_step(cntv, 1u);
            lb4_ = LB4(ha_); HI01(H_, R[0].xa.n); HI23(H_, R[0].xa.n); X[0][0] = XPAIR(H_, R[0].xa.l, 0); X[0][1] = XPAIR(H_, R[0].xa.l, 2);
            lb4_ = LB4(hb_); HI01(H_, R[0].xb.n); HI23(H_, R[0].xb.n); X[0][2] = XPAIR(H_, R[0].xb.l, 0); X[0][3] = XPAIR(H_, R[0].xb.l, 2);
            fix_x(ha_, R[0].ev, 0, u, lA[0], X[0][0], X[0][1]); fix_x(hb_, R[0].ev, 8, u, lB[0], X[0][2], X[0][3]);
            LOOK_GET("v230", 14, pv);
            STAMP2(8 + u, 0);
            wait_step(lds, J, 0u);
            STAMP2(16 + u, 0);
        }
        bool have = ok && u < 7u && __all((pv >> TAGSH) == tag);
        u32 hn = pv;
        const bool look = ok && u < 7u;
        if (have) STAMP2(un, 0);
#define RSL (u32x4){rs0, rs1, have ? WS_NEED : 0u, RSRC_W3}
#define LATE(s) do { if (!have && look) { u32 v_; LOOK_GET("v229", 7, v_); \
            if (__all((v_ >> TAGSH) == tag)) { have = true; hn = v_; STAMP2(un, s); issue(R[0], un, 0); if ((s) > 1) issue(R[1], un, 1); if ((s) > 2) issue(R[2], un, 2); } \
            else LOOK_LD("v229", fo, rsw, un * 128u); } } while (0)
        if (!have && look) LOOK_LD("v229", fo, rsw, un * 128u);
        { const u32x4 rsl = RSL; GRS_STEP(0, un, u, hv, 0xffffffffu, 1u, J); }
        LATE(1);
        { const u32x4 rsl = RSL; GRS_STEP(1, un, u, hv, 0xffffffffu, 1u, J + 1); }
        LOOK_LD("v228", fo, rsw, (u < 6u ? u + 2u : 7u) * 128u); pf_ok = have;
        LATE(2);
        { const u32x4 rsl = RSL; GRS_STEP(2, un, u, hv, 0xffffffffu, 1u, J + 2); }
        LATE(3);
        STAMP2(24 + u, 0);
        { const u32x4 rsl = RSL; GRS_STEP(3, un, un, hn, (have ? 0xffffffffu : 0u), (have ? 1u : 0u), J + 3); }
#undef LATE
#undef RSL
        hv = hn; cold = !have;
        STAMP(7 + 2 * u);
    }
#pragma unroll
    for (int c = 0; c < NC; ++c) MF(c, wf[1], X[1][3]);
#undef GRS_STEP
#undef GRS_STEP2
#undef GRS_STEP4
    STAMP(2);
    if (!ok || rfl(agent_load(my + OFF_TMO)) == call) return;
    float mx = 0.f;
#pragma unroll
    for (int c = 0; c < NC; ++c)
#pragma unroll
        for (int g = 0; g < 16; g += 2) mx = fmaxf(mx, fmaxf(fabsf(acc[c][g]), fabsf(acc[c][g + 1])));
    const u32 em = wave_umax((cvt_pk_bf16(mx, mx) >> 8) & 0x7fu);
    const u32 lb = em > 7u ? em - 7u : 0u;
    const rsrc_t dst = mk_rsrc(P.ws[q], WS_NEED);
    const u32 tau = ((me & 1u) * 2u + ((me >> 1) & 1u)) * 16u + (u32)n, k = me >> 2;
    u32 hd[2] = {0u, 0u};
#pragma unroll
    for (int b2 = 0; b2 < NC / 2; ++b2) {
        u32 L[8], H[8];
#pragma unroll
        for (int t = 0; t < 8; ++t) {
            u32 d0, d1;
            if constexpr (NC == 4) { const int g = 8 * b2 + t; d0 = cvt_pk_bf16(acc[0][g], acc[1][g]); d1 = cvt_pk_bf16(acc[2][g], acc[3][g]); }
            else { d0 = cvt_pk_bf16(acc[0][2 * t], acc[1][2 * t]); d1 = cvt_pk_bf16(acc[0][2 * t + 1], acc[1][2 * t + 1]); }
            L[t] = perm(d1, d0, 0x06040200u); H[t] = perm(d1, d0, 0x07050301u);
        }
        const u32 beta = hb * 2u + (NC == 4 ? (u32)b2 : ht);
        const u32 blk = ((k * 64 + tau) * 4 + b) * 4 + beta;
        hd[b2] = pack_store(L, H, lb, dst, OFF_PP + blk * BLK, OFF_PE + blk * ENT, OFF_PR + blk * RAWB, lane, false);
    }
    STAMP(3);
    drain();
    if (lane < NC / 2)
        sys_store(P.ws[q] + OFF_PFLAG + (tau * 4 + b) * 128u + (k * 4 + hb * 2 + (NC == 4 ? (u32)lane : ht)) * 4u, (tag << TAGSH) | (lane == 0 ? hd[0] : hd[1]));
    STAMP(4);
}

struct Blk { Half h[2]; };
__device__ __forceinline__ void unpack_f32(const Blk& k, u32 hdr, u32 entv, u32 ebase, rsrc_t ws, u32 raw_off, float (&f)[32], int lane) {
#pragma unroll
    for (int hb = 0; hb < 2; ++hb) {
        u32 H[4]; high_bytes(k.h[hb].n, (u32)hb, hdr, entv, ebase, H, lane);
#pragma unroll
        for (int i = 0; i < 4; ++i) {
            const int t = 4 * hb + i;
            f[4 * t]     = __uint_as_float(perm(H[i], k.h[hb].l[i], 0x04000c0cu));
            f[4 * t + 1] = __uint_as_float(perm(H[i], k.h[hb].l[i], 0x05010c0cu));
            f[4 * t + 2] = __uint_as_float(perm(H[i], k.h[hb].l[i], 0x06020c0cu));
            f[4 * t + 3] = __uint_as_float(perm(H[i], k.h[hb].l[i], 0x07030c0cu));
        }
    }
    if (hdr & RAWBIT) {
#pragma unroll
        for (int c = 0; c < 4; ++c) { const u32x4 o = ld16(ws, (u32)lane * 16u, raw_off + 1024u * c);
#pragma unroll
            for (int i = 0; i < 4; ++i) { f[8 * c + 2 * i] = __uint_as_float(o[i] << 16); f[8 * c + 2 * i + 1] = __uint_as_float(o[i] & 0xffff0000u); } }
    }
}

__device__ __forceinline__ void reducer_wave(const Params& P, int be, int lane, u32 call, u32 tag, bool ok, LAS unsigned char* lds, char* my, int wave) {
    const int wg = blockIdx.x;
    const u32 tau = (u32)wg >> 2, wm = (u32)wg & 3u;
    const rsrc_t ws = mk_rsrc(my, WS_NEED);
    LAS u32* ctl = (LAS u32*)(lds + LDS_CTL);
    u32 hv = 0;
    if (ok) ok = wait_tags(my + OFF_PFLAG + (tau * 4 + wm) * 128u + (u32)((lane & 1) * 4 + be) * 4u, tag, hv, my, call);
    STAMP(22);
    float y[32];
    {
        const u32 b0 = ((0 * 64 + tau) * 4 + wm) * 4 + (u32)be, b1 = ((1 * 64 + tau) * 4 + wm) * 4 + (u32)be;
        const u32 vo = (u32)lane * 16u;
        const u32 von = (u32)(lane & 31) * 16u + (u32)(lane >> 5) * 8u;
        Blk k0, k1;
#pragma unroll
        for (int hb = 0; hb < 2; ++hb) {
            k0.h[hb].n = ld8(ws, von, OFF_PP + b0 * BLK + 2048u + hb * 512u); k0.h[hb].l = ld16(ws, vo, OFF_PP + b0 * BLK + hb * 1024u);
            k1.h[hb].n = ld8(ws, von, OFF_PP + b1 * BLK + 2048u + hb * 512u); k1.h[hb].l = ld16(ws, vo, OFF_PP + b1 * BLK + hb * 1024u);
        }
        const u32 ev = ld4(ws, (lane < 8 ? OFF_PE + b0 * ENT : OFF_PE + b1 * ENT) + (u32)(lane & 7) * 4u, 0);
        cfence();
        float z[32];
        unpack_f32(k0, rdl(hv, 0), ev, 0, ws, OFF_PR + b0 * RAWB, y, lane);
        unpack_f32(k1, rdl(hv, 1), ev, 8, ws, OFF_PR + b1 * RAWB, z, lane);
#pragma unroll
        for (int e = 0; e < 32; ++e) y[e] += z[e];
    }
    float mx = 0.f;
#pragma unroll
    for (int e = 0; e < 32; e += 2) mx = fmaxf(mx, fmaxf(fabsf(y[e]), fabsf(y[e + 1])));
    u32 mb = __float_as_uint(mx);
    mb = wave_umax(mb);
    STAMP(23);
    u32 last = 0;
    if (lane == 0) {
        __hip_atomic_fetch_max(ctl + 0, mb, __ATOMIC_RELAXED, __HIP_MEMORY_SCOPE_WORKGROUP);
        if (!ok) __hip_atomic_store(ctl + 4, 1u, __ATOMIC_RELAXED, __HIP_MEMORY_SCOPE_WORKGROUP);
        last = __hip_atomic_fetch_add(ctl + 1, 1u, __ATOMIC_ACQ_REL, __HIP_MEMORY_SCOPE_WORKGROUP);
    }
    last = rfl(last);
    u32 gbits = 0; bool good;
    if (last == 3u) {
        const u32 wmax = rfl(__hip_atomic_load(ctl + 0, __ATOMIC_RELAXED, __HIP_MEMORY_SCOPE_WORKGROUP));
        const u32 fail = rfl(__hip_atomic_load(ctl + 4, __ATOMIC_RELAXED, __HIP_MEMORY_SCOPE_WORKGROUP));
        good = fail == 0u;
        if (good) {
            if (lane == 0) sys_store64(my + OFF_LG + (u32)wg * 8u, ((u64)call << 32) | wmax);
            const u64 t0 = __builtin_amdgcn_s_memrealtime();
            for (;;) {
                u64 x = 0;
                if (lane < 8) x = sys_load64(my + OFF_GG + (u32)(wg & 7) * 128u + (u32)lane * 8u);
                else if (lane == 8) x = (u64)agent_load(my + OFF_TMO);
                const bool tm = __any(lane == 8 && (u32)x == call);
                const bool all = __all(lane >= 8 || (u32)(x >> 32) == call);
                if (tm) { good = false; break; }
                if (all) { gbits = wave_umax(lane < 8 ? (u32)x : 0u); break; }
                if (__builtin_amdgcn_s_memrealtime() - t0 > DEADLINE) { agent_store(my + OFF_TMO, call); good = false; break; }
                __builtin_amdgcn_s_sleep(1);
            }
        }
        if (lane == 0) {
            __hip_atomic_store(ctl + 2, gbits, __ATOMIC_RELAXED, __HIP_MEMORY_SCOPE_WORKGROUP);
            __hip_atomic_store(ctl + 3, good ? 1u : 2u, __ATOMIC_RELEASE, __HIP_MEMORY_SCOPE_WORKGROUP);
        }
    } else {
        u32 st;
        while ((st = rfl(__hip_atomic_load(ctl + 3, __ATOMIC_ACQUIRE, __HIP_MEMORY_SCOPE_WORKGROUP))) == 0u) __builtin_amdgcn_s_sleep(1);
        good = st == 1u;
        gbits = rfl(__hip_atomic_load(ctl + 2, __ATOMIC_RELAXED, __HIP_MEMORY_SCOPE_WORKGROUP));
    }
    STAMP(24);
    const float amax = __uint_as_float(gbits);
    const float scale = amax / 448.0f;
    const float inv = 1.0f / scale;
    u32x4 o0, o1;
#pragma unroll
    for (int i = 0; i < 8; ++i) {
        u32 v = (u32)__builtin_amdgcn_cvt_pk_fp8_f32(y[4 * i] * inv, y[4 * i + 1] * inv, 0, false);
        v = (u32)__builtin_amdgcn_cvt_pk_fp8_f32(y[4 * i + 2] * inv, y[4 * i + 3] * inv, (int)v, true);
        if (!good) v = 0x7f7f7f7fu;
        if (i < 4) o0[i] = v; else o1[i - 4] = v;
    }
    const u32 qi = tau >> 5, qj = (tau >> 4) & 1u, n = tau & 15u;
    const u32 row = qi * 256u + wm * 64u + (u32)(be >> 1) * 32u + (u32)(lane & 31);
    const u32 col = qj * 2048u + n * 128u + (is_narrow(n) ? (u32)(be & 1) * 64u + (u32)(lane >> 5) * 32u : (u32)(lane >> 5) * 64u + (u32)(be & 1) * 32u);
    const rsrc_t out = mk_rsrc(P.out, 2u << 20);
    __builtin_amdgcn_raw_buffer_store_b128(o0, out, (int)(row * 4096u + col), 0, 0);
    __builtin_amdgcn_raw_buffer_store_b128(o1, out, (int)(row * 4096u + col + 16u), 0, 0);
    STAMP(25); STAMPC(31);
}

__device__ __forceinline__ void leader_wave(const Params& P, int lane, u32 call, char* my, int wave) {
    const u64 t0 = __builtin_amdgcn_s_memrealtime();
    u32 m = 0; bool good = true;
    for (u32 it = 0;; ++it) {
        bool all = true; m = 0;
#pragma unroll
        for (int i = 0; i < 4; ++i) { const u64 x = sys_load64(my + OFF_LG + (u32)(i * 64 + lane) * 8u); all = all && (u32)(x >> 32) == call; m = max(m, (u32)x); }
        if (__all(all)) break;
        if ((it & 15u) == 15u && (rfl(agent_load(my + OFF_TMO)) == call || __builtin_amdgcn_s_memrealtime() - t0 > DEADLINE)) {
            agent_store(my + OFF_TMO, call); good = false; break; }
        __builtin_amdgcn_s_sleep(1);
    }
    STAMP(26);
    if (good) {
        m = wave_umax(m);
        sys_store64(P.ws[lane >> 3] + OFF_GG + (u32)(lane & 7) * 128u + (u32)P.rank * 8u, ((u64)call << 32) | m);
    }
    if (lane == 0) agent_store(my + OFF_CALL, call);
}

__global__ void __launch_bounds__(512, 2) gemm_rs_fused(Params P) {
    __shared__ __attribute__((aligned(16))) unsigned char smem[LDS_BYTES];
    LAS unsigned char* lds = (LAS unsigned char*)smem;
    const int tid = threadIdx.x, lane = tid & 63, wave = (int)rfl((u32)tid >> 6);
    const int b = blockIdx.x, xcd = b & 7, s = b >> 3;
    char* my = P.ws[P.rank];
    if (tid < 64) ((LAS u32*)(lds + LDS_CTL))[tid] = 0u;
    const u32 call = rfl(agent_load(my + OFF_CALL)) + 1u;
    const u32 tag = call & TAGMASK;
    __syncthreads();
    STAMP(0); STAMPC(30);
    const int tq = 4 * (xcd & 1) + (s & 3), tn = 2 * (xcd >> 1) + ((s >> 2) & 1);
#define GRS_TILES do { \
    if (GRS_CFG == 0) { if (s < 24) gemm_wave<4>(P, tq, tn + (s >= 16 ? 8 : 0), 0u, wave, lane, call, tag, lds, my, wave); } \
    else if (GRS_CFG == 1) { \
        if (s < 24) gemm_wave<2>(P, tq, tn, s >= 16 ? 1u : 0u, wave, lane, call, tag, lds, my, wave); \
        else gemm_wave<4>(P, tq, tn + 8, 0u, wave, lane, call, tag, lds, my, wave); \
    } else { \
        if (s < 16) gemm_wave<4>(P, tq, tn, 0u, wave, lane, call, tag, lds, my, wave); \
        else gemm_wave<2>(P, tq, tn + 8, s >= 24 ? 1u : 0u, wave, lane, call, tag, lds, my, wave); \
    } } while (0)
    if (s < 8) {
        if ((u32)wave < SEND_WPW) sender_wave(P, s, (u32)xcd * SEND_WPW + (u32)wave, lane, call, tag, my);
        STAMP(1);
        if (b == 0 && wave == 0) leader_wave(P, lane, call, my, wave);
    } else GRS_TILES;
    if (wave >= 4) reducer_wave(P, wave - 4, lane, call, tag, true, lds, my, wave);
}

}

extern "C" void kernel_launch(int rank, int nranks, ncclComm_t comm, hipStream_t stream,
                              void* const* d_in, const int* in_sizes, int n_in,
                              void* d_out, int out_size,
                              const int* in_dtypes, int out_dtype,
                              void* const* d_out_all,
                              void* const* d_ws_all, size_t ws_bytes) {
    (void)comm; (void)in_dtypes; (void)out_dtype; (void)d_out_all;
    if (nranks != grs::NR || n_in < 2 || ws_bytes < grs::WS_NEED || out_size != 512 * 4096 ||
        in_sizes[0] != 4096 * 512 || in_sizes[1] != 512 * 4096) {
        fprintf(stderr, "gemm_rs: unexpected problem geometry\n");
        return;
    }
    grs::Params p;
    p.x = d_in[0]; p.w = d_in[1]; p.out = d_out;
    for (int i = 0; i < grs::NR; ++i) p.ws[i] = (char*)d_ws_all[i];
    p.rank = rank; p.pad = 0;
    hipLaunchKernelGGL(grs::gemm_rs_fused, dim3(256), dim3(512), 0, stream, p);
    HIP_CHECK(hipGetLastError());
}
```

```cpp
#include <hip/hip_runtime.h>
#include <rccl/rccl.h>
#include <cstdio>
#include <cstdint>

#define HIP_CHECK(expr) do { hipError_t _e = (expr); if (_e != hipSuccess) { \
    fprintf(stderr, "HIP error %s at %s:%d\n", hipGetErrorString(_e), __FILE__, __LINE__); \
    return; } } while (0)

namespace grs {

typedef unsigned u32;
typedef unsigned long long u64;
typedef unsigned u32x4 __attribute__((ext_vector_type(4)));
typedef unsigned u32x2 __attribute__((ext_vector_type(2)));
typedef unsigned short u16x2 __attribute__((ext_vector_type(2)));
typedef float f32x16 __attribute__((ext_vector_type(16)));
typedef short bf16x8 __attribute__((ext_vector_type(8)));
typedef __amdgpu_buffer_rsrc_t rsrc_t;
#define LAS __attribute__((address_space(3)))
#define GAS __attribute__((address_space(1)))

constexpr int NR = 8;
constexpr int AUX_SYS = 17;
constexpr u32 RSRC_W3 = 0x00020000u;
constexpr u64 DEADLINE = 20000000ull;
constexpr u32 TAGSH = 14, TAGMASK = 0x3ffffu, RAWBIT = 1u << 13;

constexpr u32 OFF_CALL  = 0;
constexpr u32 OFF_TMO   = 128;
constexpr u32 OFF_DUMMY = 131072;
constexpr u32 OFF_XFLAG = 4096;
constexpr u32 OFF_WFLAG = 16384;
constexpr u32 OFF_PFLAG = 32768;
constexpr u32 OFF_SCTR  = 2048;
constexpr u32 OFF_LG    = 65536;
constexpr u32 OFF_GG    = 69632;
constexpr u32 OFF_DBG   = 48u << 20;
constexpr u32 BLK = 3072, ENT = 32, RAWB = 4096;
constexpr u32 OFF_XP = 1u << 20, OFF_WP = 7u << 20, OFF_PP = 13u << 20;
constexpr u32 OFF_XE = 19u << 20, OFF_WE = OFF_XE + 65536, OFF_PE = OFF_WE + 65536;
constexpr u32 OFF_XR = 20u << 20, OFF_WR = 28u << 20, OFF_PR = 36u << 20;
constexpr u32 WS_NEED = 44u << 20;

constexpr u32 SLOT = 16384, UNITB = 8192;
constexpr u32 LDS_CTL = 4 * SLOT;
constexpr u32 LDS_BYTES = 96 * 1024;

#define GRS_NARROW 9
#define GRS_D1 2
#define GRS_D2 3
#define GRS_D3 5
constexpr unsigned NARROW_MASK = GRS_NARROW, DEF1 = GRS_D1, DEF2 = GRS_D2, DEF3 = GRS_D3;
static_assert(DEF1 <= DEF2 && DEF2 <= DEF3 && DEF3 <= 8, "deferral must grow with the group");
static_assert(__builtin_popcount(NARROW_MASK) == 2 && NARROW_MASK < 16, "two narrow groups");
__device__ __forceinline__ bool is_narrow(unsigned n) { return (NARROW_MASK >> (n >> 2)) & 1u; }

struct Params {
    const void* x; const void* w; void* out;
    char* ws[NR];
    char* my;
    int rank; u32 pad;
};

__device__ __forceinline__ u32 perm(u32 s0, u32 s1, u32 sel) { return __builtin_amdgcn_perm(s0, s1, sel); }
__device__ __forceinline__ u32 rotr4(u32 v) { return __builtin_amdgcn_alignbit(v, v, 4); }
__device__ __forceinline__ u32 rotl4(u32 v) { return __builtin_amdgcn_alignbit(v, v, 28); }
__device__ __forceinline__ u32 rfl(u32 v) { return (u32)__builtin_amdgcn_readfirstlane((int)v); }
__device__ __forceinline__ u32 rdl(u32 v, u32 l) { return (u32)__builtin_amdgcn_readlane((int)v, (int)l); }
__device__ __forceinline__ u32 pkmax(u32 a, u32 b) {
    return __builtin_bit_cast(u32, __builtin_elementwise_max(__builtin_bit_cast(u16x2, a), __builtin_bit_cast(u16x2, b)));
}
__device__ __forceinline__ u32 wave_umax(u32 v) {
    v = max(v, (u32)__builtin_amdgcn_update_dpp(0, (int)v, 0x111, 0xf, 0xf, true));
    v = max(v, (u32)__builtin_amdgcn_update_dpp(0, (int)v, 0x112, 0xf, 0xf, true));
    v = max(v, (u32)__builtin_amdgcn_update_dpp(0, (int)v, 0x114, 0xf, 0xf, true));
    v = max(v, (u32)__builtin_amdgcn_update_dpp(0, (int)v, 0x118, 0xf, 0xf, true));
    v = max(v, (u32)__builtin_amdgcn_update_dpp(0, (int)v, 0x142, 0xa, 0xf, false));
    v = max(v, (u32)__builtin_amdgcn_update_dpp(0, (int)v, 0x143, 0xc, 0xf, false));
    return rdl(v, 63);
}
__device__ __forceinline__ void drain() { asm volatile("s_waitcnt vmcnt(0)" ::: "memory"); }
__device__ __forceinline__ void cfence() { asm volatile("" ::: "memory"); }
__device__ __forceinline__ rsrc_t mk_rsrc(const void* p, u32 bytes) { return __builtin_amdgcn_make_buffer_rsrc((void*)p, 0, (int)bytes, RSRC_W3); }
#define EXP_LDAUX AUX_SYS
__device__ __forceinline__ u32x4 ld16(rsrc_t r, u32 voff, u32 soff) { return __builtin_amdgcn_raw_buffer_load_b128(r, (int)voff, (int)soff, EXP_LDAUX); }
__device__ __forceinline__ u32 ld4(rsrc_t r, u32 voff, u32 soff) { return __builtin_amdgcn_raw_buffer_load_b32(r, (int)voff, (int)soff, AUX_SYS); }
__device__ __forceinline__ void st16(rsrc_t r, u32 voff, u32 soff, u32x4 v) { __builtin_amdgcn_raw_buffer_store_b128(v, r, (int)(voff + soff), 0, AUX_SYS); }
__device__ __forceinline__ void st4(rsrc_t r, u32 voff, u32 soff, u32 v) { __builtin_amdgcn_raw_buffer_store_b32(v, r, (int)voff, (int)soff, AUX_SYS); }
__device__ __forceinline__ u32 sys_load(const char* p) { return __hip_atomic_load((const GAS u32*)p, __ATOMIC_RELAXED, __HIP_MEMORY_SCOPE_SYSTEM); }
__device__ __forceinline__ u64 sys_load64(const char* p) { return __hip_atomic_load((const GAS u64*)p, __ATOMIC_RELAXED, __HIP_MEMORY_SCOPE_SYSTEM); }
__device__ __forceinline__ void sys_store(char* p, u32 v) { __hip_atomic_store((GAS u32*)p, v, __ATOMIC_RELAXED, __HIP_MEMORY_SCOPE_SYSTEM); }
__device__ __forceinline__ void sys_store64(char* p, u64 v) { __hip_atomic_store((GAS u64*)p, v, __ATOMIC_RELAXED, __HIP_MEMORY_SCOPE_SYSTEM); }
__device__ __forceinline__ u32 agent_load(const char* p) { return __hip_atomic_load((const GAS u32*)p, __ATOMIC_RELAXED, __HIP_MEMORY_SCOPE_AGENT); }
__device__ __forceinline__ void agent_store(char* p, u32 v) { __hip_atomic_store((GAS u32*)p, v, __ATOMIC_RELAXED, __HIP_MEMORY_SCOPE_AGENT); }

#define STAMPC(idx) do { } while (0)
#define STAMP2(idx, val) do { } while (0)
#define STAMP(idx) do { } while (0)

__device__ __forceinline__ u32 poll_sys(const char* p) { u32 x; asm volatile("global_load_dword %0, %1, off sc0 sc1\n\ts_waitcnt vmcnt(0)" : "=&v"(x) : "v"(p) : "memory"); return x; }
__device__ __forceinline__ u32 poll_agent(const char* p) { u32 x; asm volatile("global_load_dword %0, %1, off sc1\n\ts_waitcnt vmcnt(0)" : "=&v"(x) : "v"(p) : "memory"); return x; }
__device__ __forceinline__ void ld16x2_now(const char* p0, const char* p1, u32x4& a, u32x4& b) {
    asm volatile("global_load_dwordx4 %0, %2, off sc0 sc1\n\tglobal_load_dwordx4 %1, %3, off sc0 sc1\n\ts_waitcnt vmcnt(0)" : "=&v"(a), "=&v"(b) : "v"(p0), "v"(p1) : "memory");
}
__device__ __forceinline__ void give_up(char* my, u32 call) {
    asm volatile("global_store_dword %0, %1, off sc1" :: "v"(my + OFF_TMO), "v"(call) : "memory");
}
__device__ __forceinline__ bool wait_tags(const char* addr, u32 tag, u32& v, char* my, u32 call) {
    const u64 t0 = __builtin_amdgcn_s_memrealtime();
    for (u32 it = 0;; ++it) {
        const u32 x = poll_sys(addr);
        if (__all((x >> TAGSH) == tag)) { v = x; return true; }
        if ((it & 15u) == 15u) {
            if (rfl(poll_agent(my + OFF_TMO)) == call || __builtin_amdgcn_s_memrealtime() - t0 > DEADLINE) {
                give_up(my, call); v = 0; return false; }
        }
        __builtin_amdgcn_s_sleep(2);
    }
}

__device__ __forceinline__ u32 lane_emax(const u32 (&H)[8]) {
    u32 m1 = 0, m2 = 0;
#pragma unroll
    for (int t = 0; t < 8; ++t) { const u32 g = H[t] | 0x80808080u; m1 = pkmax(m1, g); m2 = pkmax(m2, g << 8); }
    const u32 m = pkmax(m1, m2);
    return max((m >> 24) & 0x7fu, (m >> 8) & 0x7fu);
}
template <bool IS_W> __device__ __forceinline__ void chunks_from(const u32 (&L)[8], const u32 (&H)[8], u32x4 (&o)[4]) {
#pragma unroll
    for (int t = 0; t < 8; ++t) {
        const u32 e = perm(H[t], L[t], 0x05010400u), d = perm(H[t], L[t], 0x07030602u);
        if (IS_W) { o[2 * (t >> 2)][t & 3] = e; o[2 * (t >> 2) + 1][t & 3] = d; }
        else      { o[t >> 1][2 * (t & 1)] = e; o[t >> 1][2 * (t & 1) + 1] = d; }
    }
}
__device__ __forceinline__ u64 pack_prep(const u32 (&H)[8], u32 lb, u32x4& np) {
    const u32 lb4 = lb * 0x01010101u;
    u32 T[8], acc = 0xffffffffu;
#pragma unroll
    for (int t = 0; t < 8; ++t) { T[t] = (H[t] | 0x80808080u) - lb4; acc &= T[t]; }
    u32 n[4];
#pragma unroll
    for (int j = 0; j < 4; ++j) {
        const u32 a = (H[2 * j] & 0x80808080u) | (T[2 * j] & 0x07070707u);
        const u32 b = (H[2 * j + 1] & 0x80808080u) | (T[2 * j + 1] & 0x07070707u);
        n[j] = a | rotl4(b);
    }
    const auto s0 = __builtin_amdgcn_permlane32_swap(n[0], n[2], false, false);
    const auto s1 = __builtin_amdgcn_permlane32_swap(n[1], n[3], false, false);
    np = (u32x4){s0[0], s1[0], s0[1], s1[1]};
    return __ballot((acc & 0x80808080u) != 0x80808080u);
}
__device__ __forceinline__ u32 pack_emit(const u32 (&L)[8], const u32 (&H)[8], u32 lb, u32x4 np, u64 bm, rsrc_t dst, u32 blk_off, u32 ent_off, u32 raw_off, int lane, bool is_w) {
    const u32 vo = (u32)lane * 16u;
    st16(dst, vo, blk_off, (u32x4){L[0], L[1], L[2], L[3]});
    st16(dst, vo, blk_off + 1024u, (u32x4){L[4], L[5], L[6], L[7]});
    st16(dst, vo, blk_off + 2048u, np);
    u32 hdr = lb;

    if (bm != 0ull) {
        const u32 lb4 = lb * 0x01010101u;
        u32 T[8];
#pragma unroll
        for (int t = 0; t < 8; ++t) T[t] = (H[t] | 0x80808080u) - lb4;
        u32 bits = 0;
#pragma unroll
        for (int t = 0; t < 8; ++t) { const u32 x = (~T[t] & 0x80808080u) >> 7;
            bits |= ((x & 1u) | ((x >> 7) & 2u) | ((x >> 14) & 4u) | ((x >> 21) & 8u)) << (4 * t); }
        const u32 nb0 = __popc(bits & 0xffffu), nb1 = __popc(bits >> 16);
        u32 n0 = 0, n1 = 0, p0 = 0, p1 = 0;
        while (bm) { const u32 l = (u32)__builtin_ctzll(bm); bm &= bm - 1; if ((u32)lane == l) { p0 = n0; p1 = n1; } n0 += rdl(nb0, l); n1 += rdl(nb1, l); }
        if (n0 + n1 <= 7u) {
            hdr |= (n0 << 7) | (n1 << 10);
            p1 += n0;
            while (bits) {
                const u32 idx = (u32)__builtin_ctz(bits); bits &= bits - 1;
                const u32 t = idx >> 2, by = idx & 3u;
                u32 hv = H[0];
#pragma unroll
                for (int tt = 1; tt < 8; ++tt) hv = (t == (u32)tt) ? H[tt] : hv;
                const u32 slot = t < 4u ? p0++ : p1++;
                st4(dst, slot * 4u, ent_off, ((u32)lane << 16) | (t << 10) | (by << 8) | ((hv >> (8 * by)) & 0xffu));
            }
        } else {
            hdr |= RAWBIT;
            u32x4 o[4]; if (is_w) chunks_from<true>(L, H, o); else chunks_from<false>(L, H, o);
#pragma unroll
            for (int c = 0; c < 4; ++c) st16(dst, vo, raw_off + 1024u * c, o[c]);
        }
    }
    return hdr;
}
__device__ __forceinline__ u32 pack_store(const u32 (&L)[8], const u32 (&H)[8], u32 lb, rsrc_t dst, u32 blk_off, u32 ent_off, u32 raw_off, int lane, bool is_w) {
    u32x4 np; const u64 bm = pack_prep(H, lb, np);
    return pack_emit(L, H, lb, np, bm, dst, blk_off, ent_off, raw_off, lane, is_w);
}
struct Half { u32x4 l; u32x2 n; };
__device__ __forceinline__ void high_bytes(u32x2 n, u32 hb, u32 hdr, u32 entv, u32 ebase, u32 (&H)[4], int lane) {
    const u32 lb4 = (hdr & 0x7fu) * 0x01010101u;
    H[0] = (n[0] & 0x87878787u) + lb4; H[1] = (rotr4(n[0]) & 0x87878787u) + lb4;
    H[2] = (n[1] & 0x87878787u) + lb4; H[3] = (rotr4(n[1]) & 0x87878787u) + lb4;
    const u32 n0 = (hdr >> 7) & 7u, first = hb ? n0 : 0u, cnt = hb ? (hdr >> 10) & 7u : n0;
    for (u32 i = 0; i < cnt; ++i) {
        const u32 en = rdl(entv, ebase + first + i);
        const u32 sh = ((en >> 8) & 3u) * 8u, t = (en >> 10) & 7u;
        const u32 m = ((u32)lane == (en >> 16)) ? (0xffu << sh) : 0u, v = (en & 0xffu) << sh;
#pragma unroll
        for (int j = 0; j < 4; ++j) { const u32 mm = ((t & 3u) == (u32)j) ? m : 0u; H[j] = (H[j] & ~mm) | (v & mm); }
    }
}

#define GRS_SW 4
constexpr u32 NBLK = 512, SEND_WPW = GRS_SW, SEND_WAVES = 8 * SEND_WPW;
#define RAWCL "v200", "v201", "v202", "v203", "v204", "v205", "v206", "v207", "v208", "v209", "v210", "v211", "v212", "v213", "v214", "v215"
#define RAW_LD(OP, REG, IMM, voff, rs, so) asm volatile("buffer_load_" OP " " REG ", %0, %1, %2 offen offset:" #IMM :: "v"(voff), "s"(rs), "s"(so) : RAWCL)
#define RAW_P1(i, A, B) "v_perm_b32 %" #i ", " B ", " A ", %16\n\t"
#define RAW_P2(i, A, B) "v_perm_b32 %" #i ", " B ", " A ", %17\n\t"
#define RAW_PICK(N, A0, B0, A1, B1, A2, B2, A3, B3, A4, B4, A5, B5, A6, B6, A7, B7) \
    asm volatile("s_waitcnt vmcnt(" #N ")\n\t" \
        RAW_P1(0, A0, B0) RAW_P1(1, A1, B1) RAW_P1(2, A2, B2) RAW_P1(3, A3, B3) RAW_P1(4, A4, B4) RAW_P1(5, A5, B5) RAW_P1(6, A6, B6) RAW_P1(7, A7, B7) \
        RAW_P2(8, A0, B0) RAW_P2(9, A1, B1) RAW_P2(10, A2, B2) RAW_P2(11, A3, B3) RAW_P2(12, A4, B4) RAW_P2(13, A5, B5) RAW_P2(14, A6, B6) RAW_P2(15, A7, B7) \
        : "=v"(L[0]), "=v"(L[1]), "=v"(L[2]), "=v"(L[3]), "=v"(L[4]), "=v"(L[5]), "=v"(L[6]), "=v"(L[7]), \
          "=v"(H[0]), "=v"(H[1]), "=v"(H[2]), "=v"(H[3]), "=v"(H[4]), "=v"(H[5]), "=v"(H[6]), "=v"(H[7]) : "s"(selL), "s"(selH) : RAWCL)
__device__ __forceinline__ u32x4 rsrc_words(const void* p, u32 bytes) { return (u32x4){rfl((u32)(size_t)p), rfl((u32)((size_t)p >> 32)) & 0xffffu, bytes, RSRC_W3}; }

template <class GetCall>
__device__ __forceinline__ void sender_wave(const Params& P, int d, u32 wid, int lane, char* my, GetCall get_call) {
    const u32 di = (u32)d & 1u, dj = ((u32)d >> 1) & 1u, dk = (u32)d >> 2, me = (u32)P.rank;
    const u32 r = (u32)lane & 31u, h = (u32)lane >> 5;
    const rsrc_t dst = mk_rsrc(P.ws[d], WS_NEED);
    const u32x4 srcx = rsrc_words(P.x, 4u << 20), srcw = rsrc_words(P.w, 4u << 20);
    const u32 vx = (di * 256u + r) * 1024u + 32u * h + dk * 512u;
    const u32 cg = 16u * ((r >> 2) & 1u) + (r & 3u) + 4u * (r >> 3);
    const u32 vw0 = (16u * h + dk * 256u) * 8192u + dj * 4096u;
    const u32 vw = vw0 + cg * 8u, vwa = vw0 + cg * 4u;
    GAS u32* ctr = (GAS u32*)(my + OFF_SCTR + (u32)d * 128u);
    auto draw = [&]() { const u32 one = 1u;
        asm volatile("s_mov_b64 exec, 1\n\tglobal_atomic_add v216, %0, %1, off sc0\n\ts_mov_b64 exec, -1" :: "v"(ctr), "v"(one) : "v216", "memory"); };
    auto unit_of = [&](u32 idx, u32& rr) {
        u32 u = 0;
#pragma unroll
        for (u32 v = 0; v < 8u; ++v) {
            const u32 sz = 40u + 8u * ((v < 8u - DEF1 ? 1u : 0u) + (v < 8u - DEF2 ? 1u : 0u) + (v < 8u - DEF3 ? 1u : 0u));
            if (u == v && idx >= sz) { idx -= sz; u = v + 1u; }
        }
        if (u < 8u) { rr = idx; return u; }
        if (idx < 8u * DEF1) { rr = 40u + (idx & 7u); return 8u - DEF1 + (idx >> 3); }
        idx -= 8u * DEF1;
        if (idx < 8u * DEF2) { rr = 48u + (idx & 7u); return 8u - DEF2 + (idx >> 3); }
        idx -= 8u * DEF2;
        rr = 56u + (idx & 7u); return 8u - DEF3 + (idx >> 3);
    };
    auto load_raw = [&](u32 u, u32 rr) {
        if (rr < 32u) {
            const u32 so = ((rr >> 2) * 512u + (rr & 3u) * 64u) * 1024u + u * 64u;
            RAW_LD("dwordx4", "v[200:203]", 0, vx, srcx, so); RAW_LD("dwordx4", "v[204:207]", 16, vx, srcx, so);
            RAW_LD("dwordx4", "v[208:211]", 0, vx, srcx, so + 32768u); RAW_LD("dwordx4", "v[212:215]", 16, vx, srcx, so + 32768u);
        } else {
            const u32 so = (u * 32u + (rr & 1u) * 8u) * 8192u + ((rr - 32u) >> 1) * 256u;
            if (is_narrow((rr - 32u) >> 1)) {
                RAW_LD("dword", "v200", 0, vwa, srcw, so);               RAW_LD("dword", "v208", 128, vwa, srcw, so);
                RAW_LD("dword", "v201", 0, vwa, srcw, so + 8192u);       RAW_LD("dword", "v209", 128, vwa, srcw, so + 8192u);
                RAW_LD("dword", "v202", 0, vwa, srcw, so + 2u * 8192u);  RAW_LD("dword", "v210", 128, vwa, srcw, so + 2u * 8192u);
                RAW_LD("dword", "v203", 0, vwa, srcw, so + 3u * 8192u);  RAW_LD("dword", "v211", 128, vwa, srcw, so + 3u * 8192u);
                RAW_LD("dword", "v204", 0, vwa, srcw, so + 4u * 8192u);  RAW_LD("dword", "v212", 128, vwa, srcw, so + 4u * 8192u);
                RAW_LD("dword", "v205", 0, vwa, srcw, so + 5u * 8192u);  RAW_LD("dword", "v213", 128, vwa, srcw, so + 5u * 8192u);
                RAW_LD("dword", "v206", 0, vwa, srcw, so + 6u * 8192u);  RAW_LD("dword", "v214", 128, vwa, srcw, so + 6u * 8192u);
                RAW_LD("dword", "v207", 0, vwa, srcw, so + 7u * 8192u);  RAW_LD("dword", "v215", 128, vwa, srcw, so + 7u * 8192u);
            } else {
                RAW_LD("dwordx2", "v[200:201]", 0, vw, srcw, so);               RAW_LD("dwordx2", "v[202:203]", 0, vw, srcw, so + 8192u);
                RAW_LD("dwordx2", "v[204:205]", 0, vw, srcw, so + 2u * 8192u);  RAW_LD("dwordx2", "v[206:207]", 0, vw, srcw, so + 3u * 8192u);
                RAW_LD("dwordx2", "v[208:209]", 0, vw, srcw, so + 4u * 8192u);  RAW_LD("dwordx2", "v[210:211]", 0, vw, srcw, so + 5u * 8192u);
                RAW_LD("dwordx2", "v[212:213]", 0, vw, srcw, so + 6u * 8192u);  RAW_LD("dwordx2", "v[214:215]", 0, vw, srcw, so + 7u * 8192u);
            }
        }
    };
    u32 L[8], H[8], lb; u32x4 np; u64 bm;
    auto pick = [&](u32 rr, bool first) {
        if (first) asm volatile("s_waitcnt vmcnt(0)" ::: "memory");
        if (rr < 32u) {
            const u32 selL = 0x06040200u, selH = 0x07050301u;
            RAW_PICK(4, "v200", "v201", "v202", "v203", "v204", "v205", "v206", "v207", "v208", "v209", "v210", "v211", "v212", "v213", "v214", "v215");
        } else {
            const u32 selL = 0x06020400u, selH = 0x07030501u;
            if (is_narrow((rr - 32u) >> 1)) RAW_PICK(4, "v200", "v201", "v202", "v203", "v204", "v205", "v206", "v207", "v208", "v209", "v210", "v211", "v212", "v213", "v214", "v215");
            else RAW_PICK(4, "v200", "v202", "v204", "v206", "v208", "v210", "v212", "v214", "v201", "v203", "v205", "v207", "v209", "v211", "v213", "v215");
        }
        const u32 em = wave_umax(lane_emax(H));
        lb = em > 7u ? em - 7u : 0u;
        bm = pack_prep(H, lb, np);
    };
    auto ticket = [&]() { u32 t; asm volatile("v_readfirstlane_b32 %0, v216" : "=s"(t)); return t; };
    u32 rr, u = unit_of(wid, rr);
    load_raw(u, rr);
    draw();
    const u32 call = get_call(), tag = call & TAGMASK;
    const u32 base = (call - 1u) * NBLK - SEND_WAVES;
    pick(rr, true);
    char* prev_flag = my + OFF_DUMMY + ((u32)d * SEND_WAVES + wid) * 128u; u32 prev_word = 0;
#define SSTAMP() do { } while (0)
    while (true) {
        const u32 nxt = ticket() - base;
        const bool more = nxt < NBLK;
        u32 nrr = 0, nu = 0;
        if (more) nu = unit_of(nxt, nrr);
        const bool is_w = rr >= 32u;
        u32 blk, offP, offE, offR, flag;
        if (!is_w) { blk = ((me * 8 + u) * 8 + (rr >> 2)) * 4 + (rr & 3u); offP = OFF_XP; offE = OFF_XE; offR = OFF_XR;
                     flag = OFF_XFLAG + ((rr >> 2) * 8 + u) * 128u + ((rr & 3u) * 8 + me) * 4u; }
        else       { const u32 n = (rr - 32u) >> 1, kf = rr & 1u; blk = ((me * 8 + u) * 16 + n) * 2 + kf; offP = OFF_WP; offE = OFF_WE; offR = OFF_WR;
                     flag = OFF_WFLAG + (n * 8 + u) * 128u + (kf * 8 + me) * 4u; }
        drain();
        if (more) { load_raw(nu, nrr); draw(); }
        if (lane == 0) sys_store(prev_flag, prev_word);
        SSTAMP();
        const u32 hdr = pack_emit(L, H, lb, np, bm, dst, offP + blk * BLK, offE + blk * ENT, offR + blk * RAWB, lane, is_w);
        prev_flag = P.ws[d] + flag; prev_word = (tag << TAGSH) | hdr;
        if (!more) break;
        u = nu; rr = nrr;
        pick(rr, false);
    }
    drain();
    if (lane == 0) sys_store(prev_flag, prev_word);
    SSTAMP();
}

__device__ __forceinline__ void lds_barrier() { asm volatile("s_waitcnt lgkmcnt(0)\n\ts_barrier" ::: "memory"); }
__device__ __forceinline__ u32 cvt_pk_bf16(float lo, float hi) { u32 r; asm("v_cvt_pk_bf16_f32 %0, %1, %2" : "=v"(r) : "v"(lo), "v"(hi)); return r; }
__device__ __forceinline__ u32x2 ld8(rsrc_t r, u32 voff, u32 soff) { return __builtin_amdgcn_raw_buffer_load_b64(r, (int)voff, (int)soff, EXP_LDAUX); }
struct StepRegs { Half xa, xb, w; u32 ev; };

__device__ __forceinline__ void ald16(u32x4& d, u32 voff, u32x4 rs, u32 soff) { asm volatile("buffer_load_dwordx4 %0, %1, %2, %3 offen sc0 sc1" : "+v"(d) : "v"(voff), "s"(rs), "s"(soff)); }
__device__ __forceinline__ void ald8(u32x2& d, u32 voff, u32x4 rs, u32 soff) { asm volatile("buffer_load_dwordx2 %0, %1, %2, %3 offen sc0 sc1" : "+v"(d) : "v"(voff), "s"(rs), "s"(soff)); }
__device__ __forceinline__ void ald4(u32& d, u32 voff, u32x4 rs, u32 soff) { asm volatile("buffer_load_dword %0, %1, %2, %3 offen sc0 sc1" : "+v"(d) : "v"(voff), "s"(rs), "s"(soff)); }
#define LOOK_SET(R, val) asm volatile("v_mov_b32 " R ", %0" :: "s"(val) : R)
#define LOOK_LD(R, voff, rs, soff) asm volatile("buffer_load_dword " R ", %0, %1, %2 offen sc0 sc1" :: "v"(voff), "s"(rs), "s"(soff) : R)
#define LOOK_GET(R, N, dstv) asm volatile("s_waitcnt vmcnt(" #N ")\n\tv_mov_b32 %0, " R : "=v"(dstv))
#define VM_WAIT_SET(N, r) asm volatile("s_waitcnt vmcnt(" #N ")" : "+v"(r.w.l), "+v"(r.w.n), "+v"(r.xa.l), "+v"(r.xa.n), "+v"(r.xb.l), "+v"(r.xb.n), "+v"(r.ev))
constexpr u32 LDS_CNT = LDS_CTL + 64;
__device__ __forceinline__ void sig_step(u32 cnt_addr, u32 add) {
    asm volatile("s_mov_b64 exec, 1\n\tds_add_u32 %0, %1\n\ts_mov_b64 exec, -1" :: "v"(cnt_addr), "v"(add) : "memory");
}
__device__ __forceinline__ void wait_step(LAS unsigned char* lds, u32 J, u32 seen) {
    if (rfl(seen) == 8u) return;
    while (rfl(__hip_atomic_load((LAS u32*)(lds + LDS_CNT) + J, __ATOMIC_RELAXED, __HIP_MEMORY_SCOPE_WORKGROUP)) != 8u) __builtin_amdgcn_s_sleep(1);
    cfence();
}

#define GRS_PK4 3
#define GRS_PK2 3
template <int NC>
__device__ __forceinline__ void gemm_wave(const Params& P, int q, int n, u32 ht, int w, int lane, u32 call, u32 tag, LAS unsigned char* lds, char* my, int wave) {
    constexpr u32 PUSH_K = NC == 4 ? GRS_PK4 : GRS_PK2;
    constexpr u32 SLOTB = NC * 4096u, SUBB = NC * 1024u;
    const rsrc_t ws = mk_rsrc(my, WS_NEED);
    const u32 me = (u32)P.rank;
    const u32 vo = (u32)lane * 16u, von = (u32)(lane & 31) * 16u + (u32)(lane >> 5) * 8u;
    const u32 b = (u32)w >> 1, hb = (u32)w & 1u;
    const u32 wu = (u32)w >> 2, kf = ((u32)w >> 1) & 1u;
    const u32 wh = NC == 4 ? hb : ht;
    bool ok = true;
    const u32 fo = (lane & 8) == 0 ? OFF_XFLAG + (u32)(q * 8) * 128u + (b * 8 + (u32)(lane & 7)) * 4u
                                   : OFF_WFLAG + (u32)(n * 8) * 128u + (kf * 8 + (u32)(lane & 7)) * 4u;
    const u32 elo = (u32)(lane & 7) * 4u;
    StepRegs R[4];
#pragma unroll
    for (int s = 0; s < 4; ++s) { R[s].w.l = (u32x4){0u, 0u, 0u, 0u}; R[s].xa.l = R[s].w.l; R[s].xb.l = R[s].w.l; R[s].w.n = (u32x2){0u, 0u}; R[s].xa.n = R[s].w.n; R[s].xb.n = R[s].w.n; R[s].ev = 0u; }
    constexpr u32 USTR = 32u * BLK;
    const u32 vl = vo + hb * 1024u, vn = von + 2048u + hb * 512u;
    const u32 vwl = NC == 4 ? vl : vo + ht * 1024u + hb * 8u, vwn = NC == 4 ? vn : von + 2048u + ht * 512u + hb * 4u;
    u32 oA[4], oB[4], oW[4], eV[4], lA[4], lB[4], lW[4];
#pragma unroll
    for (int s = 0; s < 4; ++s) {
        const u32 pa = (me + 2 * s) & 7u, pb = (me + 2 * s + 1) & 7u, pw = wu ? pb : pa;
        const u32 ba = ((pa * 8) * 8 + q) * 4 + b, bb = ((pb * 8) * 8 + q) * 4 + b, bw = ((pw * 8) * 16 + n) * 2 + kf;
        oA[s] = OFF_XP + ba * BLK; oB[s] = OFF_XP + bb * BLK; oW[s] = OFF_WP + bw * BLK;
        eV[s] = (lane < 8 ? OFF_XE + ba * ENT : (lane < 16 ? OFF_XE + bb * ENT : OFF_WE + bw * ENT)) + elo;
        lA[s] = pa; lB[s] = pb; lW[s] = 8u + pw;
    }
    const u32 rs0 = rfl((u32)(size_t)my), rs1 = rfl((u32)((size_t)my >> 32)) & 0xffffu;
    const u32x4 rsw = {rs0, rs1, WS_NEED, RSRC_W3};
    auto ld_wn = [&](StepRegs& r, u32x4 rs, u32 so) {
        if (NC == 4) ald8(r.w.n, vwn, rs, so); else { u32 t = r.w.n[0]; ald4(t, vwn, rs, so); r.w.n[0] = t; } };
    auto ld_wl = [&](StepRegs& r, u32x4 rs, u32 so) {
        if (NC == 4) ald16(r.w.l, vwl, rs, so); else { u32x2 t = {r.w.l[0], r.w.l[1]}; ald8(t, vwl, rs, so); r.w.l[0] = t[0]; r.w.l[1] = t[1]; } };
    auto issue = [&](StepRegs& r, u32 u, int s) {
        const u32 uo = u * USTR;
        ld_wn(r, rsw, oW[s] + uo); ld_wl(r, rsw, oW[s] + uo);
        ald8(r.xa.n, vn, rsw, oA[s] + uo); ald16(r.xa.l, vl, rsw, oA[s] + uo);
        ald8(r.xb.n, vn, rsw, oB[s] + uo); ald16(r.xb.l, vl, rsw, oB[s] + uo);
        ald4(r.ev, eV[s], rsw, u * 1024u);
    };
    LAS unsigned char* wdst = lds + (wu * 2u + kf) * SUBB + (NC == 4 ? hb * 2048u + (u32)lane * 16u : (u32)lane * 16u + hb * 8u);
    const u32 cntsh = hb ? 10u : 7u, fixmask = (7u << cntsh) | RAWBIT;
    const u32 wcntsh = wh ? 10u : 7u, wfixmask = (7u << wcntsh) | RAWBIT;
    auto fix_x = [&](u32 hdr, u32 ev, u32 ebase, u32 u, u32 p, u32x4& a, u32x4& c) {
        if ((hdr & fixmask) == 0u) return;
        if (hdr & RAWBIT) {
            const u32 bx = ((p * 8 + u) * 8 + q) * 4 + b;
            const char* rp = my + OFF_XR + bx * RAWB + hb * 2048u + vo;
            ld16x2_now(rp, rp + 1024, a, c);
        }
        const u32 cnt = (hdr >> cntsh) & 7u, first = ebase + (hb ? (hdr >> 7) & 7u : 0u);
        for (u32 i = 0; i < cnt; ++i) {
            const u32 en = rdl(ev, first + i), t = (en >> 10) & 7u, by = (en >> 8) & 3u;
            const u32 sh = (by & 1u) * 16u + 8u, D = 2u * (t & 3u) + (by >> 1);
            const u32 m = ((u32)lane == (en >> 16)) ? (0xffu << sh) : 0u, v = (en & 0xffu) << sh;
#pragma unroll
            for (int j = 0; j < 4; ++j) {
                const u32 ma = (D == (u32)j) ? m : 0u, mc = (D == (u32)(4 + j)) ? m : 0u;
                a[j] = (a[j] & ~ma) | (v & ma); c[j] = (c[j] & ~mc) | (v & mc); }
        }
    };
    auto fix_w = [&](const StepRegs& r, u32 u, int s, u32 hw) {
        if ((hw & wfixmask) == 0u) return;
        LAS unsigned char* p = lds + s * SLOTB + (wu * 2u + kf) * SUBB + (NC == 4 ? hb * 2048u : 0u);
        if (hw & RAWBIT) {
            const u32 bw = ((((me + 2 * s + wu) & 7u) * 8 + u) * 16 + n) * 2 + kf;
            const char* rp = my + OFF_WR + bw * RAWB + wh * 2048u + vo;
            u32x4 v0, v1; ld16x2_now(rp, rp + 1024, v0, v1);
            if (NC == 4) { *(LAS u32x4*)(p + lane * 16) = v0; *(LAS u32x4*)(p + 1024 + lane * 16) = v1; }
            else { *(LAS u32x2*)(p + lane * 16 + hb * 8u) = hb ? (u32x2){v0[2], v0[3]} : (u32x2){v0[0], v0[1]};
                   *(LAS u32x2*)(p + 1024 + lane * 16 + hb * 8u) = hb ? (u32x2){v1[2], v1[3]} : (u32x2){v1[0], v1[1]}; }
        }
        const u32 cnt = (hw >> wcntsh) & 7u, first = 16u + (wh ? (hw >> 7) & 7u : 0u);
        for (u32 i = 0; i < cnt; ++i) {
            const u32 en = rdl(r.ev, first + i), t = (en >> 10) & 7u, by = (en >> 8) & 3u;
            if (NC == 2 && ((t >> 1) & 1u) != hb) continue;
            if (lane == 0) p[(en >> 16) * 16u + (by >> 1) * 1024u + (t & 3u) * 4u + (by & 1u) * 2u + 1u] = (unsigned char)(en & 0xffu);
        }
        asm volatile("s_waitcnt lgkmcnt(0)" ::: "memory");
    };
    f32x16 acc[NC];
#pragma unroll
    for (int c = 0; c < NC; ++c)
#pragma unroll
        for (int g = 0; g < 16; ++g) acc[c][g] = 0.f;
    u32x4 X[2][4]; bf16x8 wf[2][NC];
#pragma unroll
    for (int c = 0; c < NC; ++c) wf[1][c] = (bf16x8){0, 0, 0, 0, 0, 0, 0, 0};
    X[1][3] = (u32x4){0u, 0u, 0u, 0u};
    LAS unsigned char* wb = lds + lane * 16;
    const u32 cnt0 = (u32)(size_t)(lds + LDS_CNT);
#define MF(c, F, XF) acc[c] = __builtin_amdgcn_mfma_f32_32x32x16_bf16(F[c], __builtin_bit_cast(bf16x8, XF), acc[c], 0, 0, 0)
#define SB __builtin_amdgcn_sched_barrier(0)
#define RD1(s, sub, c) wf[(sub) & 1][c] = *(const LAS bf16x8*)(wb + (s) * SLOTB + (sub) * SUBB + (c) * 1024)
#define LB4(h) (((h) & 0x7fu) * 0x01010101u)
#define HI01(H, n) do { H[0] = (n[0] & 0x87878787u) + lb4_; H[1] = (rotr4(n[0]) & 0x87878787u) + lb4_; } while (0)
#define HI23(H, n) do { H[2] = (n[1] & 0x87878787u) + lb4_; H[3] = (rotr4(n[1]) & 0x87878787u) + lb4_; } while (0)
#define PE(H, l, i) perm(H[i], l[i], 0x05010400u)
#define PO(H, l, i) perm(H[i], l[i], 0x07030602u)
#define XPAIR(H, l, i) (u32x4){PE(H, l, i), PO(H, l, i), PE(H, l, (i) + 1), PO(H, l, (i) + 1)}
#define GRS_STEP4(s, ui, uc, hc, hm, sg, J) do { \
        constexpr int sn_ = ((s) + 1) & 3; \
        StepRegs& ri_ = R[s]; StepRegs& rc_ = R[sn_]; \
        u32x4 (&xo_)[4] = X[(s) & 1]; u32x4 (&xn_)[4] = X[sn_ & 1]; \
        const u32 uo_ = (ui) * USTR; \
        u32 H_[4], lb4_, hw_, ha_, hb_, seen_; u32x4 e_, o_; \
        LAS unsigned char* wp_ = wdst + sn_ * SLOTB; \
        SB; \
        MF(0, wf[1], xn_[3]); RD1(s, 0, 0); ld_wn(ri_, rsl, oW[s] + uo_); hw_ = rdl(hc, lW[sn_]) & (hm); lb4_ = LB4(hw_); SB; \
        VM_WAIT_SET(15, rc_); \
        MF(1, wf[1], xn_[3]); RD1(s, 0, 1); HI01(H_, rc_.w.n); SB; \
        MF(2, wf[1], xn_[3]); RD1(s, 0, 2); ld_wl(ri_, rsl, oW[s] + uo_); HI23(H_, rc_.w.n); SB; \
        MF(3, wf[1], xn_[3]); RD1(s, 0, 3); \
            e_[0] = PE(H_, rc_.w.l, 0); e_[1] = PE(H_, rc_.w.l, 1); e_[2] = PE(H_, rc_.w.l, 2); e_[3] = PE(H_, rc_.w.l, 3); SB; \
        MF(0, wf[0], xo_[0]); RD1(s, 1, 0); ald8(ri_.xa.n, vn, rsl, oA[s] + uo_); \
            *(LAS u32x4*)wp_ = e_; o_[0] = PO(H_, rc_.w.l, 0); o_[1] = PO(H_, rc_.w.l, 1); o_[2] = PO(H_, rc_.w.l, 2); o_[3] = PO(H_, rc_.w.l, 3); SB; \
        MF(1, wf[0], xo_[0]); RD1(s, 1, 1); \
            *(LAS u32x4*)(wp_ + 1024) = o_; ha_ = rdl(hc, lA[sn_]) & (hm); lb4_ = LB4(ha_); HI01(H_, rc_.xa.n); SB; \
        fix_w(rc_, uc, sn_, hw_); SB; \
        MF(2, wf[0], xo_[0]); RD1(s, 1, 2); ald16(ri_.xa.l, vl, rsl, oA[s] + uo_); HI23(H_, rc_.xa.n); SB; \
        MF(3, wf[0], xo_[0]); RD1(s, 1, 3); xn_[0] = XPAIR(H_, rc_.xa.l, 0); SB; \
        asm volatile("s_waitcnt lgkmcnt(2)" ::: "memory"); sig_step(cntv + (((s) + 1) * 4), sg); \
        MF(0, wf[1], xo_[1]); RD1(s, 2, 0); ald8(ri_.xb.n, vn, rsl, oB[s] + uo_); xn_[1] = XPAIR(H_, rc_.xa.l, 2); SB; \
        MF(1, wf[1], xo_[1]); RD1(s, 2, 1); hb_ = rdl(hc, lB[sn_]) & (hm); lb4_ = LB4(hb_); HI01(H_, rc_.xb.n); SB; \
        MF(2, wf[1], xo_[1]); RD1(s, 2, 2); ald16(ri_.xb.l, vl, rsl, oB[s] + uo_); HI23(H_, rc_.xb.n); SB; \
        MF(3, wf[1], xo_[1]); RD1(s, 2, 3); xn_[2] = XPAIR(H_, rc_.xb.l, 0); SB; \
        MF(0, wf[0], xo_[2]); RD1(s, 3, 0); ald4(ri_.ev, eV[s], rsl, (ui) * 1024u); xn_[3] = XPAIR(H_, rc_.xb.l, 2); SB; \
        MF(1, wf[0], xo_[2]); RD1(s, 3, 1); seen_ = *(volatile LAS u32*)(lds + LDS_CNT + ((J) + 1) * 4u); SB; \
        MF(2, wf[0], xo_[2]); RD1(s, 3, 2); SB; \
        MF(3, wf[0], xo_[2]); RD1(s, 3, 3); SB; \
        if (((ha_ | hb_) & fixmask) != 0u) { fix_x(ha_, rc_.ev, 0, uc, lA[sn_], xn_[0], xn_[1]); fix_x(hb_, rc_.ev, 8, uc, lB[sn_], xn_[2], xn_[3]); } \
        if (sg) wait_step(lds, (J) + 1, seen_); \
        } while (0)
#define GRS_STEP2(s, ui, uc, hc, hm, sg, J) do { \
        constexpr int sn_ = ((s) + 1) & 3; \
        StepRegs& ri_ = R[s]; StepRegs& rc_ = R[sn_]; \
        u32x4 (&xo_)[4] = X[(s) & 1]; u32x4 (&xn_)[4] = X[sn_ & 1]; \
        const u32 uo_ = (ui) * USTR; \
        u32 H_[4], lb4_, hw_, ha_, hb_, seen_; \
        LAS unsigned char* wp_ = wdst + sn_ * SLOTB; \
        SB; \
        VM_WAIT_SET(14, rc_); \
        MF(0, wf[1], xn_[3]); RD1(s, 0, 0); ld_wn(ri_, rsl, oW[s] + uo_); \
            hw_ = rdl(hc, lW[sn_]) & (hm); lb4_ = LB4(hw_); H_[0] = (rc_.w.n[0] & 0x87878787u) + lb4_; H_[1] = (rotr4(rc_.w.n[0]) & 0x87878787u) + lb4_; SB; \
        MF(1, wf[1], xn_[3]); RD1(s, 0, 1); ld_wl(ri_, rsl, oW[s] + uo_); \
            *(LAS u32x2*)wp_ = (u32x2){PE(H_, rc_.w.l, 0), PE(H_, rc_.w.l, 1)}; *(LAS u32x2*)(wp_ + 1024) = (u32x2){PO(H_, rc_.w.l, 0), PO(H_, rc_.w.l, 1)}; SB; \
        fix_w(rc_, uc, sn_, hw_); SB; \
        MF(0, wf[0], xo_[0]); RD1(s, 1, 0); ald8(ri_.xa.n, vn, rsl, oA[s] + uo_); \
            ha_ = rdl(hc, lA[sn_]) & (hm); lb4_ = LB4(ha_); HI01(H_, rc_.xa.n); SB; \
        MF(1, wf[0], xo_[0]); RD1(s, 1, 1); ald16(ri_.xa.l, vl, rsl, oA[s] + uo_); HI23(H_, rc_.xa.n); xn_[0] = XPAIR(H_, rc_.xa.l, 0); SB; \
        asm volatile("s_waitcnt lgkmcnt(2)" ::: "memory"); sig_step(cntv + (((s) + 1) * 4), sg); \
        MF(0, wf[1], xo_[1]); RD1(s, 2, 0); ald8(ri_.xb.n, vn, rsl, oB[s] + uo_); \
            xn_[1] = XPAIR(H_, rc_.xa.l, 2); hb_ = rdl(hc, lB[sn_]) & (hm); lb4_ = LB4(hb_); SB; \
        MF(1, wf[1], xo_[1]); RD1(s, 2, 1); ald16(ri_.xb.l, vl, rsl, oB[s] + uo_); \
            HI01(H_, rc_.xb.n); seen_ = *(volatile LAS u32*)(lds + LDS_CNT + ((J) + 1) * 4u); SB; \
        MF(0, wf[0], xo_[2]); RD1(s, 3, 0); ald4(ri_.ev, eV[s], rsl, (ui) * 1024u); HI23(H_, rc_.xb.n); xn_[2] = XPAIR(H_, rc_.xb.l, 0); SB; \
        MF(1, wf[0], xo_[2]); RD1(s, 3, 1); xn_[3] = XPAIR(H_, rc_.xb.l, 2); SB; \
        if (((ha_ | hb_) & fixmask) != 0u) { fix_x(ha_, rc_.ev, 0, uc, lA[sn_], xn_[0], xn_[1]); fix_x(hb_, rc_.ev, 8, uc, lB[sn_], xn_[2], xn_[3]); } \
        if (sg) wait_step(lds, (J) + 1, seen_); \
        } while (0)
#define GRS_STEP(s, ui, uc, hc, hm, sg, J) do { if constexpr (NC == 4) GRS_STEP4(s, ui, uc, hc, hm, sg, J); else GRS_STEP2(s, ui, uc, hc, hm, sg, J); } while (0)
    const u32 notag = ((tag + 1u) & TAGMASK) << TAGSH;
    u32 hv = 0; bool cold = true, pf_ok = false;
    LOOK_SET("v228", notag); LOOK_SET("v229", notag);
    for (u32 u = 0; u < 8; ++u) {
        const u32 J = 4u * u, cntv = cnt0 + J * 4u, un = u < 7u ? u + 1u : 7u;
        u32 pv; LOOK_GET("v228", 14, pv);
        if (!pf_ok) pv = notag;
        if (cold) {
            LOOK_GET("v229", 7, hv);
            if (ok && !__all((hv >> TAGSH) == tag)) {
                ok = wait_tags(my + fo + u * 128u, tag, hv, my, call);
                if (!ok) *(volatile LAS u32*)(lds + LDS_CTL + 24) = 1u;
            }
            STAMP(6 + 2 * u); STAMP2(u, 4);
            issue(R[0], u, 0); issue(R[1], u, 1);
            LOOK_LD("v230", fo, rsw, un * 128u);
            issue(R[2], u, 2); issue(R[3], u, 3);
            const u32 hw_ = rdl(hv, lW[0]), ha_ = rdl(hv, lA[0]), hb_ = rdl(hv, lB[0]);
            u32 H_[4], lb4_ = LB4(hw_);
            VM_WAIT_SET(22, R[0]);
            if constexpr (NC == 4) {
                HI01(H_, R[0].w.n); HI23(H_, R[0].w.n);
                *(LAS u32x4*)wdst = (u32x4){PE(H_, R[0].w.l, 0), PE(H_, R[0].w.l, 1), PE(H_, R[0].w.l, 2), PE(H_, R[0].w.l, 3)};
                *(LAS u32x4*)(wdst + 1024) = (u32x4){PO(H_, R[0].w.l, 0), PO(H_, R[0].w.l, 1), PO(H_, R[0].w.l, 2), PO(H_, R[0].w.l, 3)};
            } else {
                H_[0] = (R[0].w.n[0] & 0x87878787u) + lb4_; H_[1] = (rotr4(R[0].w.n[0]) & 0x87878787u) + lb4_;
                *(LAS u32x2*)wdst = (u32x2){PE(H_, R[0].w.l, 0), PE(H_, R[0].w.l, 1)}; *(LAS u32x2*)(wdst + 1024) = (u32x2){PO(H_, R[0].w.l, 0), PO(H_, R[0].w.l, 1)};
            }
            fix_w(R[0], u, 0, hw_);
            asm volatile("s_waitcnt lgkmcnt(0)" ::: "memory");
            sig_step(cntv, 1u);
            lb4_ = LB4(ha_); HI01(H_, R[0].xa.n); HI23(H_, R[0].xa.n); X[0][0] = XPAIR(H_, R[0].xa.l, 0); X[0][1] = XPAIR(H_, R[0].xa.l, 2);
            lb4_ = LB4(hb_); HI01(H_, R[0].xb.n); HI23(H_, R[0].xb.n); X[0][2] = XPAIR(H_, R[0].xb.l, 0); X[0][3] = XPAIR(H_, R[0].xb.l, 2);
            fix_x(ha_, R[0].ev, 0, u, lA[0], X[0][0], X[0][1]); fix_x(hb_, R[0].ev, 8, u, lB[0], X[0][2], X[0][3]);
            LOOK_GET("v230", 14, pv);
            STAMP2(8 + u, 0);
            wait_step(lds, J, 0u);
            STAMP2(16 + u, 0);
        }
        bool have = ok && u < 7u && __all((pv >> TAGSH) == tag);
        u32 hn = pv;
        const bool look = ok && u < 7u;
        if (have) STAMP2(un, 0);
#define RSL (u32x4){rs0, rs1, have ? WS_NEED : 0u, RSRC_W3}
#define LATE(s) do { if (!have && look) { u32 v_; LOOK_GET("v229", 7, v_); \
            if (__all((v_ >> TAGSH) == tag)) { have = true; hn = v_; STAMP2(un, s); issue(R[0], un, 0); if ((s) > 1) issue(R[1], un, 1); if ((s) > 2) issue(R[2], un, 2); } \
            else LOOK_LD("v229", fo, rsw, un * 128u); } } while (0)
        if (!have && look) LOOK_LD("v229", fo, rsw, un * 128u);
        { const u32x4 rsl = RSL; GRS_STEP(0, un, u, hv, 0xffffffffu, 1u, J); }
        LATE(1);
        { const u32x4 rsl = RSL; GRS_STEP(1, un, u, hv, 0xffffffffu, 1u, J + 1); }
        LOOK_LD("v228", fo, rsw, (u < 6u ? u + 2u : 7u) * 128u); pf_ok = have;
        LATE(2);
        { const u32x4 rsl = RSL; GRS_STEP(2, un, u, hv, 0xffffffffu, 1u, J + 2); }
        LATE(3);
        STAMP2(24 + u, 0);
        { const u32x4 rsl = RSL; GRS_STEP(3, un, un, hn, (have ? 0xffffffffu : 0u), (have ? 1u : 0u), J + 3); }
#undef LATE
#undef RSL
        hv = hn; cold = !have;
        STAMP(7 + 2 * u);
    }
#pragma unroll
    for (int c = 0; c < NC; ++c) MF(c, wf[1], X[1][3]);
#undef GRS_STEP
#undef GRS_STEP2
#undef GRS_STEP4
    STAMP(2);
    LAS u32* ctl = (LAS u32*)(lds + LDS_CTL);
    if (!ok || rfl(__hip_atomic_load(ctl + 6, __ATOMIC_RELAXED, __HIP_MEMORY_SCOPE_WORKGROUP)) != 0u) return;
    float mx = 0.f;
#pragma unroll
    for (int c = 0; c < NC; ++c)
#pragma unroll
        for (int g = 0; g < 16; g += 2) mx = fmaxf(mx, fmaxf(fabsf(acc[c][g]), fabsf(acc[c][g + 1])));
    const u32 em = wave_umax((cvt_pk_bf16(mx, mx) >> 8) & 0x7fu);
    const u32 lb = em > 7u ? em - 7u : 0u;
    const rsrc_t dst = mk_rsrc(P.ws[q], WS_NEED);
    const u32 tau = ((me & 1u) * 2u + ((me >> 1) & 1u)) * 16u + (u32)n, k = me >> 2;
    u32 hd[2] = {0u, 0u};
    u32 L[NC / 2][8], H[NC / 2][8]; u32x4 np[NC / 2]; u64 bm[NC / 2];
#pragma unroll
    for (int b2 = 0; b2 < NC / 2; ++b2) {
#pragma unroll
        for (int t = 0; t < 8; ++t) {
            u32 d0, d1;
            if constexpr (NC == 4) { const int g = 8 * b2 + t; d0 = cvt_pk_bf16(acc[0][g], acc[1][g]); d1 = cvt_pk_bf16(acc[2][g], acc[3][g]); }
            else { d0 = cvt_pk_bf16(acc[0][2 * t], acc[1][2 * t]); d1 = cvt_pk_bf16(acc[0][2 * t + 1], acc[1][2 * t + 1]); }
            L[b2][t] = perm(d1, d0, 0x06040200u); H[b2][t] = perm(d1, d0, 0x07050301u);
        }
        bm[b2] = pack_prep(H[b2], lb, np[b2]);
    }
    if ((u32)w >= PUSH_K) while (rfl(__hip_atomic_load(ctl + 8, __ATOMIC_RELAXED, __HIP_MEMORY_SCOPE_WORKGROUP)) + PUSH_K <= (u32)w) __builtin_amdgcn_s_sleep(2);
#pragma unroll
    for (int b2 = 0; b2 < NC / 2; ++b2) {
        const u32 beta = hb * 2u + (NC == 4 ? (u32)b2 : ht);
        const u32 blk = ((k * 64 + tau) * 4 + b) * 4 + beta;
        hd[b2] = pack_emit(L[b2], H[b2], lb, np[b2], bm[b2], dst, OFF_PP + blk * BLK, OFF_PE + blk * ENT, OFF_PR + blk * RAWB, lane, false);
    }
    STAMP(3);
    drain();
    if (PUSH_K < 8u && lane == 0) __hip_atomic_fetch_add(ctl + 8, 1u, __ATOMIC_RELAXED, __HIP_MEMORY_SCOPE_WORKGROUP);
    if (lane < NC / 2)
        sys_store(P.ws[q] + OFF_PFLAG + (tau * 4 + b) * 128u + (k * 4 + hb * 2 + (NC == 4 ? (u32)lane : ht)) * 4u, (tag << TAGSH) | (lane == 0 ? hd[0] : hd[1]));
    STAMP(4);
}

struct Blk { Half h[2]; };
__device__ __forceinline__ void unpack_f32(const Blk& k, u32 hdr, u32 entv, u32 ebase, rsrc_t ws, u32 raw_off, float (&f)[32], int lane) {
#pragma unroll
    for (int hb = 0; hb < 2; ++hb) {
        u32 H[4]; high_bytes(k.h[hb].n, (u32)hb, hdr, entv, ebase, H, lane);
#pragma unroll
        for (int i = 0; i < 4; ++i) {
            const int t = 4 * hb + i;
            f[4 * t]     = __uint_as_float(perm(H[i], k.h[hb].l[i], 0x04000c0cu));
            f[4 * t + 1] = __uint_as_float(perm(H[i], k.h[hb].l[i], 0x05010c0cu));
            f[4 * t + 2] = __uint_as_float(perm(H[i], k.h[hb].l[i], 0x06020c0cu));
            f[4 * t + 3] = __uint_as_float(perm(H[i], k.h[hb].l[i], 0x07030c0cu));
        }
    }
    if (hdr & RAWBIT) {
#pragma unroll
        for (int c = 0; c < 4; ++c) { const u32x4 o = ld16(ws, (u32)lane * 16u, raw_off + 1024u * c);
#pragma unroll
            for (int i = 0; i < 4; ++i) { f[8 * c + 2 * i] = __uint_as_float(o[i] << 16); f[8 * c + 2 * i + 1] = __uint_as_float(o[i] & 0xffff0000u); } }
    }
}

__device__ __forceinline__ void reducer_wave(const Params& P, int be, int lane, u32 call, u32 tag, bool ok, LAS unsigned char* lds, char* my, int wave) {
    const int wg = blockIdx.x;
    const u32 tau = (u32)wg >> 2, wm = (u32)wg & 3u;
    const rsrc_t ws = mk_rsrc(my, WS_NEED);
    LAS u32* ctl = (LAS u32*)(lds + LDS_CTL);
    u32 hv = 0;
    if (ok) ok = wait_tags(my + OFF_PFLAG + (tau * 4 + wm) * 128u + (u32)((lane & 1) * 4 + be) * 4u, tag, hv, my, call);
    STAMP(22);
    float y[32];
    {
        const u32 b0 = ((0 * 64 + tau) * 4 + wm) * 4 + (u32)be, b1 = ((1 * 64 + tau) * 4 + wm) * 4 + (u32)be;
        const u32 vo = (u32)lane * 16u;
        const u32 von = (u32)(lane & 31) * 16u + (u32)(lane >> 5) * 8u;
        Blk k0, k1;
#pragma unroll
        for (int hb = 0; hb < 2; ++hb) {
            k0.h[hb].n = ld8(ws, von, OFF_PP + b0 * BLK + 2048u + hb * 512u); k0.h[hb].l = ld16(ws, vo, OFF_PP + b0 * BLK + hb * 1024u);
            k1.h[hb].n = ld8(ws, von, OFF_PP + b1 * BLK + 2048u + hb * 512u); k1.h[hb].l = ld16(ws, vo, OFF_PP + b1 * BLK + hb * 1024u);
        }
        const u32 ev = ld4(ws, (lane < 8 ? OFF_PE + b0 * ENT : OFF_PE + b1 * ENT) + (u32)(lane & 7) * 4u, 0);
        cfence();
        float z[32];
        unpack_f32(k0, rdl(hv, 0), ev, 0, ws, OFF_PR + b0 * RAWB, y, lane);
        unpack_f32(k1, rdl(hv, 1), ev, 8, ws, OFF_PR + b1 * RAWB, z, lane);
#pragma unroll
        for (int e = 0; e < 32; ++e) y[e] += z[e];
    }
    float mx = 0.f;
#pragma unroll
    for (int e = 0; e < 32; e += 2) mx = fmaxf(mx, fmaxf(fabsf(y[e]), fabsf(y[e + 1])));
    u32 mb = __float_as_uint(mx);
    mb = wave_umax(mb);
    STAMP(23);
    u32 last = 0;
    if (lane == 0) {
        __hip_atomic_fetch_max(ctl + 0, mb, __ATOMIC_RELAXED, __HIP_MEMORY_SCOPE_WORKGROUP);
        if (!ok) __hip_atomic_store(ctl + 4, 1u, __ATOMIC_RELAXED, __HIP_MEMORY_SCOPE_WORKGROUP);
        last = __hip_atomic_fetch_add(ctl + 1, 1u, __ATOMIC_ACQ_REL, __HIP_MEMORY_SCOPE_WORKGROUP);
    }
    last = rfl(last);
    u32 gbits = 0; bool good;
    if (last == 3u) {
        const u32 wmax = rfl(__hip_atomic_load(ctl + 0, __ATOMIC_RELAXED, __HIP_MEMORY_SCOPE_WORKGROUP));
        const u32 fail = rfl(__hip_atomic_load(ctl + 4, __ATOMIC_RELAXED, __HIP_MEMORY_SCOPE_WORKGROUP));
        good = fail == 0u;
        if (good) {
            if (lane == 0) sys_store64(my + OFF_LG + (u32)wg * 8u, ((u64)call << 32) | wmax);
            const u64 t0 = __builtin_amdgcn_s_memrealtime();
            for (;;) {
                u64 x = 0;
                if (lane < 8) x = sys_load64(my + OFF_GG + (u32)(wg & 7) * 128u + (u32)lane * 8u);
                else if (lane == 8) x = (u64)agent_load(my + OFF_TMO);
                const bool tm = __any(lane == 8 && (u32)x == call);
                const bool all = __all(lane >= 8 || (u32)(x >> 32) == call);
                if (tm) { good = false; break; }
                if (all) { gbits = wave_umax(lane < 8 ? (u32)x : 0u); break; }
                if (__builtin_amdgcn_s_memrealtime() - t0 > DEADLINE) { agent_store(my + OFF_TMO, call); good = false; break; }
                __builtin_amdgcn_s_sleep(1);
            }
        }
        if (lane == 0) {
            __hip_atomic_store(ctl + 2, gbits, __ATOMIC_RELAXED, __HIP_MEMORY_SCOPE_WORKGROUP);
            __hip_atomic_store(ctl + 3, good ? 1u : 2u, __ATOMIC_RELEASE, __HIP_MEMORY_SCOPE_WORKGROUP);
        }
    } else {
        u32 st;
        while ((st = rfl(__hip_atomic_load(ctl + 3, __ATOMIC_ACQUIRE, __HIP_MEMORY_SCOPE_WORKGROUP))) == 0u) __builtin_amdgcn_s_sleep(1);
        good = st == 1u;
        gbits = rfl(__hip_atomic_load(ctl + 2, __ATOMIC_RELAXED, __HIP_MEMORY_SCOPE_WORKGROUP));
    }
    STAMP(24);
    const float amax = __uint_as_float(gbits);
    const float scale = amax / 448.0f;
    const float inv = 1.0f / scale;
    u32x4 o0, o1;
#pragma unroll
    for (int i = 0; i < 8; ++i) {
        u32 v = (u32)__builtin_amdgcn_cvt_pk_fp8_f32(y[4 * i] * inv, y[4 * i + 1] * inv, 0, false);
        v = (u32)__builtin_amdgcn_cvt_pk_fp8_f32(y[4 * i + 2] * inv, y[4 * i + 3] * inv, (int)v, true);
        if (!good) v = 0x7f7f7f7fu;
        if (i < 4) o0[i] = v; else o1[i - 4] = v;
    }
    const u32 qi = tau >> 5, qj = (tau >> 4) & 1u, n = tau & 15u;
    const u32 row = qi * 256u + wm * 64u + (u32)(be >> 1) * 32u + (u32)(lane & 31);
    const u32 col = qj * 2048u + n * 128u + (is_narrow(n) ? (u32)(be & 1) * 64u + (u32)(lane >> 5) * 32u : (u32)(lane >> 5) * 64u + (u32)(be & 1) * 32u);
    const rsrc_t out = mk_rsrc(P.out, 2u << 20);
    __builtin_amdgcn_raw_buffer_store_b128(o0, out, (int)(row * 4096u + col), 0, 0);
    __builtin_amdgcn_raw_buffer_store_b128(o1, out, (int)(row * 4096u + col + 16u), 0, 0);
    STAMP(25); STAMPC(31);
}

__device__ __forceinline__ void leader_wave(const Params& P, int lane, u32 call, char* my, int wave) {
    const u64 t0 = __builtin_amdgcn_s_memrealtime();
    u32 m = 0; bool good = true;
    for (u32 it = 0;; ++it) {
        bool all = true; m = 0;
#pragma unroll
        for (int i = 0; i < 4; ++i) { const u64 x = sys_load64(my + OFF_LG + (u32)(i * 64 + lane) * 8u); all = all && (u32)(x >> 32) == call; m = max(m, (u32)x); }
        if (__all(all)) break;
        if ((it & 15u) == 15u && (rfl(agent_load(my + OFF_TMO)) == call || __builtin_amdgcn_s_memrealtime() - t0 > DEADLINE)) {
            agent_store(my + OFF_TMO, call); good = false; break; }
        __builtin_amdgcn_s_sleep(1);
    }
    STAMP(26);
    if (good) {
        m = wave_umax(m);
        sys_store64(P.ws[lane >> 3] + OFF_GG + (u32)(lane & 7) * 128u + (u32)P.rank * 8u, ((u64)call << 32) | m);
    }
    if (lane == 0) agent_store(my + OFF_CALL, call);
}

__global__ void __launch_bounds__(512, 2) gemm_rs_fused(Params P) {
    __shared__ __attribute__((aligned(16))) unsigned char smem[LDS_BYTES];
    LAS unsigned char* lds = (LAS unsigned char*)smem;
    const int tid = threadIdx.x, lane = tid & 63, wave = (int)rfl((u32)tid >> 6);
    const int b = blockIdx.x, xcd = b & 7, s = b >> 3;
    char* my = P.my;
    asm volatile("global_load_dword v231, %0, off sc1" :: "v"(my + OFF_CALL) : "v231");
    if (tid < 64) ((LAS u32*)(lds + LDS_CTL))[tid] = 0u;
    __syncthreads();
    auto get_call = [&]() { u32 c; asm volatile("s_waitcnt vmcnt(0)\n\tv_readfirstlane_b32 %0, v231" : "=s"(c)); return c + 1u; };
    STAMP(0); STAMPC(30);
    if (s < 8 && (u32)wave < SEND_WPW) sender_wave(P, s, (u32)xcd * SEND_WPW + (u32)wave, lane, my, get_call);
    const u32 call = get_call();
    const u32 tag = call & TAGMASK;
    int tg = 0, ts = s - 8;
#pragma unroll
    for (int g = 0; g < 3; ++g) { const int w = ((NARROW_MASK >> g) & 1u) ? 8 : 4; if (tg == g && ts >= w) { ts -= w; tg = g + 1; } }
    const int tq = 4 * (xcd & 1) + (ts & 3), tn = 4 * tg + (xcd >> 1);
#define GRS_TILES do { \
        if (is_narrow((u32)tn)) gemm_wave<2>(P, tq, tn, (u32)ts >> 2, wave, lane, call, tag, lds, my, wave); \
        else gemm_wave<4>(P, tq, tn, 0u, wave, lane, call, tag, lds, my, wave); } while (0)
    if (s < 8) {
        STAMP(1);
        if (b == 0 && wave == 0) leader_wave(P, lane, call, my, wave);
    } else GRS_TILES;
    if (wave >= 4) reducer_wave(P, wave - 4, lane, call, tag, true, lds, my, wave);
}

}

extern "C" void kernel_launch(int rank, int nranks, ncclComm_t comm, hipStream_t stream,
                              void* const* d_in, const int* in_sizes, int n_in,
                              void* d_out, int out_size,
                              const int* in_dtypes, int out_dtype,
                              void* const* d_out_all,
                              void* const* d_ws_all, size_t ws_bytes) {
    (void)comm; (void)in_dtypes; (void)out_dtype; (void)d_out_all;
    if (nranks != grs::NR || n_in < 2 || ws_bytes < grs::WS_NEED || out_size != 512 * 4096 ||
        in_sizes[0] != 4096 * 512 || in_sizes[1] != 512 * 4096) {
        fprintf(stderr, "gemm_rs: unexpected problem geometry\n");
        return;
    }
    grs::Params p;
    p.x = d_in[0]; p.w = d_in[1]; p.out = d_out;
    for (int i = 0; i < grs::NR; ++i) p.ws[i] = (char*)d_ws_all[i];
    p.my = p.ws[rank];
    p.rank = rank; p.pad = 0;
    hipLaunchKernelGGL(grs::gemm_rs_fused, dim3(256), dim3(512), 0, stream, p);
    HIP_CHECK(hipGetLastError());
}
```

```cpp
#include <hip/hip_runtime.h>
#include <rccl/rccl.h>
#include <cstdio>
#include <cstdint>

#define HIP_CHECK(expr) do { hipError_t _e = (expr); if (_e != hipSuccess) { \
    fprintf(stderr, "HIP error %s at %s:%d\n", hipGetErrorString(_e), __FILE__, __LINE__); \
    return; } } while (0)

namespace grs {

typedef unsigned u32;
typedef unsigned long long u64;
typedef unsigned u32x4 __attribute__((ext_vector_type(4)));
typedef unsigned u32x3 __attribute__((ext_vector_type(3)));
typedef unsigned u32x2 __attribute__((ext_vector_type(2)));
typedef unsigned short u16x2 __attribute__((ext_vector_type(2)));
typedef float f32x16 __attribute__((ext_vector_type(16)));
typedef short bf16x8 __attribute__((ext_vector_type(8)));
typedef __amdgpu_buffer_rsrc_t rsrc_t;
#define LAS __attribute__((address_space(3)))
#define GAS __attribute__((address_space(1)))

constexpr int NR = 8;
constexpr int AUX_SYS = 17;
constexpr u32 RSRC_W3 = 0x00020000u;
constexpr u64 DEADLINE = 20000000ull;
constexpr u32 TAGSH = 14, TAGMASK = 0x3ffffu, RAWBIT = 1u << 13;

constexpr u32 OFF_CALL  = 0;
constexpr u32 OFF_TMO   = 128;
constexpr u32 OFF_DUMMY = 131072;
constexpr u32 OFF_XFLAG = 4096;
constexpr u32 OFF_WFLAG = 16384;
constexpr u32 OFF_PFLAG = 32768;
constexpr u32 OFF_SCTR  = 2048;
constexpr u32 OFF_LG    = 65536;
constexpr u32 OFF_GG    = 69632;
constexpr u32 OFF_DBG   = 56u << 20;
constexpr u32 BLK = 3072, ENT = 32, RAWB = 4096;
constexpr u32 OFF_XP = 1u << 20, OFF_WP = 7u << 20, OFF_PP = 13u << 20;
constexpr u32 OFF_XE = 19u << 20, OFF_WE = OFF_XE + 65536, OFF_PE = OFF_WE + 65536;
constexpr u32 OFF_XR = 20u << 20, OFF_WR = 28u << 20, OFF_PR = 36u << 20;
constexpr u32 WS_NEED = 44u << 20;

constexpr u32 SLOT = 16384, UNITB = 8192;
constexpr u32 LDS_CTL = 4 * SLOT;
constexpr u32 LDS_BYTES = 96 * 1024;

#define GRS_NARROW 9
#define GRS_D1 2
#define GRS_D2 3
#define GRS_D3 5
constexpr unsigned NARROW_MASK = GRS_NARROW, DEF1 = GRS_D1, DEF2 = GRS_D2, DEF3 = GRS_D3;
static_assert(DEF1 <= DEF2 && DEF2 <= DEF3 && DEF3 <= 8, "deferral must grow with the group");
#define GRS_SSLOTS 8
constexpr int SEND_SLOTS = GRS_SSLOTS;
static_assert(SEND_SLOTS == 4 || SEND_SLOTS == 8, "sender slots");
static_assert(NARROW_MASK < 16 && SEND_SLOTS + 16 + 4 * __builtin_popcount(NARROW_MASK) == 32, "32 slots per XCD");
__device__ __forceinline__ bool is_narrow(unsigned n) { return (NARROW_MASK >> (n >> 2)) & 1u; }

struct Params {
    const void* x; const void* w; void* out;
    char* ws[NR];
    char* my;
    int rank; u32 pad;
};

__device__ __forceinline__ u32 perm(u32 s0, u32 s1, u32 sel) { return __builtin_amdgcn_perm(s0, s1, sel); }
__device__ __forceinline__ u32 rotr4(u32 v) { return __builtin_amdgcn_alignbit(v, v, 4); }
__device__ __forceinline__ u32 rotl4(u32 v) { return __builtin_amdgcn_alignbit(v, v, 28); }
__device__ __forceinline__ u32 rfl(u32 v) { return (u32)__builtin_amdgcn_readfirstlane((int)v); }
__device__ __forceinline__ u32 rdl(u32 v, u32 l) { return (u32)__builtin_amdgcn_readlane((int)v, (int)l); }
__device__ __forceinline__ u32 pkmax(u32 a, u32 b) {
    return __builtin_bit_cast(u32, __builtin_elementwise_max(__builtin_bit_cast(u16x2, a), __builtin_bit_cast(u16x2, b)));
}
__device__ __forceinline__ u32 wave_umax(u32 v) {
    v = max(v, (u32)__builtin_amdgcn_update_dpp(0, (int)v, 0x111, 0xf, 0xf, true));
    v = max(v, (u32)__builtin_amdgcn_update_dpp(0, (int)v, 0x112, 0xf, 0xf, true));
    v = max(v, (u32)__builtin_amdgcn_update_dpp(0, (int)v, 0x114, 0xf, 0xf, true));
    v = max(v, (u32)__builtin_amdgcn_update_dpp(0, (int)v, 0x118, 0xf, 0xf, true));
    v = max(v, (u32)__builtin_amdgcn_update_dpp(0, (int)v, 0x142, 0xa, 0xf, false));
    v = max(v, (u32)__builtin_amdgcn_update_dpp(0, (int)v, 0x143, 0xc, 0xf, false));
    return rdl(v, 63);
}
__device__ __forceinline__ void drain() { asm volatile("s_waitcnt vmcnt(0)" ::: "memory"); }
__device__ __forceinline__ void cfence() { asm volatile("" ::: "memory"); }
__device__ __forceinline__ rsrc_t mk_rsrc(const void* p, u32 bytes) { return __builtin_amdgcn_make_buffer_rsrc((void*)p, 0, (int)bytes, RSRC_W3); }
#define EXP_LDAUX AUX_SYS
__device__ __forceinline__ u32x4 ld16(rsrc_t r, u32 voff, u32 soff) { return __builtin_amdgcn_raw_buffer_load_b128(r, (int)voff, (int)soff, EXP_LDAUX); }
__device__ __forceinline__ u32 ld4(rsrc_t r, u32 voff, u32 soff) { return __builtin_amdgcn_raw_buffer_load_b32(r, (int)voff, (int)soff, AUX_SYS); }
__device__ __forceinline__ void st16(rsrc_t r, u32 voff, u32 soff, u32x4 v) { __builtin_amdgcn_raw_buffer_store_b128(v, r, (int)(voff + soff), 0, AUX_SYS); }
__device__ __forceinline__ void st12(rsrc_t r, u32 voff, u32x3 v) { __builtin_amdgcn_raw_buffer_store_b96(v, r, (int)voff, 0, AUX_SYS); }
__device__ __forceinline__ void st4(rsrc_t r, u32 voff, u32 soff, u32 v) { __builtin_amdgcn_raw_buffer_store_b32(v, r, (int)voff, (int)soff, AUX_SYS); }
__device__ __forceinline__ u32 sys_load(const char* p) { return __hip_atomic_load((const GAS u32*)p, __ATOMIC_RELAXED, __HIP_MEMORY_SCOPE_SYSTEM); }
__device__ __forceinline__ u64 sys_load64(const char* p) { return __hip_atomic_load((const GAS u64*)p, __ATOMIC_RELAXED, __HIP_MEMORY_SCOPE_SYSTEM); }
__device__ __forceinline__ void sys_store(char* p, u32 v) { __hip_atomic_store((GAS u32*)p, v, __ATOMIC_RELAXED, __HIP_MEMORY_SCOPE_SYSTEM); }
__device__ __forceinline__ void sys_store64(char* p, u64 v) { __hip_atomic_store((GAS u64*)p, v, __ATOMIC_RELAXED, __HIP_MEMORY_SCOPE_SYSTEM); }
__device__ __forceinline__ u32 agent_load(const char* p) { return __hip_atomic_load((const GAS u32*)p, __ATOMIC_RELAXED, __HIP_MEMORY_SCOPE_AGENT); }
__device__ __forceinline__ void agent_store(char* p, u32 v) { __hip_atomic_store((GAS u32*)p, v, __ATOMIC_RELAXED, __HIP_MEMORY_SCOPE_AGENT); }

#define STAMPC(idx) do { } while (0)
#define STAMP2(idx, val) do { } while (0)
#define STAMP(idx) do { } while (0)

__device__ __forceinline__ u32 poll_sys(const char* p) { u32 x; asm volatile("global_load_dword %0, %1, off sc0 sc1\n\ts_waitcnt vmcnt(0)" : "=&v"(x) : "v"(p) : "memory"); return x; }
__device__ __forceinline__ u32 poll_agent(const char* p) { u32 x; asm volatile("global_load_dword %0, %1, off sc1\n\ts_waitcnt vmcnt(0)" : "=&v"(x) : "v"(p) : "memory"); return x; }
__device__ __forceinline__ void ld16x2_now(const char* p0, const char* p1, u32x4& a, u32x4& b) {
    asm volatile("global_load_dwordx4 %0, %2, off sc0 sc1\n\tglobal_load_dwordx4 %1, %3, off sc0 sc1\n\ts_waitcnt vmcnt(0)" : "=&v"(a), "=&v"(b) : "v"(p0), "v"(p1) : "memory");
}
__device__ __forceinline__ void give_up(char* my, u32 call) {
    asm volatile("global_store_dword %0, %1, off sc1" :: "v"(my + OFF_TMO), "v"(call) : "memory");
}
__device__ __forceinline__ bool wait_tags(const char* addr, u32 tag, u32& v, char* my, u32 call) {
    const u64 t0 = __builtin_amdgcn_s_memrealtime();
    for (u32 it = 0;; ++it) {
        const u32 x = poll_sys(addr);
        if (__all((x >> TAGSH) == tag)) { v = x; return true; }
        if ((it & 15u) == 15u) {
            if (rfl(poll_agent(my + OFF_TMO)) == call || __builtin_amdgcn_s_memrealtime() - t0 > DEADLINE) {
                give_up(my, call); v = 0; return false; }
        }
        __builtin_amdgcn_s_sleep(2);
    }
}

__device__ __forceinline__ u32 lane_emax(const u32 (&H)[8]) {
    u32 m1 = 0, m2 = 0;
#pragma unroll
    for (int t = 0; t < 8; ++t) { const u32 g = H[t] | 0x80808080u; m1 = pkmax(m1, g); m2 = pkmax(m2, g << 8); }
    const u32 m = pkmax(m1, m2);
    return max((m >> 24) & 0x7fu, (m >> 8) & 0x7fu);
}
template <bool IS_W> __device__ __forceinline__ void chunks_from(const u32 (&L)[8], const u32 (&H)[8], u32x4 (&o)[4]) {
#pragma unroll
    for (int t = 0; t < 8; ++t) {
        const u32 e = perm(H[t], L[t], 0x05010400u), d = perm(H[t], L[t], 0x07030602u);
        if (IS_W) { o[2 * (t >> 2)][t & 3] = e; o[2 * (t >> 2) + 1][t & 3] = d; }
        else      { o[t >> 1][2 * (t & 1)] = e; o[t >> 1][2 * (t & 1) + 1] = d; }
    }
}
__device__ __forceinline__ u64 pack_prep(const u32 (&H)[8], u32 lb, u32x4& np, bool quad = false) {
    const u32 lb4 = lb * 0x01010101u;
    u32 T[8], acc = 0xffffffffu;
#pragma unroll
    for (int t = 0; t < 8; ++t) { T[t] = (H[t] | 0x80808080u) - lb4; acc &= T[t]; }
    u32 n[4];
#pragma unroll
    for (int j = 0; j < 4; ++j) {
        const u32 a = (H[2 * j] & 0x80808080u) | (T[2 * j] & 0x07070707u);
        const u32 b = (H[2 * j + 1] & 0x80808080u) | (T[2 * j + 1] & 0x07070707u);
        n[j] = a | rotl4(b);
    }
    const auto s0 = __builtin_amdgcn_permlane32_swap(n[0], n[2], false, false);
    const auto s1 = __builtin_amdgcn_permlane32_swap(n[1], n[3], false, false);
    np = quad ? (u32x4){n[0], n[1], n[2], n[3]} : (u32x4){s0[0], s1[0], s0[1], s1[1]};
    return __ballot((acc & 0x80808080u) != 0x80808080u);
}
__device__ __forceinline__ u32 pack_emit(const u32 (&L)[8], const u32 (&H)[8], u32 lb, u32x4 np, u64 bm, rsrc_t dst, u32 blk_off, u32 ent_off, u32 raw_off, int lane, bool is_w, bool quad = false) {
    const u32 vo = (u32)lane * 16u;
    if (quad) {
#pragma unroll
        for (int j = 0; j < 4; ++j) st12(dst, (u32)lane * 12u + blk_off + 768u * j, (u32x3){L[2 * j], L[2 * j + 1], np[j]});
    } else {
        st16(dst, vo, blk_off, (u32x4){L[0], L[1], L[2], L[3]});
        st16(dst, vo, blk_off + 1024u, (u32x4){L[4], L[5], L[6], L[7]});
        st16(dst, vo, blk_off + 2048u, np);
    }
    u32 hdr = lb;

    if (bm != 0ull) {
        const u32 lb4 = lb * 0x01010101u;
        u32 T[8];
#pragma unroll
        for (int t = 0; t < 8; ++t) T[t] = (H[t] | 0x80808080u) - lb4;
        u32 bits = 0;
#pragma unroll
        for (int t = 0; t < 8; ++t) { const u32 x = (~T[t] & 0x80808080u) >> 7;
            bits |= ((x & 1u) | ((x >> 7) & 2u) | ((x >> 14) & 4u) | ((x >> 21) & 8u)) << (4 * t); }
        const u32 nb0 = __popc(bits & 0xffffu), nb1 = __popc(bits >> 16);
        u32 n0 = 0, n1 = 0, p0 = 0, p1 = 0;
        while (bm) { const u32 l = (u32)__builtin_ctzll(bm); bm &= bm - 1; if ((u32)lane == l) { p0 = n0; p1 = n1; } n0 += rdl(nb0, l); n1 += rdl(nb1, l); }
        if (n0 + n1 <= 7u) {
            hdr |= (n0 << 7) | (n1 << 10);
            p1 += n0;
            while (bits) {
                const u32 idx = (u32)__builtin_ctz(bits); bits &= bits - 1;
                const u32 t = idx >> 2, by = idx & 3u;
                u32 hv = H[0];
#pragma unroll
                for (int tt = 1; tt < 8; ++tt) hv = (t == (u32)tt) ? H[tt] : hv;
                const u32 slot = t < 4u ? p0++ : p1++;
                st4(dst, slot * 4u, ent_off, ((u32)lane << 16) | (t << 10) | (by << 8) | ((hv >> (8 * by)) & 0xffu));
            }
        } else {
            hdr |= RAWBIT;
            u32x4 o[4]; if (is_w) chunks_from<true>(L, H, o); else chunks_from<false>(L, H, o);
#pragma unroll
            for (int c = 0; c < 4; ++c) st16(dst, vo, raw_off + 1024u * c, o[c]);
        }
    }
    return hdr;
}
__device__ __forceinline__ u32 pack_store(const u32 (&L)[8], const u32 (&H)[8], u32 lb, rsrc_t dst, u32 blk_off, u32 ent_off, u32 raw_off, int lane, bool is_w) {
    u32x4 np; const u64 bm = pack_prep(H, lb, np);
    return pack_emit(L, H, lb, np, bm, dst, blk_off, ent_off, raw_off, lane, is_w);
}
struct Half { u32x4 l; u32x2 n; };
__device__ __forceinline__ void high_bytes(u32x2 n, u32 hb, u32 hdr, u32 entv, u32 ebase, u32 (&H)[4], int lane) {
    const u32 lb4 = (hdr & 0x7fu) * 0x01010101u;
    H[0] = (n[0] & 0x87878787u) + lb4; H[1] = (rotr4(n[0]) & 0x87878787u) + lb4;
    H[2] = (n[1] & 0x87878787u) + lb4; H[3] = (rotr4(n[1]) & 0x87878787u) + lb4;
    const u32 n0 = (hdr >> 7) & 7u, first = hb ? n0 : 0u, cnt = hb ? (hdr >> 10) & 7u : n0;
    for (u32 i = 0; i < cnt; ++i) {
        const u32 en = rdl(entv, ebase + first + i);
        const u32 sh = ((en >> 8) & 3u) * 8u, t = (en >> 10) & 7u;
        const u32 m = ((u32)lane == (en >> 16)) ? (0xffu << sh) : 0u, v = (en & 0xffu) << sh;
#pragma unroll
        for (int j = 0; j < 4; ++j) { const u32 mm = ((t & 3u) == (u32)j) ? m : 0u; H[j] = (H[j] & ~mm) | (v & mm); }
    }
}

#define GRS_SW (32 / SEND_SLOTS)
constexpr u32 NBLK = 512, SEND_WPW = GRS_SW, SEND_WAVES = SEND_SLOTS * SEND_WPW;
#define RAWCL "v200", "v201", "v202", "v203", "v204", "v205", "v206", "v207", "v208", "v209", "v210", "v211", "v212", "v213", "v214", "v215"
#define RAW_LD(OP, REG, IMM, voff, rs, so) asm volatile("buffer_load_" OP " " REG ", %0, %1, %2 offen offset:" #IMM :: "v"(voff), "s"(rs), "s"(so) : RAWCL)
#define RAW_P1(i, A, B) "v_perm_b32 %" #i ", " B ", " A ", %16\n\t"
#define RAW_P2(i, A, B) "v_perm_b32 %" #i ", " B ", " A ", %17\n\t"
#define RAW_PICK(N, A0, B0, A1, B1, A2, B2, A3, B3, A4, B4, A5, B5, A6, B6, A7, B7) \
    asm volatile("s_waitcnt vmcnt(" #N ")\n\t" \
        RAW_P1(0, A0, B0) RAW_P1(1, A1, B1) RAW_P1(2, A2, B2) RAW_P1(3, A3, B3) RAW_P1(4, A4, B4) RAW_P1(5, A5, B5) RAW_P1(6, A6, B6) RAW_P1(7, A7, B7) \
        RAW_P2(8, A0, B0) RAW_P2(9, A1, B1) RAW_P2(10, A2, B2) RAW_P2(11, A3, B3) RAW_P2(12, A4, B4) RAW_P2(13, A5, B5) RAW_P2(14, A6, B6) RAW_P2(15, A7, B7) \
        : "=v"(L[0]), "=v"(L[1]), "=v"(L[2]), "=v"(L[3]), "=v"(L[4]), "=v"(L[5]), "=v"(L[6]), "=v"(L[7]), \
          "=v"(H[0]), "=v"(H[1]), "=v"(H[2]), "=v"(H[3]), "=v"(H[4]), "=v"(H[5]), "=v"(H[6]), "=v"(H[7]) : "s"(selL), "s"(selH) : RAWCL)
__device__ __forceinline__ u32x4 rsrc_words(const void* p, u32 bytes) { return (u32x4){rfl((u32)(size_t)p), rfl((u32)((size_t)p >> 32)) & 0xffffu, bytes, RSRC_W3}; }

template <class GetCall>
__device__ __forceinline__ void sender_wave(const Params& P, int d, u32 wid, int lane, char* my, GetCall get_call) {
    const u32 di = (u32)d & 1u, dj = ((u32)d >> 1) & 1u, dk = (u32)d >> 2, me = (u32)P.rank;
    const u32 r = (u32)lane & 31u, h = (u32)lane >> 5;
    const rsrc_t dst = mk_rsrc(P.ws[d], WS_NEED);
    const u32x4 srcx = rsrc_words(P.x, 4u << 20), srcw = rsrc_words(P.w, 4u << 20);
    const u32 vx = (di * 256u + r) * 1024u + 32u * h + dk * 512u;
    const u32 cg = 16u * ((r >> 2) & 1u) + (r & 3u) + 4u * (r >> 3);
    const u32 vw0 = (16u * h + dk * 256u) * 8192u + dj * 4096u;
    const u32 vw = vw0 + cg * 8u, vwa = vw0 + cg * 4u;
    GAS u32* ctr = (GAS u32*)(my + OFF_SCTR + (u32)d * 128u);
    auto draw = [&]() { const u32 one = 1u;
        asm volatile("s_mov_b64 exec, 1\n\tglobal_atomic_add v216, %0, %1, off sc0\n\ts_mov_b64 exec, -1" :: "v"(ctr), "v"(one) : "v216", "memory"); };
    auto unit_of = [&](u32 idx, u32& rr) {
        u32 u = 0;
#pragma unroll
        for (u32 v = 0; v < 8u; ++v) {
            const u32 sz = 40u + 8u * ((v < 8u - DEF1 ? 1u : 0u) + (v < 8u - DEF2 ? 1u : 0u) + (v < 8u - DEF3 ? 1u : 0u));
            if (u == v && idx >= sz) { idx -= sz; u = v + 1u; }
        }
        if (u < 8u) { rr = idx; return u; }
        if (idx < 8u * DEF1) { rr = 40u + (idx & 7u); return 8u - DEF1 + (idx >> 3); }
        idx -= 8u * DEF1;
        if (idx < 8u * DEF2) { rr = 48u + (idx & 7u); return 8u - DEF2 + (idx >> 3); }
        idx -= 8u * DEF2;
        rr = 56u + (idx & 7u); return 8u - DEF3 + (idx >> 3);
    };
    auto load_raw = [&](u32 u, u32 rr) {
        if (rr < 32u) {
            const u32 so = ((rr >> 2) * 512u + (rr & 3u) * 64u) * 1024u + u * 64u;
            RAW_LD("dwordx4", "v[200:203]", 0, vx, srcx, so); RAW_LD("dwordx4", "v[204:207]", 16, vx, srcx, so);
            RAW_LD("dwordx4", "v[208:211]", 0, vx, srcx, so + 32768u); RAW_LD("dwordx4", "v[212:215]", 16, vx, srcx, so + 32768u);
        } else {
            const u32 so = (u * 32u + (rr & 1u) * 8u) * 8192u + ((rr - 32u) >> 1) * 256u;
            if (is_narrow((rr - 32u) >> 1)) {
                RAW_LD("dword", "v200", 0, vwa, srcw, so);               RAW_LD("dword", "v208", 128, vwa, srcw, so);
                RAW_LD("dword", "v201", 0, vwa, srcw, so + 8192u);       RAW_LD("dword", "v209", 128, vwa, srcw, so + 8192u);
                RAW_LD("dword", "v202", 0, vwa, srcw, so + 2u * 8192u);  RAW_LD("dword", "v210", 128, vwa, srcw, so + 2u * 8192u);
                RAW_LD("dword", "v203", 0, vwa, srcw, so + 3u * 8192u);  RAW_LD("dword", "v211", 128, vwa, srcw, so + 3u * 8192u);
                RAW_LD("dword", "v204", 0, vwa, srcw, so + 4u * 8192u);  RAW_LD("dword", "v212", 128, vwa, srcw, so + 4u * 8192u);
                RAW_LD("dword", "v205", 0, vwa, srcw, so + 5u * 8192u);  RAW_LD("dword", "v213", 128, vwa, srcw, so + 5u * 8192u);
                RAW_LD("dword", "v206", 0, vwa, srcw, so + 6u * 8192u);  RAW_LD("dword", "v214", 128, vwa, srcw, so + 6u * 8192u);
                RAW_LD("dword", "v207", 0, vwa, srcw, so + 7u * 8192u);  RAW_LD("dword", "v215", 128, vwa, srcw, so + 7u * 8192u);
            } else {
                RAW_LD("dwordx2", "v[200:201]", 0, vw, srcw, so);               RAW_LD("dwordx2", "v[202:203]", 0, vw, srcw, so + 8192u);
                RAW_LD("dwordx2", "v[204:205]", 0, vw, srcw, so + 2u * 8192u);  RAW_LD("dwordx2", "v[206:207]", 0, vw, srcw, so + 3u * 8192u);
                RAW_LD("dwordx2", "v[208:209]", 0, vw, srcw, so + 4u * 8192u);  RAW_LD("dwordx2", "v[210:211]", 0, vw, srcw, so + 5u * 8192u);
                RAW_LD("dwordx2", "v[212:213]", 0, vw, srcw, so + 6u * 8192u);  RAW_LD("dwordx2", "v[214:215]", 0, vw, srcw, so + 7u * 8192u);
            }
        }
    };
    u32 L[8], H[8], lb; u32x4 np; u64 bm;
    auto quad_blk = [&](u32 rr) { return rr >= 32u && is_narrow((rr - 32u) >> 1); };
    auto pick = [&](u32 rr, int younger) {
        if (younger == 0) asm volatile("s_waitcnt vmcnt(0)" ::: "memory");
        else if (younger == 4) asm volatile("s_waitcnt vmcnt(4)" ::: "memory");
        else asm volatile("s_waitcnt vmcnt(5)" ::: "memory");
        if (rr < 32u) {
            const u32 selL = 0x06040200u, selH = 0x07050301u;
            RAW_PICK(63, "v200", "v201", "v202", "v203", "v204", "v205", "v206", "v207", "v208", "v209", "v210", "v211", "v212", "v213", "v214", "v215");
        } else {
            const u32 selL = 0x06020400u, selH = 0x07030501u;
            if (is_narrow((rr - 32u) >> 1)) RAW_PICK(63, "v200", "v201", "v202", "v203", "v204", "v205", "v206", "v207", "v208", "v209", "v210", "v211", "v212", "v213", "v214", "v215");
            else RAW_PICK(63, "v200", "v202", "v204", "v206", "v208", "v210", "v212", "v214", "v201", "v203", "v205", "v207", "v209", "v211", "v213", "v215");
        }
        const u32 em = wave_umax(lane_emax(H));
        lb = em > 7u ? em - 7u : 0u;
        bm = pack_prep(H, lb, np, quad_blk(rr));
    };
    auto ticket = [&]() { u32 t; asm volatile("v_readfirstlane_b32 %0, v216" : "=s"(t)); return t; };
    u32 rr, u = unit_of(wid, rr);
    load_raw(u, rr);
    draw();
    const u32 call = get_call(), tag = call & TAGMASK;
    const u32 base = (call - 1u) * NBLK - SEND_WAVES;
    pick(rr, 0);
    char* prev_flag = my + OFF_DUMMY + ((u32)d * SEND_WAVES + wid) * 128u; u32 prev_word = 0;
#define SSTAMP() do { } while (0)
    while (true) {
        const u32 nxt = ticket() - base;
        const bool more = nxt < NBLK;
        u32 nrr = 0, nu = 0;
        if (more) nu = unit_of(nxt, nrr);
        const bool is_w = rr >= 32u;
        u32 blk, offP, offE, offR, flag;
        if (!is_w) { blk = ((me * 8 + u) * 8 + (rr >> 2)) * 4 + (rr & 3u); offP = OFF_XP; offE = OFF_XE; offR = OFF_XR;
                     flag = OFF_XFLAG + ((rr >> 2) * 8 + u) * 128u + ((rr & 3u) * 8 + me) * 4u; }
        else       { const u32 n = (rr - 32u) >> 1, kf = rr & 1u; blk = ((me * 8 + u) * 16 + n) * 2 + kf; offP = OFF_WP; offE = OFF_WE; offR = OFF_WR;
                     flag = OFF_WFLAG + (n * 8 + u) * 128u + (kf * 8 + me) * 4u; }
        drain();
        if (more) { load_raw(nu, nrr); draw(); }
        if (lane == 0) sys_store(prev_flag, prev_word);
        SSTAMP();
        const bool quad = quad_blk(rr);
        const u32 hdr = pack_emit(L, H, lb, np, bm, dst, offP + blk * BLK, offE + blk * ENT, offR + blk * RAWB, lane, is_w, quad);
        prev_flag = P.ws[d] + flag; prev_word = (tag << TAGSH) | hdr;
        if (!more) break;
        u = nu; rr = nrr;
        pick(rr, quad ? 5 : 4);
    }
    drain();
    if (lane == 0) sys_store(prev_flag, prev_word);
    SSTAMP();
}

__device__ __forceinline__ u32 cvt_pk_bf16(float lo, float hi) { u32 r; asm("v_cvt_pk_bf16_f32 %0, %1, %2" : "=v"(r) : "v"(lo), "v"(hi)); return r; }
__device__ __forceinline__ u32x2 ld8(rsrc_t r, u32 voff, u32 soff) { return __builtin_amdgcn_raw_buffer_load_b64(r, (int)voff, (int)soff, EXP_LDAUX); }
template <int NC> struct StepRegs;
template <> struct StepRegs<4> { Half xa, xb; u32x4 wl; u32x2 wn; };
template <> struct StepRegs<2> { Half xa, xb; u32x3 w; };

__device__ __forceinline__ void ald16(u32x4& d, u32 voff, u32x4 rs, u32 soff) { asm volatile("buffer_load_dwordx4 %0, %1, %2, %3 offen sc0 sc1" : "+v"(d) : "v"(voff), "s"(rs), "s"(soff)); }
__device__ __forceinline__ void ald12(u32x3& d, u32 voff, u32x4 rs, u32 soff) { asm volatile("buffer_load_dwordx3 %0, %1, %2, %3 offen sc0 sc1" : "+v"(d) : "v"(voff), "s"(rs), "s"(soff)); }
__device__ __forceinline__ void ald8(u32x2& d, u32 voff, u32x4 rs, u32 soff) { asm volatile("buffer_load_dwordx2 %0, %1, %2, %3 offen sc0 sc1" : "+v"(d) : "v"(voff), "s"(rs), "s"(soff)); }
__device__ __forceinline__ void ald4(u32& d, u32 voff, u32x4 rs, u32 soff) { asm volatile("buffer_load_dword %0, %1, %2, %3 offen sc0 sc1" : "+v"(d) : "v"(voff), "s"(rs), "s"(soff)); }
#define LOOK_SET(R, val) asm volatile("v_mov_b32 " R ", %0" :: "s"(val) : R)
#define LOOK_LD(R, voff, rs, soff) asm volatile("buffer_load_dword " R ", %0, %1, %2 offen sc0 sc1" :: "v"(voff), "s"(rs), "s"(soff) : R)
#define LOOK_GET(R, N, dstv) asm volatile("s_waitcnt vmcnt(" #N ")\n\tv_mov_b32 %0, " R : "=v"(dstv))
template <int N> __device__ __forceinline__ void look_get(u32& d, int which) {
    if (which == 0) asm volatile("s_waitcnt vmcnt(%1)\n\tv_mov_b32 %0, v228" : "=v"(d) : "n"(N));
    else if (which == 1) asm volatile("s_waitcnt vmcnt(%1)\n\tv_mov_b32 %0, v229" : "=v"(d) : "n"(N));
    else asm volatile("s_waitcnt vmcnt(%1)\n\tv_mov_b32 %0, v230" : "=v"(d) : "n"(N));
}
template <int N> __device__ __forceinline__ void vm_wait(StepRegs<4>& r, u32& ev) {
    asm volatile("s_waitcnt vmcnt(%7)" : "+v"(r.wl), "+v"(r.wn), "+v"(r.xa.l), "+v"(r.xa.n), "+v"(r.xb.l), "+v"(r.xb.n), "+v"(ev) : "n"(N)); }
template <int N> __device__ __forceinline__ void vm_wait(StepRegs<2>& r, u32& ev) {
    asm volatile("s_waitcnt vmcnt(%6)" : "+v"(r.w), "+v"(r.xa.l), "+v"(r.xa.n), "+v"(r.xb.l), "+v"(r.xb.n), "+v"(ev) : "n"(N)); }
__device__ __forceinline__ void reg_hold(StepRegs<4>& r) { asm volatile("" : "+v"(r.wl), "+v"(r.wn), "+v"(r.xa.l), "+v"(r.xa.n), "+v"(r.xb.l), "+v"(r.xb.n)); }
__device__ __forceinline__ void reg_hold(StepRegs<2>& r) { asm volatile("" : "+v"(r.w), "+v"(r.xa.l), "+v"(r.xa.n), "+v"(r.xb.l), "+v"(r.xb.n)); }
constexpr u32 LDS_CNT = LDS_CTL + 64;
template <int OFF> __device__ __forceinline__ void sig_step(u32 cnt_addr, u32 add) {
    asm volatile("s_mov_b64 exec, 1\n\tds_add_u32 %0, %1 offset:%2\n\ts_mov_b64 exec, -1" :: "v"(cnt_addr), "v"(add), "n"(OFF) : "memory");
}
__device__ __forceinline__ void wait_step(LAS unsigned char* lds, u32 J, u32 seen) {
    if (rfl(seen) == 8u) return;
    while (rfl(__hip_atomic_load((LAS u32*)(lds + LDS_CNT) + J, __ATOMIC_RELAXED, __HIP_MEMORY_SCOPE_WORKGROUP)) != 8u) __builtin_amdgcn_s_sleep(1);
    cfence();
}

#define GRS_PK 0x3333
template <int NC>
__device__ __forceinline__ void gemm_wave(const Params& P, int q, int n, u32 ht, int w, int lane, u32 call, u32 tag, LAS unsigned char* lds, char* my, int wave) {
    const u32 PUSH_K = (GRS_PK >> (4 * (n >> 2))) & 15u;
    constexpr u32 SLOTB = NC * 4096u, SUBB = NC * 1024u;
    constexpr int NL = NC == 4 ? 6 : 5;
    const u32 me = (u32)P.rank;
    const u32 vo = (u32)lane * 16u, von = (u32)(lane & 31) * 16u + (u32)(lane >> 5) * 8u;
    const u32 b = (u32)w >> 1, hb = (u32)w & 1u;
    const u32 wu = (u32)w >> 2, kf = ((u32)w >> 1) & 1u;
    const u32 wh = NC == 4 ? hb : ht;
    bool ok = true;
    const u32 fo = (lane & 8) == 0 ? OFF_XFLAG + (u32)(q * 8) * 128u + (b * 8 + ((me + (u32)(lane & 7)) & 7u)) * 4u
                                   : OFF_WFLAG + (u32)(n * 8) * 128u + (kf * 8 + ((me + 2u * (u32)(lane & 3) + wu) & 7u)) * 4u;
    StepRegs<NC> R[4];
#pragma unroll
    for (int s = 0; s < 4; ++s) { R[s].xa.l = (u32x4){0u, 0u, 0u, 0u}; R[s].xb.l = R[s].xa.l; R[s].xa.n = (u32x2){0u, 0u}; R[s].xb.n = R[s].xa.n;
        if constexpr (NC == 4) { R[s].wl = R[s].xa.l; R[s].wn = R[s].xa.n; } else R[s].w = (u32x3){0u, 0u, 0u}; }
    constexpr u32 USTR = 32u * BLK;
    const u32 vl = vo + hb * 1024u, vn = von + 2048u + hb * 512u;
    const u32 vw = (u32)lane * 12u + (2u * ht + hb) * 768u;
    u32 oA[4], oB[4], oW[4];
#pragma unroll
    for (int s = 0; s < 4; ++s) {
        const u32 pa = (me + 2 * s) & 7u, pb = (me + 2 * s + 1) & 7u, pw = wu ? pb : pa;
        oA[s] = OFF_XP + (((pa * 8) * 8 + q) * 4 + b) * BLK; oB[s] = OFF_XP + (((pb * 8) * 8 + q) * 4 + b) * BLK;
        oW[s] = OFF_WP + (((pw * 8) * 16 + n) * 2 + kf) * BLK;
    }
    u32 ve;
    { const u32 r = min((u32)lane / 5u, 11u), e = (u32)lane - 5u * ((u32)lane / 5u);
      const u32 px = (me + r) & 7u, pw = (me + 2u * (r & 3u) + wu) & 7u;
      ve = (r < 8u ? OFF_XE + (((px * 8) * 8 + q) * 4 + b) * ENT : OFF_WE + (((pw * 8) * 16 + n) * 2 + kf) * ENT) + e * 4u; }
    const u32 rs0 = rfl((u32)(size_t)my), rs1 = rfl((u32)((size_t)my >> 32)) & 0xffffu;
    const u32x4 rsw = {rs0, rs1, WS_NEED, RSRC_W3};
    auto unit_rsrc = [&](u32 u, bool live) {
        const u64 a = ((u64)rs0 | ((u64)rs1 << 32)) + (u64)(u * USTR);
        return (u32x4){(u32)a, (u32)(a >> 32), live ? WS_NEED : 0u, RSRC_W3}; };
    auto ld_w = [&](StepRegs<NC>& r, u32x4 rs, u32 so, int part) {
        if constexpr (NC == 4) { if (part == 0) ald8(r.wn, vn, rs, so); else ald16(r.wl, vl, rs, so); } else { if (part == 0) ald12(r.w, vw, rs, so); } };
    u32 evc = 0, evn = 0;
    auto issue = [&](int s, u32x4 rs, u32 u) {
        if (s == 0) ald4(evn, ve, rsw, u * 1024u);
        ld_w(R[s], rs, oW[s], 0); ld_w(R[s], rs, oW[s], 1);
        ald8(R[s].xa.n, vn, rs, oA[s]); ald16(R[s].xa.l, vl, rs, oA[s]);
        ald8(R[s].xb.n, vn, rs, oB[s]); ald16(R[s].xb.l, vl, rs, oB[s]);
    };
    LAS unsigned char* wdst = lds + (wu * 2u + kf) * SUBB + (NC == 4 ? hb * 2048u + (u32)lane * 16u : (u32)lane * 16u + hb * 8u);
    const u32 cntsh = hb ? 10u : 7u, wcntsh = wh ? 10u : 7u;
    const u32 fixm = (7u << ((lane & 8) == 0 ? cntsh : wcntsh)) | RAWBIT;
    auto entry = [&](u32 ev, u32 r, u32 i, u32 hdr, u32 recoff) {
        if (((hdr >> 7) & 7u) + ((hdr >> 10) & 7u) <= 5u) return rdl(ev, 5u * r + i);
        return rfl(poll_sys(my + recoff + i * 4u)); };
    auto fix_x = [&](u32 hdr, u32 ev, u32 r, u32 u, u32x4& a, u32x4& c) {
        if ((hdr & ((7u << cntsh) | RAWBIT)) == 0u) return;
        const u32 bx = ((((me + r) & 7u) * 8 + u) * 8 + q) * 4 + b;
        if (hdr & RAWBIT) {
            const char* rp = my + OFF_XR + bx * RAWB + hb * 2048u + vo;
            ld16x2_now(rp, rp + 1024, a, c);
        }
        const u32 cnt = (hdr >> cntsh) & 7u, first = hb ? (hdr >> 7) & 7u : 0u;
        for (u32 i = 0; i < cnt; ++i) {
            const u32 en = entry(ev, r, first + i, hdr, OFF_XE + bx * ENT), t = (en >> 10) & 7u, by = (en >> 8) & 3u;
            const u32 sh = (by & 1u) * 16u + 8u, D = 2u * (t & 3u) + (by >> 1);
            const u32 m = ((u32)lane == (en >> 16)) ? (0xffu << sh) : 0u, v = (en & 0xffu) << sh;
#pragma unroll
            for (int j = 0; j < 4; ++j) {
                const u32 ma = (D == (u32)j) ? m : 0u, mc = (D == (u32)(4 + j)) ? m : 0u;
                a[j] = (a[j] & ~ma) | (v & ma); c[j] = (c[j] & ~mc) | (v & mc); }
        }
    };
    auto fix_w = [&](u32 ev, u32 u, int s, u32 hw) {
        LAS unsigned char* p = lds + s * SLOTB + (wu * 2u + kf) * SUBB + (NC == 4 ? hb * 2048u : 0u);
        const u32 bw = ((((me + 2 * s + wu) & 7u) * 8 + u) * 16 + n) * 2 + kf;
        if (hw & RAWBIT) {
            const char* rp = my + OFF_WR + bw * RAWB + wh * 2048u + vo;
            u32x4 v0, v1; ld16x2_now(rp, rp + 1024, v0, v1);
            if (NC == 4) { *(LAS u32x4*)(p + lane * 16) = v0; *(LAS u32x4*)(p + 1024 + lane * 16) = v1; }
            else { *(LAS u32x2*)(p + lane * 16 + hb * 8u) = hb ? (u32x2){v0[2], v0[3]} : (u32x2){v0[0], v0[1]};
                   *(LAS u32x2*)(p + 1024 + lane * 16 + hb * 8u) = hb ? (u32x2){v1[2], v1[3]} : (u32x2){v1[0], v1[1]}; }
        }
        const u32 cnt = (hw >> wcntsh) & 7u, first = wh ? (hw >> 7) & 7u : 0u;
        for (u32 i = 0; i < cnt; ++i) {
            const u32 en = entry(ev, 8u + (u32)s, first + i, hw, OFF_WE + bw * ENT), t = (en >> 10) & 7u, by = (en >> 8) & 3u;
            if (NC == 2 && ((t >> 1) & 1u) != hb) continue;
            if (lane == 0) p[(en >> 16) * 16u + (by >> 1) * 1024u + (t & 3u) * 4u + (by & 1u) * 2u + 1u] = (unsigned char)(en & 0xffu);
        }
        asm volatile("s_waitcnt lgkmcnt(0)" ::: "memory");
    };
    f32x16 acc[NC];
#pragma unroll
    for (int c = 0; c < NC; ++c)
#pragma unroll
        for (int g = 0; g < 16; ++g) acc[c][g] = 0.f;
    u32x4 X[2][4]; bf16x8 wf[2][NC];
#pragma unroll
    for (int c = 0; c < NC; ++c) wf[1][c] = (bf16x8){0, 0, 0, 0, 0, 0, 0, 0};
    X[1][3] = (u32x4){0u, 0u, 0u, 0u};
    LAS unsigned char* wb = lds + lane * 16;
    const u32 cnt0 = (u32)(size_t)(lds + LDS_CNT);
#define MF(c, F, XF) acc[c] = __builtin_amdgcn_mfma_f32_32x32x16_bf16(F[c], __builtin_bit_cast(bf16x8, XF), acc[c], 0, 0, 0)
#define SB __builtin_amdgcn_sched_barrier(0)
#define RD1(s, sub, c) wf[(sub) & 1][c] = *(const LAS bf16x8*)(wb + (s) * SLOTB + (sub) * SUBB + (c) * 1024)
#define HI01(H, n) do { H[0] = (n[0] & 0x87878787u) + lb4_; H[1] = (rotr4(n[0]) & 0x87878787u) + lb4_; } while (0)
#define HI23(H, n) do { H[2] = (n[1] & 0x87878787u) + lb4_; H[3] = (rotr4(n[1]) & 0x87878787u) + lb4_; } while (0)
#define PE(H, l, i) perm(H[i], l[i], 0x05010400u)
#define PO(H, l, i) perm(H[i], l[i], 0x07030602u)
#define XPAIR(H, l, i) (u32x4){PE(H, l, i), PO(H, l, i), PE(H, l, (i) + 1), PO(H, l, (i) + 1)}
#define LBREP(h) perm(0u, (h) & 0x7fu, 0u)
#define FIXBITS(h) ((u32)__ballot(((h) & fixm) != 0u) & 0xfffu)
#define GRS_STEP4(s, uc, hc, lbc, fxc, evx, sg, J) do { \
        constexpr int sn_ = ((s) + 1) & 3; \
        StepRegs<NC>& ri_ = R[s]; StepRegs<NC>& rc_ = R[sn_]; \
        u32x4 (&xo_)[4] = X[(s) & 1]; u32x4 (&xn_)[4] = X[sn_ & 1]; \
        u32 H_[4], lb4_, seen_; u32x4 e_, o_; \
        LAS unsigned char* wp_ = wdst + sn_ * SLOTB; \
        SB; \
        if ((s) == 0) ald4(evn, ve, rsw, ue); \
        MF(0, wf[1], xn_[3]); RD1(s, 0, 0); ld_w(ri_, rsl, oW[s], 0); lb4_ = rdl(lbc, 8 + sn_); SB; \
        vm_wait<2 * NL + 1>(rc_, evx); \
        MF(1, wf[1], xn_[3]); RD1(s, 0, 1); HI01(H_, rc_.wn); SB; \
        MF(2, wf[1], xn_[3]); RD1(s, 0, 2); ld_w(ri_, rsl, oW[s], 1); HI23(H_, rc_.wn); SB; \
        MF(3, wf[1], xn_[3]); RD1(s, 0, 3); \
            e_[0] = PE(H_, rc_.wl, 0); e_[1] = PE(H_, rc_.wl, 1); e_[2] = PE(H_, rc_.wl, 2); e_[3] = PE(H_, rc_.wl, 3); SB; \
        MF(0, wf[0], xo_[0]); RD1(s, 1, 0); ald8(ri_.xa.n, vn, rsl, oA[s]); \
            *(LAS u32x4*)wp_ = e_; o_[0] = PO(H_, rc_.wl, 0); o_[1] = PO(H_, rc_.wl, 1); o_[2] = PO(H_, rc_.wl, 2); o_[3] = PO(H_, rc_.wl, 3); SB; \
        MF(1, wf[0], xo_[0]); RD1(s, 1, 1); \
            *(LAS u32x4*)(wp_ + 1024) = o_; lb4_ = rdl(lbc, 2 * sn_); HI01(H_, rc_.xa.n); SB; \
        if (((fxc) >> (8 + sn_)) & 1u) fix_w(evx, uc, sn_, rdl(hc, 8 + sn_)); \
        SB; \
        sig_step<((s) + 1) * 4>(cntv, sg); \
        MF(2, wf[0], xo_[0]); RD1(s, 1, 2); ald16(ri_.xa.l, vl, rsl, oA[s]); HI23(H_, rc_.xa.n); SB; \
        MF(3, wf[0], xo_[0]); RD1(s, 1, 3); xn_[0] = XPAIR(H_, rc_.xa.l, 0); SB; \
        MF(0, wf[1], xo_[1]); RD1(s, 2, 0); ald8(ri_.xb.n, vn, rsl, oB[s]); xn_[1] = XPAIR(H_, rc_.xa.l, 2); SB; \
        MF(1, wf[1], xo_[1]); RD1(s, 2, 1); lb4_ = rdl(lbc, 2 * sn_ + 1); HI01(H_, rc_.xb.n); SB; \
        MF(2, wf[1], xo_[1]); RD1(s, 2, 2); ald16(ri_.xb.l, vl, rsl, oB[s]); HI23(H_, rc_.xb.n); SB; \
        MF(3, wf[1], xo_[1]); RD1(s, 2, 3); xn_[2] = XPAIR(H_, rc_.xb.l, 0); SB; \
        MF(0, wf[0], xo_[2]); RD1(s, 3, 0); xn_[3] = XPAIR(H_, rc_.xb.l, 2); SB; \
        MF(1, wf[0], xo_[2]); RD1(s, 3, 1); seen_ = *(volatile LAS u32*)(cntp + (s) + 1); SB; \
        MF(2, wf[0], xo_[2]); RD1(s, 3, 2); SB; \
        MF(3, wf[0], xo_[2]); RD1(s, 3, 3); SB; \
        if (((fxc) >> (2 * sn_)) & 3u) { fix_x(rdl(hc, 2 * sn_), evx, 2 * sn_, uc, xn_[0], xn_[1]); fix_x(rdl(hc, 2 * sn_ + 1), evx, 2 * sn_ + 1, uc, xn_[2], xn_[3]); } \
        if (sg) wait_step(lds, (J) + 1, seen_); \
        } while (0)
#define GRS_STEP2(s, uc, hc, lbc, fxc, evx, sg, J) do { \
        constexpr int sn_ = ((s) + 1) & 3; \
        StepRegs<NC>& ri_ = R[s]; StepRegs<NC>& rc_ = R[sn_]; \
        u32x4 (&xo_)[4] = X[(s) & 1]; u32x4 (&xn_)[4] = X[sn_ & 1]; \
        u32 H_[4], lb4_, seen_; \
        LAS unsigned char* wp_ = wdst + sn_ * SLOTB; \
        SB; \
        if ((s) == 0) ald4(evn, ve, rsw, ue); \
        vm_wait<2 * NL>(rc_, evx); \
        MF(0, wf[1], xn_[3]); RD1(s, 0, 0); ld_w(ri_, rsl, oW[s], 0); \
            lb4_ = rdl(lbc, 8 + sn_); H_[0] = (rc_.w[2] & 0x87878787u) + lb4_; H_[1] = (rotr4(rc_.w[2]) & 0x87878787u) + lb4_; SB; \
        MF(1, wf[1], xn_[3]); RD1(s, 0, 1); ald8(ri_.xa.n, vn, rsl, oA[s]); \
            *(LAS u32x2*)wp_ = (u32x2){PE(H_, rc_.w, 0), PE(H_, rc_.w, 1)}; *(LAS u32x2*)(wp_ + 1024) = (u32x2){PO(H_, rc_.w, 0), PO(H_, rc_.w, 1)}; SB; \
        if (((fxc) >> (8 + sn_)) & 1u) fix_w(evx, uc, sn_, rdl(hc, 8 + sn_)); \
        SB; \
        sig_step<((s) + 1) * 4>(cntv, sg); \
        MF(0, wf[0], xo_[0]); RD1(s, 1, 0); lb4_ = rdl(lbc, 2 * sn_); HI01(H_, rc_.xa.n); SB; \
        MF(1, wf[0], xo_[0]); RD1(s, 1, 1); ald16(ri_.xa.l, vl, rsl, oA[s]); HI23(H_, rc_.xa.n); xn_[0] = XPAIR(H_, rc_.xa.l, 0); SB; \
        MF(0, wf[1], xo_[1]); RD1(s, 2, 0); ald8(ri_.xb.n, vn, rsl, oB[s]); xn_[1] = XPAIR(H_, rc_.xa.l, 2); lb4_ = rdl(lbc, 2 * sn_ + 1); SB; \
        MF(1, wf[1], xo_[1]); RD1(s, 2, 1); ald16(ri_.xb.l, vl, rsl, oB[s]); \
            HI01(H_, rc_.xb.n); seen_ = *(volatile LAS u32*)(cntp + (s) + 1); SB; \
        MF(0, wf[0], xo_[2]); RD1(s, 3, 0); HI23(H_, rc_.xb.n); xn_[2] = XPAIR(H_, rc_.xb.l, 0); SB; \
        MF(1, wf[0], xo_[2]); RD1(s, 3, 1); xn_[3] = XPAIR(H_, rc_.xb.l, 2); SB; \
        if (((fxc) >> (2 * sn_)) & 3u) { fix_x(rdl(hc, 2 * sn_), evx, 2 * sn_, uc, xn_[0], xn_[1]); fix_x(rdl(hc, 2 * sn_ + 1), evx, 2 * sn_ + 1, uc, xn_[2], xn_[3]); } \
        if (sg) wait_step(lds, (J) + 1, seen_); \
        } while (0)
#define GRS_STEP(s, uc, hc, lbc, fxc, evx, sg, J) do { { const u32x4 rsl = rsv; if constexpr (NC == 4) GRS_STEP4(s, uc, hc, lbc, fxc, evx, sg, J); else GRS_STEP2(s, uc, hc, lbc, fxc, evx, sg, J); } } while (0)
    const u32 notag = ((tag + 1u) & TAGMASK) << TAGSH;
    u32 hv = 0; bool cold = true, pf_ok = false;
    LOOK_SET("v228", notag); LOOK_SET("v229", notag);
    for (u32 u = 0; u < 8; ++u) {
        const u32 J = 4u * u, cntv = cnt0 + J * 4u, un = u < 7u ? u + 1u : 7u;
        LAS u32* cntp = (LAS u32*)(lds + LDS_CNT) + J;
        u32 pv; look_get<2 * NL>(pv, 0);
        if (!pf_ok) pv = notag;
        if (cold) {
            look_get<NL>(hv, 1);
            if (ok && !__all((hv >> TAGSH) == tag)) {
                ok = wait_tags(my + fo + u * 128u, tag, hv, my, call);
                if (!ok) *(volatile LAS u32*)(lds + LDS_CTL + 24) = 1u;
            }
            STAMP(6 + 2 * u); STAMP2(u, 4);
            const u32x4 rsu = unit_rsrc(u, true);
            issue(0, rsu, u); issue(1, rsu, u);
            LOOK_LD("v230", fo, rsw, un * 128u);
            const u32 lb0 = LBREP(hv), fx0 = FIXBITS(hv);
            u32 H_[4], lb4_ = rdl(lb0, 8);
            vm_wait<NL + 1>(R[0], evn);
            evc = evn;
            if constexpr (NC == 4) {
                HI01(H_, R[0].wn); HI23(H_, R[0].wn);
                *(LAS u32x4*)wdst = (u32x4){PE(H_, R[0].wl, 0), PE(H_, R[0].wl, 1), PE(H_, R[0].wl, 2), PE(H_, R[0].wl, 3)};
                *(LAS u32x4*)(wdst + 1024) = (u32x4){PO(H_, R[0].wl, 0), PO(H_, R[0].wl, 1), PO(H_, R[0].wl, 2), PO(H_, R[0].wl, 3)};
            } else {
                H_[0] = (R[0].w[2] & 0x87878787u) + lb4_; H_[1] = (rotr4(R[0].w[2]) & 0x87878787u) + lb4_;
                *(LAS u32x2*)wdst = (u32x2){PE(H_, R[0].w, 0), PE(H_, R[0].w, 1)}; *(LAS u32x2*)(wdst + 1024) = (u32x2){PO(H_, R[0].w, 0), PO(H_, R[0].w, 1)};
            }
            if ((fx0 >> 8) & 1u) fix_w(evc, u, 0, rdl(hv, 8));
            sig_step<0>(cntv, 1u);
            issue(2, rsu, u); issue(3, rsu, u);
            lb4_ = rdl(lb0, 0); HI01(H_, R[0].xa.n); HI23(H_, R[0].xa.n); X[0][0] = XPAIR(H_, R[0].xa.l, 0); X[0][1] = XPAIR(H_, R[0].xa.l, 2);
            lb4_ = rdl(lb0, 1); HI01(H_, R[0].xb.n); HI23(H_, R[0].xb.n); X[0][2] = XPAIR(H_, R[0].xb.l, 0); X[0][3] = XPAIR(H_, R[0].xb.l, 2);
            if (fx0 & 3u) { fix_x(rdl(hv, 0), evc, 0, u, X[0][0], X[0][1]); fix_x(rdl(hv, 1), evc, 1, u, X[0][2], X[0][3]); }
            look_get<2 * NL>(pv, 2);
            STAMP2(8 + u, 0);
            wait_step(lds, J, 0u);
            STAMP2(16 + u, 0);
        }
        bool have = ok && u < 7u && __all((pv >> TAGSH) == tag);
        u32 hn = pv;
        const bool look = ok && u < 7u;
        if (have) STAMP2(un, 0);
        const u32x4 rsn = unit_rsrc(un, true);
        u32x4 rsv = rsn; rsv[2] = have ? WS_NEED : 0u;
        const u32 ue = un * 1024u;
        const u32 lbc = LBREP(hv), fxc = FIXBITS(hv);
#define LATE(s) do { if (!have && look) { u32 v_; look_get<NL>(v_, 1); \
            if (__all((v_ >> TAGSH) == tag)) { have = true; hn = v_; STAMP2(un, s); rsv[2] = WS_NEED; \
                issue(0, rsn, un); if ((s) > 1) issue(1, rsn, un); if ((s) > 2) issue(2, rsn, un); } \
            else LOOK_LD("v229", fo, rsw, un * 128u); } } while (0)
        if (!have && look) LOOK_LD("v229", fo, rsw, un * 128u);
        GRS_STEP(0, u, hv, lbc, fxc, evc, 1u, J);
        LATE(1);
        GRS_STEP(1, u, hv, lbc, fxc, evc, 1u, J + 1);
        LOOK_LD("v228", fo, rsw, (u < 6u ? u + 2u : 7u) * 128u); pf_ok = have;
        LATE(2);
        GRS_STEP(2, u, hv, lbc, fxc, evc, 1u, J + 2);
        LATE(3);
        STAMP2(24 + u, 0);
        { const u32 lbn = LBREP(hn), fxn = have ? FIXBITS(hn) : 0u; GRS_STEP(3, un, hn, lbn, fxn, evn, (have ? 1u : 0u), J + 3); }
#undef LATE
        evc = evn;
        hv = hn; cold = !have;
        STAMP(7 + 2 * u);
        if (u == 0) STAMPC(28);
    }
    STAMPC(29);
    asm volatile("s_waitcnt vmcnt(0)" : "+v"(evn));
    reg_hold(R[0]); reg_hold(R[1]); reg_hold(R[2]); reg_hold(R[3]);
#pragma unroll
    for (int c = 0; c < NC; ++c) MF(c, wf[1], X[1][3]);
#undef GRS_STEP
#undef GRS_STEP2
#undef GRS_STEP4
    STAMP(2);
    LAS u32* ctl = (LAS u32*)(lds + LDS_CTL);
    if (!ok || rfl(__hip_atomic_load(ctl + 6, __ATOMIC_RELAXED, __HIP_MEMORY_SCOPE_WORKGROUP)) != 0u) return;
    float mx = 0.f;
#pragma unroll
    for (int c = 0; c < NC; ++c)
#pragma unroll
        for (int g = 0; g < 16; g += 2) mx = fmaxf(mx, fmaxf(fabsf(acc[c][g]), fabsf(acc[c][g + 1])));
    const u32 em = wave_umax((cvt_pk_bf16(mx, mx) >> 8) & 0x7fu);
    const u32 lb = em > 7u ? em - 7u : 0u;
    const rsrc_t dst = mk_rsrc(P.ws[q], WS_NEED);
    const u32 tau = ((me & 1u) * 2u + ((me >> 1) & 1u)) * 16u + (u32)n, k = me >> 2;
    u32 hd[2] = {0u, 0u};
    u32 L[NC / 2][8], H[NC / 2][8]; u32x4 np[NC / 2]; u64 bm[NC / 2];
#pragma unroll
    for (int b2 = 0; b2 < NC / 2; ++b2) {
#pragma unroll
        for (int t = 0; t < 8; ++t) {
            u32 d0, d1;
            if constexpr (NC == 4) { const int g = 8 * b2 + t; d0 = cvt_pk_bf16(acc[0][g], acc[1][g]); d1 = cvt_pk_bf16(acc[2][g], acc[3][g]); }
            else { d0 = cvt_pk_bf16(acc[0][2 * t], acc[1][2 * t]); d1 = cvt_pk_bf16(acc[0][2 * t + 1], acc[1][2 * t + 1]); }
            L[b2][t] = perm(d1, d0, 0x06040200u); H[b2][t] = perm(d1, d0, 0x07050301u);
        }
        bm[b2] = pack_prep(H[b2], lb, np[b2]);
    }
    if ((u32)w >= PUSH_K) while (rfl(__hip_atomic_load(ctl + 8, __ATOMIC_RELAXED, __HIP_MEMORY_SCOPE_WORKGROUP)) + PUSH_K <= (u32)w) __builtin_amdgcn_s_sleep(2);
#pragma unroll
    for (int b2 = 0; b2 < NC / 2; ++b2) {
        const u32 beta = hb * 2u + (NC == 4 ? (u32)b2 : ht);
        const u32 blk = ((k * 64 + tau) * 4 + b) * 4 + beta;
        hd[b2] = pack_emit(L[b2], H[b2], lb, np[b2], bm[b2], dst, OFF_PP + blk * BLK, OFF_PE + blk * ENT, OFF_PR + blk * RAWB, lane, false);
    }
    STAMP(3);
    drain();
    if (PUSH_K < 8u && lane == 0) __hip_atomic_fetch_add(ctl + 8, 1u, __ATOMIC_RELAXED, __HIP_MEMORY_SCOPE_WORKGROUP);
    if (lane < NC / 2)
        sys_store(P.ws[q] + OFF_PFLAG + (tau * 4 + b) * 128u + (k * 4 + hb * 2 + (NC == 4 ? (u32)lane : ht)) * 4u, (tag << TAGSH) | (lane == 0 ? hd[0] : hd[1]));
    STAMP(4);
}

struct Blk { Half h[2]; };
__device__ __forceinline__ void unpack_f32(const Blk& k, u32 hdr, u32 entv, u32 ebase, rsrc_t ws, u32 raw_off, float (&f)[32], int lane) {
#pragma unroll
    for (int hb = 0; hb < 2; ++hb) {
        u32 H[4]; high_bytes(k.h[hb].n, (u32)hb, hdr, entv, ebase, H, lane);
#pragma unroll
        for (int i = 0; i < 4; ++i) {
            const int t = 4 * hb + i;
            f[4 * t]     = __uint_as_float(perm(H[i], k.h[hb].l[i], 0x04000c0cu));
            f[4 * t + 1] = __uint_as_float(perm(H[i], k.h[hb].l[i], 0x05010c0cu));
            f[4 * t + 2] = __uint_as_float(perm(H[i], k.h[hb].l[i], 0x06020c0cu));
            f[4 * t + 3] = __uint_as_float(perm(H[i], k.h[hb].l[i], 0x07030c0cu));
        }
    }
    if (hdr & RAWBIT) {
#pragma unroll
        for (int c = 0; c < 4; ++c) { const u32x4 o = ld16(ws, (u32)lane * 16u, raw_off + 1024u * c);
#pragma unroll
            for (int i = 0; i < 4; ++i) { f[8 * c + 2 * i] = __uint_as_float(o[i] << 16); f[8 * c + 2 * i + 1] = __uint_as_float(o[i] & 0xffff0000u); } }
    }
}

__device__ __forceinline__ void reducer_wave(const Params& P, int be, int lane, u32 call, u32 tag, bool ok, LAS unsigned char* lds, char* my, int wave) {
    const int wg = blockIdx.x;
    const u32 tau = (u32)wg >> 2, wm = (u32)wg & 3u;
    const rsrc_t ws = mk_rsrc(my, WS_NEED);
    LAS u32* ctl = (LAS u32*)(lds + LDS_CTL);
    u32 hv = 0;
    if (ok) ok = wait_tags(my + OFF_PFLAG + (tau * 4 + wm) * 128u + (u32)((lane & 1) * 4 + be) * 4u, tag, hv, my, call);
    STAMP(22);
    float y[32];
    {
        const u32 b0 = ((0 * 64 + tau) * 4 + wm) * 4 + (u32)be, b1 = ((1 * 64 + tau) * 4 + wm) * 4 + (u32)be;
        const u32 vo = (u32)lane * 16u;
        const u32 von = (u32)(lane & 31) * 16u + (u32)(lane >> 5) * 8u;
        Blk k0, k1;
#pragma unroll
        for (int hb = 0; hb < 2; ++hb) {
            k0.h[hb].n = ld8(ws, von, OFF_PP + b0 * BLK + 2048u + hb * 512u); k0.h[hb].l = ld16(ws, vo, OFF_PP + b0 * BLK + hb * 1024u);
            k1.h[hb].n = ld8(ws, von, OFF_PP + b1 * BLK + 2048u + hb * 512u); k1.h[hb].l = ld16(ws, vo, OFF_PP + b1 * BLK + hb * 1024u);
        }
        const u32 ev = ld4(ws, (lane < 8 ? OFF_PE + b0 * ENT : OFF_PE + b1 * ENT) + (u32)(lane & 7) * 4u, 0);
        cfence();
        float z[32];
        unpack_f32(k0, rdl(hv, 0), ev, 0, ws, OFF_PR + b0 * RAWB, y, lane);
        unpack_f32(k1, rdl(hv, 1), ev, 8, ws, OFF_PR + b1 * RAWB, z, lane);
#pragma unroll
        for (int e = 0; e < 32; ++e) y[e] += z[e];
    }
    float mx = 0.f;
#pragma unroll
    for (int e = 0; e < 32; e += 2) mx = fmaxf(mx, fmaxf(fabsf(y[e]), fabsf(y[e + 1])));
    u32 mb = __float_as_uint(mx);
    mb = wave_umax(mb);
    STAMP(23);
    u32 last = 0;
    if (lane == 0) {
        __hip_atomic_fetch_max(ctl + 0, mb, __ATOMIC_RELAXED, __HIP_MEMORY_SCOPE_WORKGROUP);
        if (!ok) __hip_atomic_store(ctl + 4, 1u, __ATOMIC_RELAXED, __HIP_MEMORY_SCOPE_WORKGROUP);
        last = __hip_atomic_fetch_add(ctl + 1, 1u, __ATOMIC_ACQ_REL, __HIP_MEMORY_SCOPE_WORKGROUP);
    }
    last = rfl(last);
    u32 gbits = 0; bool good;
    if (last == 3u) {
        const u32 wmax = rfl(__hip_atomic_load(ctl + 0, __ATOMIC_RELAXED, __HIP_MEMORY_SCOPE_WORKGROUP));
        const u32 fail = rfl(__hip_atomic_load(ctl + 4, __ATOMIC_RELAXED, __HIP_MEMORY_SCOPE_WORKGROUP));
        good = fail == 0u;
        if (good) {
            if (lane == 0) sys_store64(my + OFF_LG + (u32)wg * 8u, ((u64)call << 32) | wmax);
            const u64 t0 = __builtin_amdgcn_s_memrealtime();
            for (;;) {
                u64 x = 0;
                if (lane < 8) x = sys_load64(my + OFF_GG + (u32)(wg & 7) * 128u + (u32)lane * 8u);
                else if (lane == 8) x = (u64)agent_load(my + OFF_TMO);
                const bool tm = __any(lane == 8 && (u32)x == call);
                const bool all = __all(lane >= 8 || (u32)(x >> 32) == call);
                if (tm) { good = false; break; }
                if (all) { gbits = wave_umax(lane < 8 ? (u32)x : 0u); break; }
                if (__builtin_amdgcn_s_memrealtime() - t0 > DEADLINE) { agent_store(my + OFF_TMO, call); good = false; break; }
                __builtin_amdgcn_s_sleep(1);
            }
        }
        if (lane == 0) {
            __hip_atomic_store(ctl + 2, gbits, __ATOMIC_RELAXED, __HIP_MEMORY_SCOPE_WORKGROUP);
            __hip_atomic_store(ctl + 3, good ? 1u : 2u, __ATOMIC_RELEASE, __HIP_MEMORY_SCOPE_WORKGROUP);
        }
    } else {
        u32 st;
        while ((st = rfl(__hip_atomic_load(ctl + 3, __ATOMIC_ACQUIRE, __HIP_MEMORY_SCOPE_WORKGROUP))) == 0u) __builtin_amdgcn_s_sleep(1);
        good = st == 1u;
        gbits = rfl(__hip_atomic_load(ctl + 2, __ATOMIC_RELAXED, __HIP_MEMORY_SCOPE_WORKGROUP));
    }
    STAMP(24);
    const float amax = __uint_as_float(gbits);
    const float scale = amax / 448.0f;
    const float inv = 1.0f / scale;
    u32x4 o0, o1;
#pragma unroll
    for (int i = 0; i < 8; ++i) {
        u32 v = (u32)__builtin_amdgcn_cvt_pk_fp8_f32(y[4 * i] * inv, y[4 * i + 1] * inv, 0, false);
        v = (u32)__builtin_amdgcn_cvt_pk_fp8_f32(y[4 * i + 2] * inv, y[4 * i + 3] * inv, (int)v, true);
        if (!good) v = 0x7f7f7f7fu;
        if (i < 4) o0[i] = v; else o1[i - 4] = v;
    }
    const u32 qi = tau >> 5, qj = (tau >> 4) & 1u, n = tau & 15u;
    const u32 row = qi * 256u + wm * 64u + (u32)(be >> 1) * 32u + (u32)(lane & 31);
    const u32 col = qj * 2048u + n * 128u + (is_narrow(n) ? (u32)(be & 1) * 64u + (u32)(lane >> 5) * 32u : (u32)(lane >> 5) * 64u + (u32)(be & 1) * 32u);
    const rsrc_t out = mk_rsrc(P.out, 2u << 20);
    __builtin_amdgcn_raw_buffer_store_b128(o0, out, (int)(row * 4096u + col), 0, 0);
    __builtin_amdgcn_raw_buffer_store_b128(o1, out, (int)(row * 4096u + col + 16u), 0, 0);
    STAMP(25); STAMPC(31);
}

__device__ __forceinline__ void leader_wave(const Params& P, int lane, u32 call, char* my, int wave) {
    const u64 t0 = __builtin_amdgcn_s_memrealtime();
    u32 m = 0; bool good = true;
    for (u32 it = 0;; ++it) {
        bool all = true; m = 0;
#pragma unroll
        for (int i = 0; i < 4; ++i) { const u64 x = sys_load64(my + OFF_LG + (u32)(i * 64 + lane) * 8u); all = all && (u32)(x >> 32) == call; m = max(m, (u32)x); }
        if (__all(all)) break;
        if ((it & 15u) == 15u && (rfl(agent_load(my + OFF_TMO)) == call || __builtin_amdgcn_s_memrealtime() - t0 > DEADLINE)) {
            agent_store(my + OFF_TMO, call); good = false; break; }
        __builtin_amdgcn_s_sleep(1);
    }
    STAMP(26);
    if (good) {
        m = wave_umax(m);
        sys_store64(P.ws[lane >> 3] + OFF_GG + (u32)(lane & 7) * 128u + (u32)P.rank * 8u, ((u64)call << 32) | m);
    }
    if (lane == 0) agent_store(my + OFF_CALL, call);
}

__global__ void __launch_bounds__(512, 2) gemm_rs_fused(Params P) {
    __shared__ __attribute__((aligned(16))) unsigned char smem[LDS_BYTES];
    LAS unsigned char* lds = (LAS unsigned char*)smem;
    const int tid = threadIdx.x, lane = tid & 63, wave = (int)rfl((u32)tid >> 6);
    const int b = blockIdx.x, xcd = b & 7, s = b >> 3;
    char* my = P.my;
    asm volatile("global_load_dword v231, %0, off sc1" :: "v"(my + OFF_CALL) : "v231");
    if (tid < 64) ((LAS u32*)(lds + LDS_CTL))[tid] = 0u;
    __syncthreads();
    auto get_call = [&]() { u32 c; asm volatile("s_waitcnt vmcnt(0)\n\tv_readfirstlane_b32 %0, v231" : "=s"(c)); return c + 1u; };
    STAMP(0); STAMPC(30);
    if (s < SEND_SLOTS && (u32)wave < SEND_WPW) {
        if (SEND_SLOTS == 8) sender_wave(P, s, (u32)xcd * SEND_WPW + (u32)wave, lane, my, get_call);
        else sender_wave(P, 2 * s + (xcd & 1), (u32)(xcd >> 1) * SEND_WPW + (u32)wave, lane, my, get_call);
    }
    const u32 call = get_call();
    const u32 tag = call & TAGMASK;
    int tg = 0, ts = s - SEND_SLOTS;
#pragma unroll
    for (int g = 0; g < 3; ++g) { const int w = ((NARROW_MASK >> g) & 1u) ? 8 : 4; if (tg == g && ts >= w) { ts -= w; tg = g + 1; } }
    const int tq = 4 * (xcd & 1) + (ts & 3), tn = 4 * tg + (xcd >> 1);
#define GRS_TILES do { \
        if (is_narrow((u32)tn)) gemm_wave<2>(P, tq, tn, (u32)ts >> 2, wave, lane, call, tag, lds, my, wave); \
        else gemm_wave<4>(P, tq, tn, 0u, wave, lane, call, tag, lds, my, wave); } while (0)
    if (s < SEND_SLOTS) {
        STAMP(1);
        if (b == 0 && wave == 0) leader_wave(P, lane, call, my, wave);
    } else GRS_TILES;
    if (wave >= 4) reducer_wave(P, wave - 4, lane, call, tag, true, lds, my, wave);
}

}

extern "C" void kernel_launch(int rank, int nranks, ncclComm_t comm, hipStream_t stream,
                              void* const* d_in, const int* in_sizes, int n_in,
                              void* d_out, int out_size,
                              const int* in_dtypes, int out_dtype,
                              void* const* d_out_all,
                              void* const* d_ws_all, size_t ws_bytes) {
    (void)comm; (void)in_dtypes; (void)out_dtype; (void)d_out_all;
    if (nranks != grs::NR || n_in < 2 || ws_bytes < grs::WS_NEED || out_size != 512 * 4096 ||
        in_sizes[0] != 4096 * 512 || in_sizes[1] != 512 * 4096) {
        fprintf(stderr, "gemm_rs: unexpected problem geometry\n");
        return;
    }
    grs::Params p;
    p.x = d_in[0]; p.w = d_in[1]; p.out = d_out;
    for (int i = 0; i < grs::NR; ++i) p.ws[i] = (char*)d_ws_all[i];
    p.my = p.ws[rank];
    p.rank = rank; p.pad = 0;
    hipLaunchKernelGGL(grs::gemm_rs_fused, dim3(256), dim3(512), 0, stream, p);
    HIP_CHECK(hipGetLastError());
}
```

```cpp
#include <hip/hip_runtime.h>
#include <rccl/rccl.h>
#include <cstdio>
#include <cstdint>

#define HIP_CHECK(expr) do { hipError_t _e = (expr); if (_e != hipSuccess) { \
    fprintf(stderr, "HIP error %s at %s:%d\n", hipGetErrorString(_e), __FILE__, __LINE__); \
    return; } } while (0)

namespace grs {

typedef unsigned u32;
typedef unsigned long long u64;
typedef unsigned u32x4 __attribute__((ext_vector_type(4)));
typedef unsigned u32x3 __attribute__((ext_vector_type(3)));
typedef unsigned u32x2 __attribute__((ext_vector_type(2)));
typedef unsigned short u16x2 __attribute__((ext_vector_type(2)));
typedef float f32x16 __attribute__((ext_vector_type(16)));
typedef short bf16x8 __attribute__((ext_vector_type(8)));
typedef __amdgpu_buffer_rsrc_t rsrc_t;
#define LAS __attribute__((address_space(3)))
#define GAS __attribute__((address_space(1)))

constexpr int NR = 8;
constexpr int AUX_SYS = 17;
constexpr u32 RSRC_W3 = 0x00020000u;
#define GRS_OUTAUX 0
#define GRS_TURNSLEEP 2
#define GRS_TAGSLEEP 2
constexpr u64 DEADLINE = 20000000ull;
constexpr u32 TAGSH = 14, TAGMASK = 0x3ffffu, RAWBIT = 1u << 13;

constexpr u32 OFF_CALL  = 0;
constexpr u32 OFF_TMO   = 128;
constexpr u32 OFF_DUMMY = 131072;
constexpr u32 OFF_XFLAG = 4096;
constexpr u32 OFF_WFLAG = 16384;
constexpr u32 OFF_PFLAG = 32768;
constexpr u32 OFF_SCTR  = 2048;
constexpr u32 OFF_LG    = 65536;
constexpr u32 OFF_GG    = 69632;
constexpr u32 OFF_DBG   = 56u << 20;
constexpr u32 BLK = 3072, ENT = 32, RAWB = 4096;
constexpr u32 OFF_XP = 1u << 20, OFF_WP = 7u << 20, OFF_PP = 13u << 20;
constexpr u32 OFF_XE = 19u << 20, OFF_WE = OFF_XE + 65536, OFF_PE = OFF_WE + 65536;
constexpr u32 OFF_XR = 20u << 20, OFF_WR = 28u << 20, OFF_PR = 36u << 20;
constexpr u32 OFF_HX = 44u << 20;
constexpr u32 OFF_HFLAG = 73728;
constexpr u32 WS_NEED = 48u << 20;

constexpr u32 SLOT = 16384, UNITB = 8192;
constexpr u32 LDS_CTL = 4 * SLOT;
constexpr u32 LDS_BYTES = 96 * 1024;

#define GRS_NARROW 9
#define GRS_D1 1
#define GRS_D2 3
#define GRS_D3 8
constexpr unsigned NARROW_MASK = GRS_NARROW, DEF1 = GRS_D1, DEF2 = GRS_D2, DEF3 = GRS_D3;
static_assert(DEF1 <= DEF2 && DEF2 <= DEF3 && DEF3 <= 8, "deferral must grow with the group");
#define GRS_SSLOTS 8
constexpr int SEND_SLOTS = GRS_SSLOTS;
static_assert(SEND_SLOTS == 4 || SEND_SLOTS == 8, "sender slots");
static_assert(NARROW_MASK < 16 && SEND_SLOTS + 16 + 4 * __builtin_popcount(NARROW_MASK) == 32, "32 slots per XCD");
__device__ __forceinline__ bool is_narrow(unsigned n) { return (NARROW_MASK >> (n >> 2)) & 1u; }

#define GRS_KSPLIT 5
constexpr u32 KSPLIT = GRS_KSPLIT;
static_assert(KSPLIT >= 1 && KSPLIT <= 8 && (KSPLIT == 8 || (SEND_SLOTS == 8 && (NARROW_MASK & 8u))), "helpers: 8 sender slots, narrow last group");

struct Params {
    const void* x; const void* w; void* out;
    char* ws[NR];
    char* my;
    int rank; u32 pad;
};

__device__ __forceinline__ u32 perm(u32 s0, u32 s1, u32 sel) { return __builtin_amdgcn_perm(s0, s1, sel); }
__device__ __forceinline__ u32 rotr4(u32 v) { return __builtin_amdgcn_alignbit(v, v, 4); }
__device__ __forceinline__ u32 rotl4(u32 v) { return __builtin_amdgcn_alignbit(v, v, 28); }
__device__ __forceinline__ u32 rfl(u32 v) { return (u32)__builtin_amdgcn_readfirstlane((int)v); }
__device__ __forceinline__ u32 rdl(u32 v, u32 l) { return (u32)__builtin_amdgcn_readlane((int)v, (int)l); }
__device__ __forceinline__ u32 pkmax(u32 a, u32 b) {
    return __builtin_bit_cast(u32, __builtin_elementwise_max(__builtin_bit_cast(u16x2, a), __builtin_bit_cast(u16x2, b)));
}
__device__ __forceinline__ u32 wave_umax(u32 v) {
    v = max(v, (u32)__builtin_amdgcn_update_dpp(0, (int)v, 0x111, 0xf, 0xf, true));
    v = max(v, (u32)__builtin_amdgcn_update_dpp(0, (int)v, 0x112, 0xf, 0xf, true));
    v = max(v, (u32)__builtin_amdgcn_update_dpp(0, (int)v, 0x114, 0xf, 0xf, true));
    v = max(v, (u32)__builtin_amdgcn_update_dpp(0, (int)v, 0x118, 0xf, 0xf, true));
    v = max(v, (u32)__builtin_amdgcn_update_dpp(0, (int)v, 0x142, 0xa, 0xf, false));
    v = max(v, (u32)__builtin_amdgcn_update_dpp(0, (int)v, 0x143, 0xc, 0xf, false));
    return rdl(v, 63);
}
__device__ __forceinline__ void drain() { asm volatile("s_waitcnt vmcnt(0)" ::: "memory"); }
__device__ __forceinline__ void cfence() { asm volatile("" ::: "memory"); }
__device__ __forceinline__ rsrc_t mk_rsrc(const void* p, u32 bytes) { return __builtin_amdgcn_make_buffer_rsrc((void*)p, 0, (int)bytes, RSRC_W3); }
#define EXP_LDAUX AUX_SYS
__device__ __forceinline__ u32x4 ld16(rsrc_t r, u32 voff, u32 soff) { return __builtin_amdgcn_raw_buffer_load_b128(r, (int)voff, (int)soff, EXP_LDAUX); }
__device__ __forceinline__ u32 ld4(rsrc_t r, u32 voff, u32 soff) { return __builtin_amdgcn_raw_buffer_load_b32(r, (int)voff, (int)soff, AUX_SYS); }
__device__ __forceinline__ void st16(rsrc_t r, u32 voff, u32 soff, u32x4 v) { __builtin_amdgcn_raw_buffer_store_b128(v, r, (int)(voff + soff), 0, AUX_SYS); }
__device__ __forceinline__ void st12(rsrc_t r, u32 voff, u32x3 v) { __builtin_amdgcn_raw_buffer_store_b96(v, r, (int)voff, 0, AUX_SYS); }
__device__ __forceinline__ void st4(rsrc_t r, u32 voff, u32 soff, u32 v) { __builtin_amdgcn_raw_buffer_store_b32(v, r, (int)voff, (int)soff, AUX_SYS); }
__device__ __forceinline__ u32 sys_load(const char* p) { return __hip_atomic_load((const GAS u32*)p, __ATOMIC_RELAXED, __HIP_MEMORY_SCOPE_SYSTEM); }
__device__ __forceinline__ u64 sys_load64(const char* p) { return __hip_atomic_load((const GAS u64*)p, __ATOMIC_RELAXED, __HIP_MEMORY_SCOPE_SYSTEM); }
__device__ __forceinline__ void sys_store(char* p, u32 v) { __hip_atomic_store((GAS u32*)p, v, __ATOMIC_RELAXED, __HIP_MEMORY_SCOPE_SYSTEM); }
__device__ __forceinline__ void sys_store64(char* p, u64 v) { __hip_atomic_store((GAS u64*)p, v, __ATOMIC_RELAXED, __HIP_MEMORY_SCOPE_SYSTEM); }
__device__ __forceinline__ u32 agent_load(const char* p) { return __hip_atomic_load((const GAS u32*)p, __ATOMIC_RELAXED, __HIP_MEMORY_SCOPE_AGENT); }
__device__ __forceinline__ void agent_store(char* p, u32 v) { __hip_atomic_store((GAS u32*)p, v, __ATOMIC_RELAXED, __HIP_MEMORY_SCOPE_AGENT); }

#define STAMPC(idx) do { } while (0)
#define STAMP2(idx, val) do { } while (0)
#define STAMP(idx) do { } while (0)

__device__ __forceinline__ u32 poll_sys(const char* p) { u32 x; asm volatile("global_load_dword %0, %1, off sc0 sc1\n\ts_waitcnt vmcnt(0)" : "=&v"(x) : "v"(p) : "memory"); return x; }
__device__ __forceinline__ u32 poll_agent(const char* p) { u32 x; asm volatile("global_load_dword %0, %1, off sc1\n\ts_waitcnt vmcnt(0)" : "=&v"(x) : "v"(p) : "memory"); return x; }
__device__ __forceinline__ void ld16x2_now(const char* p0, const char* p1, u32x4& a, u32x4& b) {
    asm volatile("global_load_dwordx4 %0, %2, off sc0 sc1\n\tglobal_load_dwordx4 %1, %3, off sc0 sc1\n\ts_waitcnt vmcnt(0)" : "=&v"(a), "=&v"(b) : "v"(p0), "v"(p1) : "memory");
}
__device__ __forceinline__ void give_up(char* my, u32 call) {
    asm volatile("global_store_dword %0, %1, off sc1" :: "v"(my + OFF_TMO), "v"(call) : "memory");
}
__device__ __forceinline__ bool wait_tags(const char* addr, u32 tag, u32& v, char* my, u32 call) {
    const u64 t0 = __builtin_amdgcn_s_memrealtime();
    for (u32 it = 0;; ++it) {
        const u32 x = poll_sys(addr);
        if (__all((x >> TAGSH) == tag)) { v = x; return true; }
        if ((it & 15u) == 15u) {
            if (rfl(poll_agent(my + OFF_TMO)) == call || __builtin_amdgcn_s_memrealtime() - t0 > DEADLINE) {
                give_up(my, call); v = 0; return false; }
        }
        __builtin_amdgcn_s_sleep(GRS_TAGSLEEP);
    }
}

__device__ __forceinline__ u32 lane_emax(const u32 (&H)[8]) {
    u32 m1 = 0, m2 = 0;
#pragma unroll
    for (int t = 0; t < 8; ++t) { const u32 g = H[t] | 0x80808080u; m1 = pkmax(m1, g); m2 = pkmax(m2, g << 8); }
    const u32 m = pkmax(m1, m2);
    return max((m >> 24) & 0x7fu, (m >> 8) & 0x7fu);
}
template <bool IS_W> __device__ __forceinline__ void chunks_from(const u32 (&L)[8], const u32 (&H)[8], u32x4 (&o)[4]) {
#pragma unroll
    for (int t = 0; t < 8; ++t) {
        const u32 e = perm(H[t], L[t], 0x05010400u), d = perm(H[t], L[t], 0x07030602u);
        if (IS_W) { o[2 * (t >> 2)][t & 3] = e; o[2 * (t >> 2) + 1][t & 3] = d; }
        else      { o[t >> 1][2 * (t & 1)] = e; o[t >> 1][2 * (t & 1) + 1] = d; }
    }
}
__device__ __forceinline__ u64 pack_prep(const u32 (&H)[8], u32 lb, u32x4& np, bool quad = false) {
    const u32 lb4 = lb * 0x01010101u;
    u32 T[8], acc = 0xffffffffu;
#pragma unroll
    for (int t = 0; t < 8; ++t) { T[t] = (H[t] | 0x80808080u) - lb4; acc &= T[t]; }
    u32 n[4];
#pragma unroll
    for (int j = 0; j < 4; ++j) {
        const u32 a = (H[2 * j] & 0x80808080u) | (T[2 * j] & 0x07070707u);
        const u32 b = (H[2 * j + 1] & 0x80808080u) | (T[2 * j + 1] & 0x07070707u);
        n[j] = a | rotl4(b);
    }
    const auto s0 = __builtin_amdgcn_permlane32_swap(n[0], n[2], false, false);
    const auto s1 = __builtin_amdgcn_permlane32_swap(n[1], n[3], false, false);
    np = quad ? (u32x4){n[0], n[1], n[2], n[3]} : (u32x4){s0[0], s1[0], s0[1], s1[1]};
    return __ballot((acc & 0x80808080u) != 0x80808080u);
}
__device__ __forceinline__ u32 pack_emit(const u32 (&L)[8], const u32 (&H)[8], u32 lb, u32x4 np, u64 bm, rsrc_t dst, u32 blk_off, u32 ent_off, u32 raw_off, int lane, bool is_w, bool quad = false, bool low_sent = false) {
    const u32 vo = (u32)lane * 16u;
    if (low_sent) {
        st16(dst, vo, blk_off + 2048u, np);
    } else if (quad) {
#pragma unroll
        for (int j = 0; j < 4; ++j) st12(dst, (u32)lane * 12u + blk_off + 768u * j, (u32x3){L[2 * j], L[2 * j + 1], np[j]});
    } else {
        st16(dst, vo, blk_off, (u32x4){L[0], L[1], L[2], L[3]});
        st16(dst, vo, blk_off + 1024u, (u32x4){L[4], L[5], L[6], L[7]});
        st16(dst, vo, blk_off + 2048u, np);
    }
    u32 hdr = lb;

    if (bm != 0ull) {
        const u32 lb4 = lb * 0x01010101u;
        u32 T[8];
#pragma unroll
        for (int t = 0; t < 8; ++t) T[t] = (H[t] | 0x80808080u) - lb4;
        u32 bits = 0;
#pragma unroll
        for (int t = 0; t < 8; ++t) { const u32 x = (~T[t] & 0x80808080u) >> 7;
            bits |= ((x & 1u) | ((x >> 7) & 2u) | ((x >> 14) & 4u) | ((x >> 21) & 8u)) << (4 * t); }
        const u32 nb0 = __popc(bits & 0xffffu), nb1 = __popc(bits >> 16);
        u32 n0 = 0, n1 = 0, p0 = 0, p1 = 0;
        while (bm) { const u32 l = (u32)__builtin_ctzll(bm); bm &= bm - 1; if ((u32)lane == l) { p0 = n0; p1 = n1; } n0 += rdl(nb0, l); n1 += rdl(nb1, l); }
        if (n0 + n1 <= 7u) {
            hdr |= (n0 << 7) | (n1 << 10);
            p1 += n0;
            while (bits) {
                const u32 idx = (u32)__builtin_ctz(bits); bits &= bits - 1;
                const u32 t = idx >> 2, by = idx & 3u;
                u32 hv = H[0];
#pragma unroll
                for (int tt = 1; tt < 8; ++tt) hv = (t == (u32)tt) ? H[tt] : hv;
                const u32 slot = t < 4u ? p0++ : p1++;
                const u32 where = is_w ? (u32)lane * 16u + (by >> 1) * 1024u + (t & 3u) * 4u + (by & 1u) * 2u + 1u : (u32)lane;
                st4(dst, slot * 4u, ent_off, (where << 16) | (t << 10) | (by << 8) | ((hv >> (8 * by)) & 0xffu));
            }
        } else {
            hdr |= RAWBIT;
            u32x4 o[4]; if (is_w) chunks_from<true>(L, H, o); else chunks_from<false>(L, H, o);
#pragma unroll
            for (int c = 0; c < 4; ++c) st16(dst, vo, raw_off + 1024u * c, o[c]);
        }
    }
    return hdr;
}
__device__ __forceinline__ u32 pack_store(const u32 (&L)[8], const u32 (&H)[8], u32 lb, rsrc_t dst, u32 blk_off, u32 ent_off, u32 raw_off, int lane, bool is_w) {
    u32x4 np; const u64 bm = pack_prep(H, lb, np);
    return pack_emit(L, H, lb, np, bm, dst, blk_off, ent_off, raw_off, lane, is_w);
}
struct Half { u32x4 l; u32x2 n; };
__device__ __forceinline__ void high_bytes(u32x2 n, u32 hb, u32 hdr, u32 entv, u32 ebase, u32 (&H)[4], int lane) {
    const u32 lb4 = (hdr & 0x7fu) * 0x01010101u;
    H[0] = (n[0] & 0x87878787u) + lb4; H[1] = (rotr4(n[0]) & 0x87878787u) + lb4;
    H[2] = (n[1] & 0x87878787u) + lb4; H[3] = (rotr4(n[1]) & 0x87878787u) + lb4;
    const u32 n0 = (hdr >> 7) & 7u, first = hb ? n0 : 0u, cnt = hb ? (hdr >> 10) & 7u : n0;
    for (u32 i = 0; i < cnt; ++i) {
        const u32 en = rdl(entv, ebase + first + i);
        const u32 sh = ((en >> 8) & 3u) * 8u, t = (en >> 10) & 7u;
        const u32 m = ((u32)lane == (en >> 16)) ? (0xffu << sh) : 0u, v = (en & 0xffu) << sh;
#pragma unroll
        for (int j = 0; j < 4; ++j) { const u32 mm = ((t & 3u) == (u32)j) ? m : 0u; H[j] = (H[j] & ~mm) | (v & mm); }
    }
}

#define GRS_SW (32 / SEND_SLOTS)
constexpr u32 NBLK = 512, SEND_WPW = GRS_SW, SEND_WAVES = SEND_SLOTS * SEND_WPW;
#define RAWCL "v200", "v201", "v202", "v203", "v204", "v205", "v206", "v207", "v208", "v209", "v210", "v211", "v212", "v213", "v214", "v215"
#define RAW_LD(OP, REG, IMM, voff, rs, so) asm volatile("buffer_load_" OP " " REG ", %0, %1, %2 offen offset:" #IMM :: "v"(voff), "s"(rs), "s"(so) : RAWCL)
#define RAW_P1(i, A, B) "v_perm_b32 %" #i ", " B ", " A ", %16\n\t"
#define RAW_P2(i, A, B) "v_perm_b32 %" #i ", " B ", " A ", %17\n\t"
#define RAW_PICK(N, A0, B0, A1, B1, A2, B2, A3, B3, A4, B4, A5, B5, A6, B6, A7, B7) \
    asm volatile("s_waitcnt vmcnt(" #N ")\n\t" \
        RAW_P1(0, A0, B0) RAW_P1(1, A1, B1) RAW_P1(2, A2, B2) RAW_P1(3, A3, B3) RAW_P1(4, A4, B4) RAW_P1(5, A5, B5) RAW_P1(6, A6, B6) RAW_P1(7, A7, B7) \
        RAW_P2(8, A0, B0) RAW_P2(9, A1, B1) RAW_P2(10, A2, B2) RAW_P2(11, A3, B3) RAW_P2(12, A4, B4) RAW_P2(13, A5, B5) RAW_P2(14, A6, B6) RAW_P2(15, A7, B7) \
        : "=v"(L[0]), "=v"(L[1]), "=v"(L[2]), "=v"(L[3]), "=v"(L[4]), "=v"(L[5]), "=v"(L[6]), "=v"(L[7]), \
          "=v"(H[0]), "=v"(H[1]), "=v"(H[2]), "=v"(H[3]), "=v"(H[4]), "=v"(H[5]), "=v"(H[6]), "=v"(H[7]) : "s"(selL), "s"(selH) : RAWCL)
__device__ __forceinline__ u32x4 rsrc_words(const void* p, u32 bytes) { return (u32x4){rfl((u32)(size_t)p), rfl((u32)((size_t)p >> 32)) & 0xffffu, bytes, RSRC_W3}; }

template <class GetCall>
__device__ __forceinline__ void sender_wave(const Params& P, int d, u32 wid, int lane, char* my, GetCall get_call) {
    const u32 di = (u32)d & 1u, dj = ((u32)d >> 1) & 1u, dk = (u32)d >> 2, me = (u32)P.rank;
    const u32 r = (u32)lane & 31u, h = (u32)lane >> 5;
    const rsrc_t dst = mk_rsrc(P.ws[d], WS_NEED);
    const u32x4 srcx = rsrc_words(P.x, 4u << 20), srcw = rsrc_words(P.w, 4u << 20);
    const u32 vx = (di * 256u + r) * 1024u + 32u * h + dk * 512u;
    const u32 cg = 16u * ((r >> 2) & 1u) + (r & 3u) + 4u * (r >> 3);
    const u32 vw0 = (16u * h + dk * 256u) * 8192u + dj * 4096u;
    const u32 vw = vw0 + cg * 8u, vwa = vw0 + cg * 4u;
    GAS u32* ctr = (GAS u32*)(my + OFF_SCTR + (u32)d * 128u);
    auto draw = [&]() { const u32 one = 1u;
        asm volatile("s_mov_b64 exec, 1\n\tglobal_atomic_add v216, %0, %1, off sc0\n\ts_mov_b64 exec, -1" :: "v"(ctr), "v"(one) : "v216", "memory"); };
    auto unit_of = [&](u32 idx, u32& rr) {
        u32 u = 0;
#pragma unroll
        for (u32 v = 0; v < 8u; ++v) {
            const u32 sz = 40u + 8u * ((v < 8u - DEF1 ? 1u : 0u) + (v < 8u - DEF2 ? 1u : 0u) + (v < 8u - DEF3 ? 1u : 0u));
            if (u == v && idx >= sz) { idx -= sz; u = v + 1u; }
        }
        if (u < 8u) { rr = idx; return u; }
        if (idx < 8u * DEF1) { rr = 40u + (idx & 7u); return 8u - DEF1 + (idx >> 3); }
        idx -= 8u * DEF1;
        if (idx < 8u * DEF2) { rr = 48u + (idx & 7u); return 8u - DEF2 + (idx >> 3); }
        idx -= 8u * DEF2;
        rr = 56u + (idx & 7u); return 8u - DEF3 + (idx >> 3);
    };
    auto load_raw = [&](u32 u, u32 rr) {
        if (rr < 32u) {
            const u32 so = ((rr >> 2) * 512u + (rr & 3u) * 64u) * 1024u + u * 64u;
            RAW_LD("dwordx4", "v[200:203]", 0, vx, srcx, so); RAW_LD("dwordx4", "v[204:207]", 16, vx, srcx, so);
            RAW_LD("dwordx4", "v[208:211]", 0, vx, srcx, so + 32768u); RAW_LD("dwordx4", "v[212:215]", 16, vx, srcx, so + 32768u);
        } else {
            const u32 so = (u * 32u + (rr & 1u) * 8u) * 8192u + ((rr - 32u) >> 1) * 256u;
            if (is_narrow((rr - 32u) >> 1)) {
                RAW_LD("dword", "v200", 0, vwa, srcw, so);               RAW_LD("dword", "v208", 128, vwa, srcw, so);
                RAW_LD("dword", "v201", 0, vwa, srcw, so + 8192u);       RAW_LD("dword", "v209", 128, vwa, srcw, so + 8192u);
                RAW_LD("dword", "v202", 0, vwa, srcw, so + 2u * 8192u);  RAW_LD("dword", "v210", 128, vwa, srcw, so + 2u * 8192u);
                RAW_LD("dword", "v203", 0, vwa, srcw, so + 3u * 8192u);  RAW_LD("dword", "v211", 128, vwa, srcw, so + 3u * 8192u);
                RAW_LD("dword", "v204", 0, vwa, srcw, so + 4u * 8192u);  RAW_LD("dword", "v212", 128, vwa, srcw, so + 4u * 8192u);
                RAW_LD("dword", "v205", 0, vwa, srcw, so + 5u * 8192u);  RAW_LD("dword", "v213", 128, vwa, srcw, so + 5u * 8192u);
                RAW_LD("dword", "v206", 0, vwa, srcw, so + 6u * 8192u);  RAW_LD("dword", "v214", 128, vwa, srcw, so + 6u * 8192u);
                RAW_LD("dword", "v207", 0, vwa, srcw, so + 7u * 8192u);  RAW_LD("dword", "v215", 128, vwa, srcw, so + 7u * 8192u);
            } else {
                RAW_LD("dwordx2", "v[200:201]", 0, vw, srcw, so);               RAW_LD("dwordx2", "v[202:203]", 0, vw, srcw, so + 8192u);
                RAW_LD("dwordx2", "v[204:205]", 0, vw, srcw, so + 2u * 8192u);  RAW_LD("dwordx2", "v[206:207]", 0, vw, srcw, so + 3u * 8192u);
                RAW_LD("dwordx2", "v[208:209]", 0, vw, srcw, so + 4u * 8192u);  RAW_LD("dwordx2", "v[210:211]", 0, vw, srcw, so + 5u * 8192u);
                RAW_LD("dwordx2", "v[212:213]", 0, vw, srcw, so + 6u * 8192u);  RAW_LD("dwordx2", "v[214:215]", 0, vw, srcw, so + 7u * 8192u);
            }
        }
    };
    u32 L[8], H[8], lb; u32x4 np; u64 bm;
    auto quad_blk = [&](u32 rr) { return rr >= 32u && is_narrow((rr - 32u) >> 1); };
    auto pick = [&](u32 rr, u32 younger) {
        if (younger >= 8u) asm volatile("s_waitcnt vmcnt(8)" ::: "memory");
        else if (younger >= 6u) asm volatile("s_waitcnt vmcnt(6)" ::: "memory");
        else if (younger == 5u) asm volatile("s_waitcnt vmcnt(5)" ::: "memory");
        else if (younger == 4u) asm volatile("s_waitcnt vmcnt(4)" ::: "memory");
        else asm volatile("s_waitcnt vmcnt(0)" ::: "memory");
        if (rr < 32u) {
            const u32 selL = 0x06040200u, selH = 0x07050301u;
            RAW_PICK(63, "v200", "v201", "v202", "v203", "v204", "v205", "v206", "v207", "v208", "v209", "v210", "v211", "v212", "v213", "v214", "v215");
        } else {
            const u32 selL = 0x06020400u, selH = 0x07030501u;
            if (is_narrow((rr - 32u) >> 1)) RAW_PICK(63, "v200", "v201", "v202", "v203", "v204", "v205", "v206", "v207", "v208", "v209", "v210", "v211", "v212", "v213", "v214", "v215");
            else RAW_PICK(63, "v200", "v202", "v204", "v206", "v208", "v210", "v212", "v214", "v201", "v203", "v205", "v207", "v209", "v211", "v213", "v215");
        }
        const u32 em = wave_umax(lane_emax(H));
        lb = em > 7u ? em - 7u : 0u;
        bm = pack_prep(H, lb, np, quad_blk(rr));
    };
    auto ticket = [&]() { u32 t; asm volatile("v_readfirstlane_b32 %0, v216" : "=s"(t)); return t; };
    u32 rr, u = unit_of(wid, rr);
    load_raw(u, rr);
    draw();
    const u32 call = get_call(), tag = call & TAGMASK;
    const u32 base = (call - 1u) * NBLK - SEND_WAVES;
    pick(rr, 0u);
    char* prev_flag = my + OFF_DUMMY + ((u32)d * SEND_WAVES + wid) * 128u; u32 prev_word = 0;
#define SSTAMP() do { } while (0)
    while (true) {
        const u32 nxt = ticket() - base;
        const bool more = nxt < NBLK;
        u32 nrr = 0, nu = 0;
        if (more) nu = unit_of(nxt, nrr);
        const bool is_w = rr >= 32u;
        u32 blk, offP, offE, offR, flag;
        if (!is_w) { blk = ((me * 8 + u) * 8 + (rr >> 2)) * 4 + (rr & 3u); offP = OFF_XP; offE = OFF_XE; offR = OFF_XR;
                     flag = OFF_XFLAG + ((rr >> 2) * 8 + u) * 128u + ((rr & 3u) * 8 + me) * 4u; }
        else       { const u32 n = (rr - 32u) >> 1, kf = rr & 1u; blk = ((me * 8 + u) * 16 + n) * 2 + kf; offP = OFF_WP; offE = OFF_WE; offR = OFF_WR;
                     flag = OFF_WFLAG + (n * 8 + u) * 128u + (kf * 8 + me) * 4u; }
        drain();
        if (more) { load_raw(nu, nrr); draw(); }
        if (lane == 0) sys_store(prev_flag, prev_word);
        SSTAMP();
        const bool quad = quad_blk(rr);
        const u64 bme = bm;
        const u32 hdr = pack_emit(L, H, lb, np, bm, dst, offP + blk * BLK, offE + blk * ENT, offR + blk * RAWB, lane, is_w, quad);
        prev_flag = P.ws[d] + flag; prev_word = (tag << TAGSH) | hdr;
        if (!more) break;
        u = nu; rr = nrr;
        pick(rr, (quad ? 5u : 4u) + ((hdr & RAWBIT) ? 4u : (bme != 0ull ? 1u : 0u)));
    }
    drain();
    if (lane == 0) sys_store(prev_flag, prev_word);
    SSTAMP();
}

__device__ __forceinline__ u32 cvt_pk_bf16(float lo, float hi) { u32 r; asm("v_cvt_pk_bf16_f32 %0, %1, %2" : "=v"(r) : "v"(lo), "v"(hi)); return r; }
__device__ __forceinline__ u32x2 ld8(rsrc_t r, u32 voff, u32 soff) { return __builtin_amdgcn_raw_buffer_load_b64(r, (int)voff, (int)soff, EXP_LDAUX); }
template <int NC> struct StepRegs;
template <> struct StepRegs<4> { Half xa, xb; u32x4 wl; u32x2 wn; };
template <> struct StepRegs<2> { Half xa, xb; u32x3 w; };

__device__ __forceinline__ void ald16(u32x4& d, u32 voff, u32x4 rs, u32 soff) { asm volatile("buffer_load_dwordx4 %0, %1, %2, %3 offen sc0 sc1" : "+v"(d) : "v"(voff), "s"(rs), "s"(soff)); }
__device__ __forceinline__ void ald12(u32x3& d, u32 voff, u32x4 rs, u32 soff) { asm volatile("buffer_load_dwordx3 %0, %1, %2, %3 offen sc0 sc1" : "+v"(d) : "v"(voff), "s"(rs), "s"(soff)); }
__device__ __forceinline__ void ald8(u32x2& d, u32 voff, u32x4 rs, u32 soff) { asm volatile("buffer_load_dwordx2 %0, %1, %2, %3 offen sc0 sc1" : "+v"(d) : "v"(voff), "s"(rs), "s"(soff)); }
__device__ __forceinline__ void ald4(u32& d, u32 voff, u32x4 rs, u32 soff) { asm volatile("buffer_load_dword %0, %1, %2, %3 offen sc0 sc1" : "+v"(d) : "v"(voff), "s"(rs), "s"(soff)); }
#define LOOK_SET(R, val) asm volatile("v_mov_b32 " R ", %0" :: "s"(val) : R)
#define LOOK_LD(R, voff, rs, soff) asm volatile("buffer_load_dword " R ", %0, %1, %2 offen sc0 sc1" :: "v"(voff), "s"(rs), "s"(soff) : R)
#define LOOK_GET(R, N, dstv) asm volatile("s_waitcnt vmcnt(" #N ")\n\tv_mov_b32 %0, " R : "=v"(dstv))
template <int N> __device__ __forceinline__ void look_get(u32& d, int which) {
    if (which == 0) asm volatile("s_waitcnt vmcnt(%1)\n\tv_mov_b32 %0, v228" : "=v"(d) : "n"(N));
    else if (which == 1) asm volatile("s_waitcnt vmcnt(%1)\n\tv_mov_b32 %0, v229" : "=v"(d) : "n"(N));
    else asm volatile("s_waitcnt vmcnt(%1)\n\tv_mov_b32 %0, v230" : "=v"(d) : "n"(N));
}
template <int N> __device__ __forceinline__ void vm_wait(StepRegs<4>& r, u32x2& ev) {
    asm volatile("s_waitcnt vmcnt(%7)" : "+v"(r.wl), "+v"(r.wn), "+v"(r.xa.l), "+v"(r.xa.n), "+v"(r.xb.l), "+v"(r.xb.n), "+v"(ev) : "n"(N)); }
template <int N> __device__ __forceinline__ void vm_wait(StepRegs<2>& r, u32x2& ev) {
    asm volatile("s_waitcnt vmcnt(%6)" : "+v"(r.w), "+v"(r.xa.l), "+v"(r.xa.n), "+v"(r.xb.l), "+v"(r.xb.n), "+v"(ev) : "n"(N)); }
__device__ __forceinline__ void reg_hold(StepRegs<4>& r) { asm volatile("" : "+v"(r.wl), "+v"(r.wn), "+v"(r.xa.l), "+v"(r.xa.n), "+v"(r.xb.l), "+v"(r.xb.n)); }
__device__ __forceinline__ void reg_hold(StepRegs<2>& r) { asm volatile("" : "+v"(r.w), "+v"(r.xa.l), "+v"(r.xa.n), "+v"(r.xb.l), "+v"(r.xb.n)); }
constexpr u32 LDS_CNT = LDS_CTL + 64;
template <int OFF> __device__ __forceinline__ void sig_step(u32 cnt_addr, u32 add) {
    asm volatile("s_mov_b64 exec, 1\n\tds_add_u32 %0, %1 offset:%2\n\ts_mov_b64 exec, -1" :: "v"(cnt_addr), "v"(add), "n"(OFF) : "memory");
}
template <int OFF, int LANE> __device__ __forceinline__ void sig_fix(u32 cnt_addr, u32 add, u32 fx, u32 ad, u32 byte) {
    asm volatile("s_and_b32 exec_lo, %2, %5\n\ts_mov_b32 exec_hi, 0\n\tds_write_b8 %3, %4\n\t"
                 "s_mov_b64 exec, 1\n\tds_add_u32 %0, %1 offset:%6\n\ts_mov_b64 exec, -1"
                 :: "v"(cnt_addr), "v"(add), "s"(fx), "v"(ad), "v"(byte), "n"(1 << LANE), "n"(OFF) : "memory", "scc");
}
__device__ __forceinline__ void wait_step(LAS unsigned char* lds, u32 J, u32 seen) {
    if (__builtin_expect(rfl(seen) == 8u, 1)) return;
    while (rfl(__hip_atomic_load((LAS u32*)(lds + LDS_CNT) + J, __ATOMIC_RELAXED, __HIP_MEMORY_SCOPE_WORKGROUP)) != 8u) __builtin_amdgcn_s_sleep(1);
    cfence();
}

#define PL_LEAF(n) "v_readlane_b32 %8, %" #n ", m0\n\ts_and_b32 %8, %8, %11\n\ts_or_b32 %8, %8, %12\n\tv_writelane_b32 %" #n ", %8, m0\n\t"
__device__ __forceinline__ void patch_lane(u32x4& a, u32x4& c, u32 D, u32 ln, u32 keep, u32 v) {
    u32 a0 = a[0], a1 = a[1], a2 = a[2], a3 = a[3], c0 = c[0], c1 = c[1], c2 = c[2], c3 = c[3], t;
    asm volatile("s_mov_b32 m0, %10\n\t"
                 "s_bitcmp1_b32 %9, 2\n\ts_cbranch_scc1 .Lpl4_%=\n\t"
                 "s_bitcmp1_b32 %9, 1\n\ts_cbranch_scc1 .Lpl2_%=\n\t"
                 "s_bitcmp1_b32 %9, 0\n\ts_cbranch_scc1 .Lpl1_%=\n\t"
                 PL_LEAF(0) "s_branch .Lple_%=\n"
                 ".Lpl1_%=:\n\t" PL_LEAF(1) "s_branch .Lple_%=\n"
                 ".Lpl2_%=:\n\ts_bitcmp1_b32 %9, 0\n\ts_cbranch_scc1 .Lpl3_%=\n\t"
                 PL_LEAF(2) "s_branch .Lple_%=\n"
                 ".Lpl3_%=:\n\t" PL_LEAF(3) "s_branch .Lple_%=\n"
                 ".Lpl4_%=:\n\ts_bitcmp1_b32 %9, 1\n\ts_cbranch_scc1 .Lpl6_%=\n\t"
                 "s_bitcmp1_b32 %9, 0\n\ts_cbranch_scc1 .Lpl5_%=\n\t"
                 PL_LEAF(4) "s_branch .Lple_%=\n"
                 ".Lpl5_%=:\n\t" PL_LEAF(5) "s_branch .Lple_%=\n"
                 ".Lpl6_%=:\n\ts_bitcmp1_b32 %9, 0\n\ts_cbranch_scc1 .Lpl7_%=\n\t"
                 PL_LEAF(6) "s_branch .Lple_%=\n"
                 ".Lpl7_%=:\n\t" PL_LEAF(7)
                 ".Lple_%=:"
                 : "+v"(a0), "+v"(a1), "+v"(a2), "+v"(a3), "+v"(c0), "+v"(c1), "+v"(c2), "+v"(c3), "=&s"(t)
                 : "s"(D), "s"(ln), "s"(keep), "s"(v) : "m0", "scc");
    a = (u32x4){a0, a1, a2, a3}; c = (u32x4){c0, c1, c2, c3};
}

#define P2_LEAF(n) "v_readlane_b32 %16, %" #n ", m0\n\ts_andn2_b32 %16, %16, %18\n\ts_or_b32 %16, %16, %19\n\tv_writelane_b32 %" #n ", %16, m0\n\t"
#define P2_HALF(L, END, r0, r1, r2, r3, r4, r5, r6, r7) \
    "s_bfe_u32 %16, %17, 0x10008\n\ts_lshr_b32 m0, %17, 16\n\ts_lshl4_add_u32 %16, %16, 8\n\ts_and_b32 %19, %17, 0xff\n\t" \
    "s_lshl_b32 %18, 0xff, %16\n\ts_lshl_b32 %19, %19, %16\n\t" \
    "s_bitcmp1_b32 %17, 11\n\ts_cbranch_scc1 .Lq" L "4_%=\n\t" \
    "s_bitcmp1_b32 %17, 10\n\ts_cbranch_scc1 .Lq" L "2_%=\n\t" \
    "s_bitcmp1_b32 %17, 9\n\ts_cbranch_scc1 .Lq" L "1_%=\n\t" \
    P2_LEAF(r0) END \
    ".Lq" L "1_%=:\n\t" P2_LEAF(r1) END \
    ".Lq" L "2_%=:\n\ts_bitcmp1_b32 %17, 9\n\ts_cbranch_scc1 .Lq" L "3_%=\n\t" \
    P2_LEAF(r2) END \
    ".Lq" L "3_%=:\n\t" P2_LEAF(r3) END \
    ".Lq" L "4_%=:\n\ts_bitcmp1_b32 %17, 10\n\ts_cbranch_scc1 .Lq" L "6_%=\n\t" \
    "s_bitcmp1_b32 %17, 9\n\ts_cbranch_scc1 .Lq" L "5_%=\n\t" \
    P2_LEAF(r4) END \
    ".Lq" L "5_%=:\n\t" P2_LEAF(r5) END \
    ".Lq" L "6_%=:\n\ts_bitcmp1_b32 %17, 9\n\ts_cbranch_scc1 .Lq" L "7_%=\n\t" \
    P2_LEAF(r6) END \
    ".Lq" L "7_%=:\n\t" P2_LEAF(r7) END
template <int S> __device__ __forceinline__ void patch_step(u32x4 (&x)[4], u32x2 ev, u32 fx, u32 two) {
    u32 r[16], t, en, msk, val, more; const u32 e0 = ev[0], e1 = ev[1];
#pragma unroll
    for (int i = 0; i < 16; ++i) r[i] = x[i >> 2][i & 3];
    asm volatile("s_and_b32 %16, %23, %27\n\ts_cbranch_scc1 .Lqs_%=\n"
                 ".Lqr_%=:\n\t"
                 ".subsection 1\n"
                 ".Lqs_%=:\n\ts_bitcmp1_b32 %23, %25\n\ts_cbranch_scc0 .Lqb_%=\n\t"
                 "v_readlane_b32 %17, %21, %25\n\ts_and_b32 %20, %23, %29\n\ts_and_b32 %16, %24, %28\n\ts_or_b32 %20, %20, %16\n"
                 ".Lqad_%=:\n\t"
                 P2_HALF("a", "s_cmp_lg_u32 %20, 0\n\ts_cbranch_scc0 .Lqr_%=\n\ts_branch .Lqam_%=\n", 0, 1, 2, 3, 4, 5, 6, 7)
                 ".Lqam_%=:\n\ts_bitcmp1_b32 %20, %25\n\ts_cbranch_scc0 .Lqb_%=\n\t"
                 "s_bitset0_b32 %20, %25\n\tv_readlane_b32 %17, %22, %25\n\ts_branch .Lqad_%=\n"
                 ".Lqb_%=:\n\t"
                 "v_readlane_b32 %17, %21, %26\n\ts_and_b32 %20, %24, %29\n"
                 ".Lqbd_%=:\n\t"
                 P2_HALF("b", "s_cmp_lg_u32 %20, 0\n\ts_cbranch_scc0 .Lqr_%=\n\ts_branch .Lqbm_%=\n", 8, 9, 10, 11, 12, 13, 14, 15)
                 ".Lqbm_%=:\n\ts_mov_b32 %20, 0\n\tv_readlane_b32 %17, %22, %26\n\ts_branch .Lqbd_%=\n"
                 "\t.subsection 0"
                 : "+v"(r[0]), "+v"(r[1]), "+v"(r[2]), "+v"(r[3]), "+v"(r[4]), "+v"(r[5]), "+v"(r[6]), "+v"(r[7]),
                   "+v"(r[8]), "+v"(r[9]), "+v"(r[10]), "+v"(r[11]), "+v"(r[12]), "+v"(r[13]), "+v"(r[14]), "+v"(r[15]),
                   "=&s"(t), "=&s"(en), "=&s"(msk), "=&s"(val), "=&s"(more)
                 : "v"(e0), "v"(e1), "s"(fx), "s"(two), "n"(2 * S), "n"(2 * S + 1), "n"(3 << (2 * S)), "n"(1 << (2 * S)), "n"(2 << (2 * S))
                 : "m0", "scc");
#pragma unroll
    for (int i = 0; i < 16; ++i) x[i >> 2][i & 3] = r[i];
}

#define RARE(c) __builtin_expect(!!(c), 0)
#define GRS_PK 0x2222
#define GRS_LOWFIRST 1
#define GRS_LOWX 0
#define GRS_RELN 2
#define GRS_RELW 5
template <int NC>
__device__ __forceinline__ void gemm_wave(const Params& P, int q, int n, u32 ht, int w, int lane, u32 call, u32 tag, LAS unsigned char* lds, char* my, int wave,
                                          u32 ubeg, u32 uend, u32 role, u32 hid) {
    const u32 PUSH_K = (GRS_PK >> (4 * (n >> 2))) & 15u;
    constexpr u32 SLOTB = NC * 4096u, SUBB = NC * 1024u;
    constexpr int NL = NC == 4 ? 6 : 5;
    const u32 me = (u32)P.rank;
    const u32 vo = (u32)lane * 16u, von = (u32)(lane & 31) * 16u + (u32)(lane >> 5) * 8u;
    const u32 b = (u32)w >> 1, hb = (u32)w & 1u;
    const u32 wu = (u32)w >> 2, kf = ((u32)w >> 1) & 1u;
    const u32 wh = NC == 4 ? hb : ht;
    bool ok = true;
    const u32 fo = (lane & 8) == 0 ? OFF_XFLAG + (u32)(q * 8) * 128u + (b * 8 + ((me + (u32)(lane & 7)) & 7u)) * 4u
                                   : OFF_WFLAG + (u32)(n * 8) * 128u + (kf * 8 + ((me + 2u * (u32)(lane & 3) + wu) & 7u)) * 4u;
    StepRegs<NC> R[4];
#pragma unroll
    for (int s = 0; s < 4; ++s) { R[s].xa.l = (u32x4){0u, 0u, 0u, 0u}; R[s].xb.l = R[s].xa.l; R[s].xa.n = (u32x2){0u, 0u}; R[s].xb.n = R[s].xa.n;
        if constexpr (NC == 4) { R[s].wl = R[s].xa.l; R[s].wn = R[s].xa.n; } else R[s].w = (u32x3){0u, 0u, 0u}; }
    constexpr u32 USTR = 32u * BLK;
    const u32 vl = vo + hb * 1024u, vn = von + 2048u + hb * 512u;
    const u32 vw = (u32)lane * 12u + (2u * ht + hb) * 768u;
    u32 oA[4], oB[4], oW[4];
#pragma unroll
    for (int s = 0; s < 4; ++s) {
        const u32 pa = (me + 2 * s) & 7u, pb = (me + 2 * s + 1) & 7u, pw = wu ? pb : pa;
        oA[s] = OFF_XP + (((pa * 8) * 8 + q) * 4 + b) * BLK; oB[s] = OFF_XP + (((pb * 8) * 8 + q) * 4 + b) * BLK;
        oW[s] = OFF_WP + (((pw * 8) * 16 + n) * 2 + kf) * BLK;
    }
    u32 ve, vh;
    { const u32 px = (me + (u32)(lane & 7)) & 7u, pw = (me + 2u * (u32)(lane & 3) + wu) & 7u;
      ve = (lane & 8) == 0 ? OFF_XE + (((px * 8) * 8 + q) * 4 + b) * ENT : OFF_WE + (((pw * 8) * 16 + n) * 2 + kf) * ENT;
      vh = ((lane & 8) == 0 ? hb : wh) ? 0x1cu : 0u; }
#define EVADDR(h) (ve + (((h) >> 5) & vh))
    const u32 rs0 = rfl((u32)(size_t)my), rs1 = rfl((u32)((size_t)my >> 32)) & 0xffffu;
    const u32x4 rsw = {rs0, rs1, WS_NEED, RSRC_W3};
    auto unit_rsrc = [&](u32 u, bool live) {
        const u64 a = ((u64)rs0 | ((u64)rs1 << 32)) + (u64)(u * USTR);
        return (u32x4){(u32)a, (u32)(a >> 32), live ? WS_NEED : 0u, RSRC_W3}; };
    auto ld_w = [&](StepRegs<NC>& r, u32x4 rs, u32 so, int part) {
        if constexpr (NC == 4) { if (part == 0) ald8(r.wn, vn, rs, so); else ald16(r.wl, vl, rs, so); } else { if (part == 0) ald12(r.w, vw, rs, so); } };
    u32x2 evc = {0u, 0u}, evn = {0u, 0u};
    auto issue = [&](int s, u32x4 rs, u32 u, u32 h) {
        if (s == 0) ald8(evn, EVADDR(h), rsw, u * 1024u);
        ld_w(R[s], rs, oW[s], 0); ld_w(R[s], rs, oW[s], 1);
        ald8(R[s].xa.n, vn, rs, oA[s]); ald16(R[s].xa.l, vl, rs, oA[s]);
        ald8(R[s].xb.n, vn, rs, oB[s]); ald16(R[s].xb.l, vl, rs, oB[s]);
    };
    LAS unsigned char* wdst = lds + (wu * 2u + kf) * SUBB + (NC == 4 ? hb * 2048u + (u32)lane * 16u : (u32)lane * 16u + hb * 8u);
    const u32 cntsh = hb ? 10u : 7u, wcntsh = wh ? 10u : 7u;
    const u32 fsh = (lane & 8) == 0 ? cntsh : wcntsh;
#define FST_BEGIN do { } while (0)
#define FST_END(acc, cnt) do { } while (0)
    auto put_x = [&](u32 en, u32x4& a, u32x4& c) {
        const u32 sh = ((en >> 4) & 16u) + 8u;
        patch_lane(a, c, (en >> 9) & 7u, en >> 16, ~(0xffu << sh), (en & 0xffu) << sh); };
    auto put_w = [&](u32 en, int s) {
        const u32 ad = (u32)(size_t)(lds + s * SLOTB + (wu * 2u + kf) * SUBB + (NC == 4 ? hb * 2048u : 0u)) + (en >> 16);
        u32 va, vv;
        asm volatile("v_mov_b32 %0, %2\n\tv_mov_b32 %1, %3\n\ts_mov_b64 exec, 1\n\tds_write_b8 %0, %1\n\ts_mov_b64 exec, -1"
                     : "=&v"(va), "=&v"(vv) : "s"(ad), "s"(en) : "memory"); };
    auto rep_x = [&](bool slow, u32 ev, u32 h, u32 r, u32 u, u32x4& a, u32x4& c) {
        u32 left = 1u; const char* ep = nullptr;
        if (RARE(slow)) {
            const u32 hdr = rdl(h, r), bx = ((((me + r) & 7u) * 8 + u) * 8 + q) * 4 + b;
            if (hdr & RAWBIT) {
                const char* rp = my + OFF_XR + bx * RAWB + hb * 2048u + vo;
                ld16x2_now(rp, rp + 1024, a, c);
            }
            left = (hdr >> cntsh) & 7u; ep = my + OFF_XE + bx * ENT + (hb ? (hdr >> 7) & 7u : 0u) * 4u;
        }
        for (; left != 0u; --left) { u32 en; if (slow) { en = rfl(poll_sys(ep)); ep += 4; } else en = rdl(ev, r); put_x(en, a, c); }
    };
    struct FixBits { u32 fx, two; };
    static_assert(RAWBIT == 1u << 13, "fixbits");
    auto fixbits = [](u32 c) {
        const u32 any = (u32)__ballot(c != 0u) & 0xfffu, slow = (u32)__ballot(c > 2u) & 0xfffu, two = (u32)__ballot(c == 2u) & 0xfffu;
        return FixBits{(any & ~slow) | (slow << 16), two}; };
    auto fix_w = [&](u32 fx, u32 ev, u32 h, u32 u, int s) {
        FST_BEGIN;
        const bool slow = (fx >> (24 + s)) & 1u;
        u32 left = 1u; const char* ep = nullptr;
        if (RARE(slow)) {
            LAS unsigned char* p = lds + s * SLOTB + (wu * 2u + kf) * SUBB + (NC == 4 ? hb * 2048u : 0u);
            const u32 hw = rdl(h, 8 + s), bw = ((((me + 2 * s + wu) & 7u) * 8 + u) * 16 + n) * 2 + kf;
            if (hw & RAWBIT) {
                const char* rp = my + OFF_WR + bw * RAWB + wh * 2048u + vo;
                u32x4 v0, v1; ld16x2_now(rp, rp + 1024, v0, v1);
                if (NC == 4) { *(LAS u32x4*)(p + lane * 16) = v0; *(LAS u32x4*)(p + 1024 + lane * 16) = v1; }
                else { *(LAS u32x2*)(p + lane * 16 + hb * 8u) = hb ? (u32x2){v0[2], v0[3]} : (u32x2){v0[0], v0[1]};
                       *(LAS u32x2*)(p + 1024 + lane * 16 + hb * 8u) = hb ? (u32x2){v1[2], v1[3]} : (u32x2){v1[0], v1[1]}; }
            }
            left = (hw >> wcntsh) & 7u; ep = my + OFF_WE + bw * ENT + (wh ? (hw >> 7) & 7u : 0u) * 4u;
        }
        for (; left != 0u; --left) { u32 en; if (slow) { en = rfl(poll_sys(ep)); ep += 4; } else en = rdl(ev, 8 + s); put_w(en, s); }
        FST_END(st_fw, n_fw);
    };
    auto fix_x = [&](u32 fx, u32 ev, u32 h, u32 u, int s, u32x4 (&x)[4]) {
        FST_BEGIN;
        if (((fx | (fx >> 16)) >> (2 * s)) & 1u) rep_x((fx >> (16 + 2 * s)) & 1u, ev, h, 2 * s, u, x[0], x[1]);
        if (((fx | (fx >> 16)) >> (2 * s + 1)) & 1u) rep_x((fx >> (17 + 2 * s)) & 1u, ev, h, 2 * s + 1, u, x[2], x[3]);
        FST_END(st_fx, n_fx);
    };
#define FIX_X(f, ev, h, u, s, x) do { patch_step<s>(x, ev, (f).fx, (f).two); \
        if (RARE(((f).fx >> (16 + 2 * (s))) & 3u)) fix_x((f).fx & ~0xffu, ev[0], h, u, s, x); } while (0)
    const u32 wlb = (u32)(size_t)(lds + (u32)(lane & 3) * SLOTB + (wu * 2u + kf) * SUBB + (NC == 4 ? hb * 2048u : 0u));
#define FIX_W(f, ev, h, u, s) do { if (RARE((((f).two | ((f).fx >> 16)) >> (8 + (s))) & 1u)) { \
        if (((f).fx >> (24 + (s))) & 1u) fix_w((f).fx, ev[0], h, u, s); else put_w(rdl(ev[1], 8 + (s)), s); } } while (0)
    f32x16 acc[NC];
#pragma unroll
    for (int c = 0; c < NC; ++c)
#pragma unroll
        for (int g = 0; g < 16; ++g) acc[c][g] = 0.f;
    u32x4 X[2][4]; bf16x8 wf[2][NC];
#pragma unroll
    for (int c = 0; c < NC; ++c) wf[1][c] = (bf16x8){0, 0, 0, 0, 0, 0, 0, 0};
    X[1][3] = (u32x4){0u, 0u, 0u, 0u};
    LAS unsigned char* wb = lds + lane * 16;
    const u32 cnt0 = (u32)(size_t)(lds + LDS_CNT);
#define MF(c, F, XF) acc[c] = __builtin_amdgcn_mfma_f32_32x32x16_bf16(F[c], __builtin_bit_cast(bf16x8, XF), acc[c], 0, 0, 0)
#define SB __builtin_amdgcn_sched_barrier(0)
#define RD1(s, sub, c) wf[(sub) & 1][c] = *(const LAS bf16x8*)(wb + (s) * SLOTB + (sub) * SUBB + (c) * 1024)
#define HI01(H, n) do { H[0] = (n[0] & 0x87878787u) + lb4_; H[1] = (rotr4(n[0]) & 0x87878787u) + lb4_; } while (0)
#define HI23(H, n) do { H[2] = (n[1] & 0x87878787u) + lb4_; H[3] = (rotr4(n[1]) & 0x87878787u) + lb4_; } while (0)
#define PE(H, l, i) perm(H[i], l[i], 0x05010400u)
#define PO(H, l, i) perm(H[i], l[i], 0x07030602u)
#define XPAIR(H, l, i) (u32x4){PE(H, l, i), PO(H, l, i), PE(H, l, (i) + 1), PO(H, l, (i) + 1)}
#define LBREP(h) perm(0u, (h) & 0x7fu, 0u)
#define FIXBITS(h) fixbits(__builtin_amdgcn_ubfe(h, fsh, 3u) | (((h) >> 11) & 4u))
#define WAIT_STEP(sg, J, seen) do { if (sg) wait_step(lds, (J) + 1, seen); } while (0)
#define GRS_STEP4(s, uc, hc, lbc, fxc, evx, sg, J) do { \
        constexpr int sn_ = ((s) + 1) & 3; \
        StepRegs<NC>& ri_ = R[s]; StepRegs<NC>& rc_ = R[sn_]; \
        u32x4 (&xo_)[4] = X[(s) & 1]; u32x4 (&xn_)[4] = X[sn_ & 1]; \
        u32 H_[4], lb4_, seen_; u32x4 e_, o_; \
        LAS unsigned char* wp_ = wdst + sn_ * SLOTB; \
        SB; \
        if ((s) == 0) ald8(evn, EVADDR(hn), rsw, ue); \
        MF(0, wf[1], xn_[3]); RD1(s, 0, 0); ld_w(ri_, rsl, oW[s], 0); lb4_ = rdl(lbc, 8 + sn_); SB; \
        vm_wait<2 * NL + 1>(rc_, evx); \
        MF(1, wf[1], xn_[3]); RD1(s, 0, 1); HI01(H_, rc_.wn); SB; \
        MF(2, wf[1], xn_[3]); RD1(s, 0, 2); ld_w(ri_, rsl, oW[s], 1); HI23(H_, rc_.wn); SB; \
        MF(3, wf[1], xn_[3]); RD1(s, 0, 3); \
            e_[0] = PE(H_, rc_.wl, 0); e_[1] = PE(H_, rc_.wl, 1); e_[2] = PE(H_, rc_.wl, 2); e_[3] = PE(H_, rc_.wl, 3); SB; \
        MF(0, wf[0], xo_[0]); RD1(s, 1, 0); ald8(ri_.xa.n, vn, rsl, oA[s]); \
            *(LAS u32x4*)wp_ = e_; o_[0] = PO(H_, rc_.wl, 0); o_[1] = PO(H_, rc_.wl, 1); o_[2] = PO(H_, rc_.wl, 2); o_[3] = PO(H_, rc_.wl, 3); SB; \
        MF(1, wf[0], xo_[0]); RD1(s, 1, 1); \
            *(LAS u32x4*)(wp_ + 1024) = o_; lb4_ = rdl(lbc, 2 * sn_); HI01(H_, rc_.xa.n); SB; \
        FIX_W(fxc, evx, hc, uc, sn_); \
        SB; \
        sig_fix<((s) + 1) * 4, 8 + sn_>(cntv, sg, (fxc).fx, wlb + (evx[0] >> 16), evx[0]); \
        MF(2, wf[0], xo_[0]); RD1(s, 1, 2); ald16(ri_.xa.l, vl, rsl, oA[s]); HI23(H_, rc_.xa.n); SB; \
        MF(3, wf[0], xo_[0]); RD1(s, 1, 3); xn_[0] = XPAIR(H_, rc_.xa.l, 0); SB; \
        MF(0, wf[1], xo_[1]); RD1(s, 2, 0); ald8(ri_.xb.n, vn, rsl, oB[s]); xn_[1] = XPAIR(H_, rc_.xa.l, 2); SB; \
        MF(1, wf[1], xo_[1]); RD1(s, 2, 1); lb4_ = rdl(lbc, 2 * sn_ + 1); HI01(H_, rc_.xb.n); SB; \
        MF(2, wf[1], xo_[1]); RD1(s, 2, 2); ald16(ri_.xb.l, vl, rsl, oB[s]); HI23(H_, rc_.xb.n); SB; \
        MF(3, wf[1], xo_[1]); RD1(s, 2, 3); xn_[2] = XPAIR(H_, rc_.xb.l, 0); SB; \
        MF(0, wf[0], xo_[2]); RD1(s, 3, 0); xn_[3] = XPAIR(H_, rc_.xb.l, 2); SB; \
        MF(1, wf[0], xo_[2]); RD1(s, 3, 1); seen_ = *(volatile LAS u32*)(cntp + (s) + 1); SB; \
        MF(2, wf[0], xo_[2]); RD1(s, 3, 2); SB; \
        MF(3, wf[0], xo_[2]); RD1(s, 3, 3); SB; \
        FIX_X(fxc, evx, hc, uc, sn_, xn_); \
        WAIT_STEP(sg, J, seen_); \
        } while (0)
#define GRS_STEP2(s, uc, hc, lbc, fxc, evx, sg, J) do { \
        constexpr int sn_ = ((s) + 1) & 3; \
        StepRegs<NC>& ri_ = R[s]; StepRegs<NC>& rc_ = R[sn_]; \
        u32x4 (&xo_)[4] = X[(s) & 1]; u32x4 (&xn_)[4] = X[sn_ & 1]; \
        u32 H_[4], lb4_, seen_; \
        LAS unsigned char* wp_ = wdst + sn_ * SLOTB; \
        SB; \
        if ((s) == 0) ald8(evn, EVADDR(hn), rsw, ue); \
        vm_wait<2 * NL>(rc_, evx); \
        MF(0, wf[1], xn_[3]); RD1(s, 0, 0); ld_w(ri_, rsl, oW[s], 0); \
            lb4_ = rdl(lbc, 8 + sn_); H_[0] = (rc_.w[2] & 0x87878787u) + lb4_; H_[1] = (rotr4(rc_.w[2]) & 0x87878787u) + lb4_; SB; \
        MF(1, wf[1], xn_[3]); RD1(s, 0, 1); ald8(ri_.xa.n, vn, rsl, oA[s]); \
            *(LAS u32x2*)wp_ = (u32x2){PE(H_, rc_.w, 0), PE(H_, rc_.w, 1)}; *(LAS u32x2*)(wp_ + 1024) = (u32x2){PO(H_, rc_.w, 0), PO(H_, rc_.w, 1)}; SB; \
        FIX_W(fxc, evx, hc, uc, sn_); \
        SB; \
        sig_fix<((s) + 1) * 4, 8 + sn_>(cntv, sg, (fxc).fx, wlb + (evx[0] >> 16), evx[0]); \
        MF(0, wf[0], xo_[0]); RD1(s, 1, 0); lb4_ = rdl(lbc, 2 * sn_); HI01(H_, rc_.xa.n); SB; \
        MF(1, wf[0], xo_[0]); RD1(s, 1, 1); ald16(ri_.xa.l, vl, rsl, oA[s]); HI23(H_, rc_.xa.n); xn_[0] = XPAIR(H_, rc_.xa.l, 0); SB; \
        MF(0, wf[1], xo_[1]); RD1(s, 2, 0); ald8(ri_.xb.n, vn, rsl, oB[s]); xn_[1] = XPAIR(H_, rc_.xa.l, 2); lb4_ = rdl(lbc, 2 * sn_ + 1); SB; \
        MF(1, wf[1], xo_[1]); RD1(s, 2, 1); ald16(ri_.xb.l, vl, rsl, oB[s]); \
            HI01(H_, rc_.xb.n); seen_ = *(volatile LAS u32*)(cntp + (s) + 1); SB; \
        MF(0, wf[0], xo_[2]); RD1(s, 3, 0); HI23(H_, rc_.xb.n); xn_[2] = XPAIR(H_, rc_.xb.l, 0); SB; \
        MF(1, wf[0], xo_[2]); RD1(s, 3, 1); xn_[3] = XPAIR(H_, rc_.xb.l, 2); SB; \
        FIX_X(fxc, evx, hc, uc, sn_, xn_); \
        WAIT_STEP(sg, J, seen_); \
        } while (0)
#define GRS_STEP(s, uc, hc, lbc, fxc, evx, sg, J) do { { const u32x4 rsl = rsv; if constexpr (NC == 4) GRS_STEP4(s, uc, hc, lbc, fxc, evx, sg, J); else GRS_STEP2(s, uc, hc, lbc, fxc, evx, sg, J); } } while (0)
    const u32 notag = ((tag + 1u) & TAGMASK) << TAGSH;
    const u32 hfo = OFF_HFLAG + hid * 128u + (u32)w * 4u;
    u32 hv = 0; bool cold = true, pf_ok = false;
    LOOK_SET("v228", notag); LOOK_SET("v229", notag);
    for (u32 u = ubeg; u < uend; ++u) {
        const bool nx = u + 1u < uend;
        const u32 J = 4u * u, cntv = cnt0 + J * 4u, un = nx ? u + 1u : u;
        LAS u32* cntp = (LAS u32*)(lds + LDS_CNT) + J;
        u32 pv; look_get<2 * NL>(pv, 0);
        if (!pf_ok) pv = notag;
        if (RARE(cold)) {
            look_get<NL>(hv, 1);
            if (ok && !__all((hv >> TAGSH) == tag)) {
                ok = wait_tags(my + fo + u * 128u, tag, hv, my, call);
                if (!ok) *(volatile LAS u32*)(lds + LDS_CTL + 24) = 1u;
            }
            STAMP(6 + 2 * u); STAMP2(u, 4);
            const u32x4 rsu = unit_rsrc(u, true);
            issue(0, rsu, u, hv); issue(1, rsu, u, hv);
            LOOK_LD("v230", fo, rsw, un * 128u);
            const FixBits f0 = FIXBITS(hv);
            const u32 lb0 = LBREP(hv), fx0 = f0.fx | (f0.two << 16);
            u32 H_[4], lb4_ = rdl(lb0, 8);
            vm_wait<NL + 1>(R[0], evn);
            evc = evn;
            if constexpr (NC == 4) {
                HI01(H_, R[0].wn); HI23(H_, R[0].wn);
                *(LAS u32x4*)wdst = (u32x4){PE(H_, R[0].wl, 0), PE(H_, R[0].wl, 1), PE(H_, R[0].wl, 2), PE(H_, R[0].wl, 3)};
                *(LAS u32x4*)(wdst + 1024) = (u32x4){PO(H_, R[0].wl, 0), PO(H_, R[0].wl, 1), PO(H_, R[0].wl, 2), PO(H_, R[0].wl, 3)};
            } else {
                H_[0] = (R[0].w[2] & 0x87878787u) + lb4_; H_[1] = (rotr4(R[0].w[2]) & 0x87878787u) + lb4_;
                *(LAS u32x2*)wdst = (u32x2){PE(H_, R[0].w, 0), PE(H_, R[0].w, 1)}; *(LAS u32x2*)(wdst + 1024) = (u32x2){PO(H_, R[0].w, 0), PO(H_, R[0].w, 1)};
            }
            if (fx0 & 0x1000100u) fix_w(fx0, evc[0], hv, u, 0);
            sig_step<0>(cntv, 1u);
            issue(2, rsu, u, hv); issue(3, rsu, u, hv);
            lb4_ = rdl(lb0, 0); HI01(H_, R[0].xa.n); HI23(H_, R[0].xa.n); X[0][0] = XPAIR(H_, R[0].xa.l, 0); X[0][1] = XPAIR(H_, R[0].xa.l, 2);
            lb4_ = rdl(lb0, 1); HI01(H_, R[0].xb.n); HI23(H_, R[0].xb.n); X[0][2] = XPAIR(H_, R[0].xb.l, 0); X[0][3] = XPAIR(H_, R[0].xb.l, 2);
            if (fx0 & 0x30003u) fix_x(fx0, evc[0], hv, u, 0, X[0]);
            look_get<2 * NL>(pv, 2);
            STAMP2(8 + u, 0);
            wait_step(lds, J, 0u);
            STAMP2(16 + u, 0);
        }
        bool have = ok && nx && __all((pv >> TAGSH) == tag);
        u32 hn = pv;
        const bool look = ok && nx;
        if (have) STAMP2(un, 0);
        const u32x4 rsn = unit_rsrc(un, true);
        u32x4 rsv = rsn; rsv[2] = have ? WS_NEED : 0u;
        const u32 ue = un * 1024u;
        const u32 lbc = LBREP(hv); const FixBits fxc = FIXBITS(hv);
#define LATE(s) do { if (RARE(!have && look)) { u32 v_; look_get<NL>(v_, 1); \
            if (__all((v_ >> TAGSH) == tag)) { have = true; hn = v_; STAMP2(un, s); rsv[2] = WS_NEED; \
                issue(0, rsn, un, hn); if ((s) > 1) issue(1, rsn, un, hn); if ((s) > 2) issue(2, rsn, un, hn); } \
            else LOOK_LD("v229", fo, rsw, un * 128u); } } while (0)
        if (RARE(!have && look)) LOOK_LD("v229", fo, rsw, un * 128u);
        GRS_STEP(0, u, hv, lbc, fxc, evc, 1u, J);
        LATE(1);
        GRS_STEP(1, u, hv, lbc, fxc, evc, 1u, J + 1);
        LOOK_LD("v228", fo, rsw, (u + 2u < uend ? u + 2u : uend - 1u) * 128u); pf_ok = have;
        LATE(2);
        GRS_STEP(2, u, hv, lbc, fxc, evc, 1u, J + 2);
        LATE(3);
        if constexpr (NC == 2 && KSPLIT < 8u) { if (RARE(role == 1u && !nx)) LOOK_LD("v230", hfo, rsw, 0u); }
        STAMP2(24 + u, 0);
        { const u32 lbn = LBREP(hn); const FixBits fxn = have ? FIXBITS(hn) : FixBits{0u, 0u}; GRS_STEP(3, un, hn, lbn, fxn, evn, (have ? 1u : 0u), J + 3); }
#undef LATE
        evc = evn;
        hv = hn; cold = !have;
        STAMP(7 + 2 * u);
        if (u == 0) STAMPC(28);
    }
    STAMPC(29);
    asm volatile("s_waitcnt vmcnt(0)" : "+v"(evn));
    reg_hold(R[0]); reg_hold(R[1]); reg_hold(R[2]); reg_hold(R[3]);
#pragma unroll
    for (int c = 0; c < NC; ++c) MF(c, wf[1], X[1][3]);
#undef GRS_STEP
#undef GRS_STEP2
#undef GRS_STEP4
    STAMP(2);
    LAS u32* ctl = (LAS u32*)(lds + LDS_CTL);
    if (!ok || rfl(__hip_atomic_load(ctl + 6, __ATOMIC_RELAXED, __HIP_MEMORY_SCOPE_WORKGROUP)) != 0u) return;
    if constexpr (NC == 2 && KSPLIT < 8u) {
        const rsrc_t hx = mk_rsrc(my, WS_NEED);
        const u32 ho = OFF_HX + (hid * 8u + (u32)w) * 8192u; char* hf = my + OFF_HFLAG + hid * 128u + (u32)w * 4u;
        if (role == 2u) {
#pragma unroll
            for (int i = 0; i < 8; ++i) {
                const f32x16 a = acc[i >> 2]; const int g = 4 * (i & 3);
                st16(hx, vo, ho + 1024u * i, (u32x4){__float_as_uint(a[g]), __float_as_uint(a[g + 1]), __float_as_uint(a[g + 2]), __float_as_uint(a[g + 3])});
            }
            drain();
            if (lane == 0) sys_store(hf, tag << TAGSH);
            STAMP(4);
            return;
        }
        if (role == 1u) {
            u32 hw_; look_get<0>(hw_, 2);
            if (!__all((hw_ >> TAGSH) == tag) && !wait_tags(hf, tag, hw_, my, call)) return;
            u32x4 o[8];
#pragma unroll
            for (int i = 0; i < 8; ++i) o[i] = ld16(hx, vo, ho + 1024u * i);
#pragma unroll
            for (int i = 0; i < 8; ++i)
#pragma unroll
                for (int j = 0; j < 4; ++j) acc[i >> 2][4 * (i & 3) + j] += __uint_as_float(o[i][j]);
        }
    }
    const rsrc_t dst = mk_rsrc(P.ws[q], WS_NEED);
    const u32 tau = ((me & 1u) * 2u + ((me >> 1) & 1u)) * 16u + (u32)n, k = me >> 2;
    u32 hd[2] = {0u, 0u};
    u32 L[NC / 2][8], H[NC / 2][8]; u32x4 np[NC / 2]; u64 bm[NC / 2];
#pragma unroll
    for (int b2 = 0; b2 < NC / 2; ++b2) {
#pragma unroll
        for (int t = 0; t < 8; ++t) {
            u32 d0, d1;
            if constexpr (NC == 4) { const int g = 8 * b2 + t; d0 = cvt_pk_bf16(acc[0][g], acc[1][g]); d1 = cvt_pk_bf16(acc[2][g], acc[3][g]); }
            else { d0 = cvt_pk_bf16(acc[0][2 * t], acc[1][2 * t]); d1 = cvt_pk_bf16(acc[0][2 * t + 1], acc[1][2 * t + 1]); }
            L[b2][t] = perm(d1, d0, 0x06040200u); H[b2][t] = perm(d1, d0, 0x07050301u);
        }
    }
    const bool first = GRS_LOWFIRST && (u32)w < PUSH_K + GRS_LOWX;
    if (first) {
#pragma unroll
        for (int b2 = 0; b2 < NC / 2; ++b2) {
            const u32 blk = ((k * 64 + tau) * 4 + b) * 4 + hb * 2u + (NC == 4 ? (u32)b2 : ht);
            st16(dst, vo, OFF_PP + blk * BLK, (u32x4){L[b2][0], L[b2][1], L[b2][2], L[b2][3]});
            st16(dst, vo, OFF_PP + blk * BLK + 1024u, (u32x4){L[b2][4], L[b2][5], L[b2][6], L[b2][7]});
        }
    }
    float mx = 0.f;
#pragma unroll
    for (int c = 0; c < NC; ++c)
#pragma unroll
        for (int g = 0; g < 16; g += 2) mx = fmaxf(mx, fmaxf(fabsf(acc[c][g]), fabsf(acc[c][g + 1])));
    const u32 em = wave_umax((cvt_pk_bf16(mx, mx) >> 8) & 0x7fu);
    const u32 lb = em > 7u ? em - 7u : 0u;
#pragma unroll
    for (int b2 = 0; b2 < NC / 2; ++b2) bm[b2] = pack_prep(H[b2], lb, np[b2]);
    if ((u32)w >= PUSH_K) while (rfl(__hip_atomic_load(ctl + 8, __ATOMIC_RELAXED, __HIP_MEMORY_SCOPE_WORKGROUP)) + PUSH_K <= (u32)w) __builtin_amdgcn_s_sleep(GRS_TURNSLEEP);
#pragma unroll
    for (int b2 = 0; b2 < NC / 2; ++b2) {
        const u32 beta = hb * 2u + (NC == 4 ? (u32)b2 : ht);
        const u32 blk = ((k * 64 + tau) * 4 + b) * 4 + beta;
        hd[b2] = pack_emit(L[b2], H[b2], lb, np[b2], bm[b2], dst, OFF_PP + blk * BLK, OFF_PE + blk * ENT, OFF_PR + blk * RAWB, lane, false, false, first);
    }
    STAMP(3);
    asm volatile("s_waitcnt vmcnt(%0)" :: "n"(NC == 4 ? GRS_RELW : GRS_RELN) : "memory");
    if (PUSH_K < 8u && lane == 0) __hip_atomic_fetch_add(ctl + 8, 1u, __ATOMIC_RELAXED, __HIP_MEMORY_SCOPE_WORKGROUP);
    drain();
    if (lane < NC / 2)
        sys_store(P.ws[q] + OFF_PFLAG + (tau * 4 + b) * 128u + (k * 4 + hb * 2 + (NC == 4 ? (u32)lane : ht)) * 4u, (tag << TAGSH) | (lane == 0 ? hd[0] : hd[1]));
    STAMP(4);
}

struct Blk { Half h[2]; };
__device__ __forceinline__ void unpack_f32(const Blk& k, u32 hdr, u32 entv, u32 ebase, rsrc_t ws, u32 raw_off, float (&f)[32], int lane) {
#pragma unroll
    for (int hb = 0; hb < 2; ++hb) {
        u32 H[4]; high_bytes(k.h[hb].n, (u32)hb, hdr, entv, ebase, H, lane);
#pragma unroll
        for (int i = 0; i < 4; ++i) {
            const int t = 4 * hb + i;
            f[4 * t]     = __uint_as_float(perm(H[i], k.h[hb].l[i], 0x04000c0cu));
            f[4 * t + 1] = __uint_as_float(perm(H[i], k.h[hb].l[i], 0x05010c0cu));
            f[4 * t + 2] = __uint_as_float(perm(H[i], k.h[hb].l[i], 0x06020c0cu));
            f[4 * t + 3] = __uint_as_float(perm(H[i], k.h[hb].l[i], 0x07030c0cu));
        }
    }
    if (hdr & RAWBIT) {
#pragma unroll
        for (int c = 0; c < 4; ++c) { const u32x4 o = ld16(ws, (u32)lane * 16u, raw_off + 1024u * c);
#pragma unroll
            for (int i = 0; i < 4; ++i) { f[8 * c + 2 * i] = __uint_as_float(o[i] << 16); f[8 * c + 2 * i + 1] = __uint_as_float(o[i] & 0xffff0000u); } }
    }
}

__device__ __forceinline__ void reducer_wave(const Params& P, int be, int lane, u32 call, u32 tag, bool ok, LAS unsigned char* lds, char* my, int wave) {
    const int wg = blockIdx.x;
    const u32 tau = (u32)wg >> 2, wm = (u32)wg & 3u;
    const rsrc_t ws = mk_rsrc(my, WS_NEED);
    LAS u32* ctl = (LAS u32*)(lds + LDS_CTL);
    u32 hv = 0;
    if (ok) ok = wait_tags(my + OFF_PFLAG + (tau * 4 + wm) * 128u + (u32)((lane & 1) * 4 + be) * 4u, tag, hv, my, call);
    STAMP(22);
    float y[32];
    {
        const u32 b0 = ((0 * 64 + tau) * 4 + wm) * 4 + (u32)be, b1 = ((1 * 64 + tau) * 4 + wm) * 4 + (u32)be;
        const u32 vo = (u32)lane * 16u;
        const u32 von = (u32)(lane & 31) * 16u + (u32)(lane >> 5) * 8u;
        Blk k0, k1;
#pragma unroll
        for (int hb = 0; hb < 2; ++hb) {
            k0.h[hb].n = ld8(ws, von, OFF_PP + b0 * BLK + 2048u + hb * 512u); k0.h[hb].l = ld16(ws, vo, OFF_PP + b0 * BLK + hb * 1024u);
            k1.h[hb].n = ld8(ws, von, OFF_PP + b1 * BLK + 2048u + hb * 512u); k1.h[hb].l = ld16(ws, vo, OFF_PP + b1 * BLK + hb * 1024u);
        }
        const u32 ev = ld4(ws, (lane < 8 ? OFF_PE + b0 * ENT : OFF_PE + b1 * ENT) + (u32)(lane & 7) * 4u, 0);
        cfence();
        float z[32];
        unpack_f32(k0, rdl(hv, 0), ev, 0, ws, OFF_PR + b0 * RAWB, y, lane);
        unpack_f32(k1, rdl(hv, 1), ev, 8, ws, OFF_PR + b1 * RAWB, z, lane);
#pragma unroll
        for (int e = 0; e < 32; ++e) y[e] += z[e];
    }
    float mx = 0.f;
#pragma unroll
    for (int e = 0; e < 32; e += 2) mx = fmaxf(mx, fmaxf(fabsf(y[e]), fabsf(y[e + 1])));
    u32 mb = __float_as_uint(mx);
    mb = wave_umax(mb);
    STAMP(23);
    u32 last = 0;
    if (lane == 0) {
        __hip_atomic_fetch_max(ctl + 0, mb, __ATOMIC_RELAXED, __HIP_MEMORY_SCOPE_WORKGROUP);
        if (!ok) __hip_atomic_store(ctl + 4, 1u, __ATOMIC_RELAXED, __HIP_MEMORY_SCOPE_WORKGROUP);
        last = __hip_atomic_fetch_add(ctl + 1, 1u, __ATOMIC_ACQ_REL, __HIP_MEMORY_SCOPE_WORKGROUP);
    }
    last = rfl(last);
    u32 gbits = 0; bool good;
    if (last == 3u) {
        const u32 wmax = rfl(__hip_atomic_load(ctl + 0, __ATOMIC_RELAXED, __HIP_MEMORY_SCOPE_WORKGROUP));
        const u32 fail = rfl(__hip_atomic_load(ctl + 4, __ATOMIC_RELAXED, __HIP_MEMORY_SCOPE_WORKGROUP));
        good = fail == 0u;
        if (good) {
            if (lane == 0) sys_store64(my + OFF_LG + (u32)wg * 8u, ((u64)call << 32) | wmax);
            const u64 t0 = __builtin_amdgcn_s_memrealtime();
            for (;;) {
                u64 x = 0;
                if (lane < 8) x = sys_load64(my + OFF_GG + (u32)(wg & 7) * 128u + (u32)lane * 8u);
                else if (lane == 8) x = (u64)agent_load(my + OFF_TMO);
                const bool tm = __any(lane == 8 && (u32)x == call);
                const bool all = __all(lane >= 8 || (u32)(x >> 32) == call);
                if (tm) { good = false; break; }
                if (all) { gbits = wave_umax(lane < 8 ? (u32)x : 0u); break; }
                if (__builtin_amdgcn_s_memrealtime() - t0 > DEADLINE) { agent_store(my + OFF_TMO, call); good = false; break; }
                __builtin_amdgcn_s_sleep(1);
            }
        }
        if (lane == 0) {
            __hip_atomic_store(ctl + 2, gbits, __ATOMIC_RELAXED, __HIP_MEMORY_SCOPE_WORKGROUP);
            __hip_atomic_store(ctl + 3, good ? 1u : 2u, __ATOMIC_RELEASE, __HIP_MEMORY_SCOPE_WORKGROUP);
        }
    } else {
        u32 st;
        while ((st = rfl(__hip_atomic_load(ctl + 3, __ATOMIC_ACQUIRE, __HIP_MEMORY_SCOPE_WORKGROUP))) == 0u) __builtin_amdgcn_s_sleep(1);
        good = st == 1u;
        gbits = rfl(__hip_atomic_load(ctl + 2, __ATOMIC_RELAXED, __HIP_MEMORY_SCOPE_WORKGROUP));
    }
    STAMP(24);
    const float amax = __uint_as_float(gbits);
    const float scale = amax / 448.0f;
    const float inv = 1.0f / scale;
    u32x4 o0, o1;
#pragma unroll
    for (int i = 0; i < 8; ++i) {
        u32 v = (u32)__builtin_amdgcn_cvt_pk_fp8_f32(y[4 * i] * inv, y[4 * i + 1] * inv, 0, false);
        v = (u32)__builtin_amdgcn_cvt_pk_fp8_f32(y[4 * i + 2] * inv, y[4 * i + 3] * inv, (int)v, true);
        if (!good) v = 0x7f7f7f7fu;
        if (i < 4) o0[i] = v; else o1[i - 4] = v;
    }
    const u32 qi = tau >> 5, qj = (tau >> 4) & 1u, n = tau & 15u;
    const u32 row = qi * 256u + wm * 64u + (u32)(be >> 1) * 32u + (u32)(lane & 31);
    const u32 col = qj * 2048u + n * 128u + (is_narrow(n) ? (u32)(be & 1) * 64u + (u32)(lane >> 5) * 32u : (u32)(lane >> 5) * 64u + (u32)(be & 1) * 32u);
    const rsrc_t out = mk_rsrc(P.out, 2u << 20);
    __builtin_amdgcn_raw_buffer_store_b128(o0, out, (int)(row * 4096u + col), 0, GRS_OUTAUX);
    __builtin_amdgcn_raw_buffer_store_b128(o1, out, (int)(row * 4096u + col + 16u), 0, GRS_OUTAUX);
    STAMP(25); STAMPC(31);
}

#define GRS_LEADERS 1
static_assert(GRS_LEADERS >= 1 && GRS_LEADERS <= 4, "waves 0..3 of workgroup 0");
__device__ __forceinline__ void leader_wave(const Params& P, int lane, u32 call, char* my, int wave) {
    const u64 t0 = __builtin_amdgcn_s_memrealtime();
    u32 m = 0; bool good = true;
    for (int i = 0; i < wave; ++i) __builtin_amdgcn_s_sleep(7);
    for (u32 it = 0;; ++it) {
        bool all = true; m = 0;
#pragma unroll
        for (int i = 0; i < 4; ++i) { const u64 x = sys_load64(my + OFF_LG + (u32)(i * 64 + lane) * 8u); all = all && (u32)(x >> 32) == call; m = max(m, (u32)x); }
        if (__all(all)) break;
        if ((it & 15u) == 15u && (rfl(agent_load(my + OFF_TMO)) == call || __builtin_amdgcn_s_memrealtime() - t0 > DEADLINE)) {
            agent_store(my + OFF_TMO, call); good = false; break; }
        __builtin_amdgcn_s_sleep(1);
    }
    STAMP(26);
    if (good) {
        m = wave_umax(m);
        sys_store64(P.ws[lane >> 3] + OFF_GG + (u32)(lane & 7) * 128u + (u32)P.rank * 8u, ((u64)call << 32) | m);
    }
    if (lane == 0) agent_store(my + OFF_CALL, call);
}

__global__ void __launch_bounds__(512, 2) gemm_rs_fused(Params P) {
    __shared__ __attribute__((aligned(16))) unsigned char smem[LDS_BYTES];
    LAS unsigned char* lds = (LAS unsigned char*)smem;
    const int tid = threadIdx.x, lane = tid & 63, wave = (int)rfl((u32)tid >> 6);
    const int b = blockIdx.x, xcd = b & 7, s = b >> 3;
    char* my = P.my;
    asm volatile("global_load_dword v231, %0, off sc1" :: "v"(my + OFF_CALL) : "v231");
    if (tid < 64) ((LAS u32*)(lds + LDS_CTL))[tid] = 0u;
    __syncthreads();
    auto get_call = [&]() { u32 c; asm volatile("s_waitcnt vmcnt(0)\n\tv_readfirstlane_b32 %0, v231" : "=s"(c)); return c + 1u; };
    STAMP(0); STAMPC(30);
    if (s < SEND_SLOTS && (u32)wave < SEND_WPW) {
        if (SEND_SLOTS == 8) sender_wave(P, s, (u32)xcd * SEND_WPW + (u32)wave, lane, my, get_call);
        else sender_wave(P, 2 * s + (xcd & 1), (u32)(xcd >> 1) * SEND_WPW + (u32)wave, lane, my, get_call);
    }
    const u32 call = get_call();
    const u32 tag = call & TAGMASK;
    int tg = 0, ts = s - SEND_SLOTS;
#pragma unroll
    for (int g = 0; g < 3; ++g) { const int w = ((NARROW_MASK >> g) & 1u) ? 8 : 4; if (tg == g && ts >= w) { ts -= w; tg = g + 1; } }
    const bool helper = KSPLIT < 8u && s < SEND_SLOTS;
    if (helper) { tg = 3; ts = s; }
    const u32 role = helper ? 2u : (KSPLIT < 8u && tg == 3 ? 1u : 0u);
    const int tq = 4 * (xcd & 1) + (ts & 3), tn = 4 * tg + (xcd >> 1);
#define GRS_TILES do { \
        if (is_narrow((u32)tn)) gemm_wave<2>(P, tq, tn, (u32)ts >> 2, wave, lane, call, tag, lds, my, wave, \
                                             role == 2u ? KSPLIT : 0u, role == 1u ? KSPLIT : 8u, role, (u32)(xcd * 8 + ts)); \
        else gemm_wave<4>(P, tq, tn, 0u, wave, lane, call, tag, lds, my, wave, 0u, 8u, 0u, 0u); } while (0)
    if (s < SEND_SLOTS) STAMP(1);
    if (helper) __syncthreads();
    if (s >= SEND_SLOTS || helper) GRS_TILES;
    if (b == 0 && wave < GRS_LEADERS) leader_wave(P, lane, call, my, wave);
    if (wave >= 4) reducer_wave(P, wave - 4, lane, call, tag, true, lds, my, wave);
}

}

extern "C" void kernel_launch(int rank, int nranks, ncclComm_t comm, hipStream_t stream,
                              void* const* d_in, const int* in_sizes, int n_in,
                              void* d_out, int out_size,
                              const int* in_dtypes, int out_dtype,
                              void* const* d_out_all,
                              void* const* d_ws_all, size_t ws_bytes) {
    (void)comm; (void)in_dtypes; (void)out_dtype; (void)d_out_all;
    if (nranks != grs::NR || n_in < 2 || ws_bytes < grs::WS_NEED || out_size != 512 * 4096 ||
        in_sizes[0] != 4096 * 512 || in_sizes[1] != 512 * 4096) {
        fprintf(stderr, "gemm_rs: unexpected problem geometry\n");
        return;
    }
    grs::Params p;
    p.x = d_in[0]; p.w = d_in[1]; p.out = d_out;
    for (int i = 0; i < grs::NR; ++i) p.ws[i] = (char*)d_ws_all[i];
    p.my = p.ws[rank];
    p.rank = rank; p.pad = 0;
    hipLaunchKernelGGL(grs::gemm_rs_fused, dim3(256), dim3(512), 0, stream, p);
    HIP_CHECK(hipGetLastError());
}
```
